```python
import numpy as np

kernel_name = "gpu_gemm_ar_m4096_k4096_n16384_bf16_gelu"


ABSMAX_RTOL = 0.008

COMPUTE_DTYPE = "bfloat16"
EPILOGUE = "gelu"
IN_QUANT = None
OUT_QUANT = None
IN_DTYPES = {"x": "bfloat16", "w_mat": "bfloat16"}
WORLD_SIZE = 8

def _to_bf16(x):
    u = np.ascontiguousarray(x, dtype=np.float32).view(np.uint32)
    return ((u + 0x7FFF + ((u >> 16) & 1)) & 0xFFFF0000).view(np.float32)

def setup_inputs():

    x = _to_bf16(np.random.randn(4096, 4096).astype(np.float32))
    w_mat = _to_bf16((np.random.randn(4096, 16384) / np.sqrt(4096)).astype(np.float32))
    return {"x": x, "w_mat": w_mat}

def _epilogue(y):
    c = 0.7978845608028654
    return (0.5 * y * (1.0 + np.tanh(c * (y + 0.044715 * y ** 3)))).astype(np.float32)

def reference(x, w_mat):
    y = (x @ w_mat).astype(np.float32)
    return _epilogue(y)
```

```cpp
#include <hip/hip_runtime.h>
#include <rccl/rccl.h>
#include <cstdio>
#include <cstdint>

#define HIP_CHECK(expr) do { hipError_t _e = (expr); if (_e != hipSuccess) { \
    fprintf(stderr, "HIP error %s at %s:%d\n", hipGetErrorString(_e), __FILE__, __LINE__); \
    return; } } while (0)

namespace {

#define LAS __attribute__((address_space(3)))
typedef short bf16x8 __attribute__((ext_vector_type(8)));
typedef float f32x4 __attribute__((ext_vector_type(4)));
typedef unsigned u32x4 __attribute__((ext_vector_type(4)));
typedef __attribute__((address_space(1))) unsigned gu32;

constexpr int M_ = 4096, K_ = 4096, N_ = 16384, P_ = 8, KS_ = K_ / P_;
#define ND_ 4
constexpr int ND = ND_;
constexpr int NCOM = 64 - 8 * ND;
constexpr int NWP = ND + NCOM;
constexpr int NXP = M_ / 256;
constexpr int CPD = 4;
constexpr int NCOPY = 8 * CPD;
constexpr int NWG = 256;
constexpr int GPX = NWG / 8 - CPD;
constexpr int WPR = GPX / 2;
constexpr int NROUND = (NWP + WPR - 1) / WPR;
constexpr int WLAST = NWP - (NROUND - 1) * WPR;
static_assert(WLAST % 2 == 0 && GPX % 4 == 0 && ND <= WPR, "schedule");
static_assert(NROUND <= 3, "schedule");
constexpr int CW = CPD * 8;
constexpr int wp_of(int r) { return r + 1 < NROUND ? WPR : (r + 1 == NROUND ? WLAST : 0); }
constexpr int NB0 = (NXP + wp_of(0)) * 4;
constexpr int NB1 = wp_of(1) * 4;
constexpr int NB2 = wp_of(2) * 4;
constexpr int S1 = 8 * NB0, S2 = S1 + 8 * NB1, S3 = S2 + 8 * NB2;
constexpr int NBO = M_ * ND * 512 / 8192;
constexpr int STEPS_IN = S3 / CW, STEPS_ALL = (S3 + NBO) / CW;
constexpr unsigned PSH = 128;
static_assert(S3 % CW == 0 && NBO % CW == 0 && STEPS_ALL < (int)PSH, "steps");
static_assert(ND == 4 || ND == 2 || ND == 8, "output batch shape");
__host__ __device__ constexpr int steps_for(int st) {
    return ((st < 8 ? (st + 1) * NB0 : st < 16 ? S1 + (st - 7) * NB1 : S2 + (st - 15) * NB2) + CW - 1) / CW;
}

constexpr unsigned long long pubmask() { unsigned long long m = 0; for (int st = 0; st < 8 * NROUND; ++st) m |= 1ull << steps_for(st); return m; }
constexpr unsigned long long PUBMASK = pubmask();
static_assert(STEPS_IN < 64, "publish mask");

constexpr size_t PANEL = 2u << 20;
constexpr size_t OFF_PROG = 0;
constexpr size_t OFF_TMO = 4096;
constexpr size_t OFF_DONE = 4096 + 256;
constexpr size_t OFF_CALL = 8192;
constexpr size_t OFF_DBG = 65536;
constexpr size_t OFF_IMG = 1u << 20;
constexpr size_t IMG_BYTES = (size_t)(NXP + NWP) * PANEL;
constexpr size_t WS_NEED = OFF_IMG + 2 * IMG_BYTES;
constexpr unsigned long long DEADLINE_TICKS = 100000000ull;

constexpr int HTB = 16384;
constexpr int LDS_BYTES = 8 * HTB + 256;
constexpr int AUX_SC1 = 16, AUX_SYS = 17;
constexpr int RSRC3 = 0x00020000;

struct Params {
    const char* x; const char* w;
    char* out[P_]; char* ws[P_];
    int rank;
};

__device__ __forceinline__ unsigned ld_sys(const char* p) { return __hip_atomic_load((gu32*)p, __ATOMIC_RELAXED, __HIP_MEMORY_SCOPE_SYSTEM); }
__device__ __forceinline__ unsigned ld_agent(const char* p) { return __hip_atomic_load((gu32*)p, __ATOMIC_RELAXED, __HIP_MEMORY_SCOPE_AGENT); }
__device__ __forceinline__ void st_sys(char* p, unsigned v) { __hip_atomic_store((gu32*)p, v, __ATOMIC_RELAXED, __HIP_MEMORY_SCOPE_SYSTEM); }
__device__ __forceinline__ void st_agent(char* p, unsigned v) { __hip_atomic_store((gu32*)p, v, __ATOMIC_RELAXED, __HIP_MEMORY_SCOPE_AGENT); }
__device__ __forceinline__ unsigned uni(unsigned v) { return (unsigned)__builtin_amdgcn_readfirstlane((int)v); }

#define STAMP(ws, idx) do { } while (0)

__device__ __forceinline__ unsigned wave_wait_ge(const char* lane_word, unsigned need, char* tmo, unsigned call) {
    const unsigned long long t0 = __builtin_amdgcn_s_memrealtime();
    unsigned v;
    for (;;) {
        v = ld_sys(lane_word);
        if (__all(v >= need)) break;
        if (uni(ld_agent(tmo)) == call || __builtin_amdgcn_s_memrealtime() - t0 > DEADLINE_TICKS) {
            if ((threadIdx.x & 63) == 0) st_agent(tmo, call);
            return 0u;
        }
        __builtin_amdgcn_s_sleep(8);
    }
#pragma unroll
    for (int o = 32; o >= 1; o >>= 1) { const unsigned u = (unsigned)__shfl_xor((int)v, o); v = u < v ? u : v; }
    return uni(v);
}

__device__ __forceinline__ unsigned cvt_pk_bf16(float lo, float hi) { unsigned r; asm("v_cvt_pk_bf16_f32 %0, %1, %2" : "=v"(r) : "v"(lo), "v"(hi)); return r; }
__device__ __forceinline__ float gelu(float y) {
    constexpr float k1 = -2.0f * 0.7978845608028654f * 1.4426950408889634f, k2 = k1 * 0.044715f;
    const float t = y * __builtin_fmaf(y * y, k2, k1);
    return y * __builtin_amdgcn_rcpf(1.0f + __builtin_amdgcn_exp2f(t));
}

struct Batch {
    int type;
    int stage;
    unsigned so, s0, s1, s2;
    unsigned la, lb;
    unsigned dof, d0, d1, d2;
};
__device__ __forceinline__ int stage_of(int n) {
    if (n < S1) return n / NB0;
    if (NB1 > 0 && n < S2) return 8 + (n - S1) / (NB1 > 0 ? NB1 : 1);
    if (NB2 > 0 && n < S3) return 16 + (n - S2) / (NB2 > 0 ? NB2 : 1);
    return 24;
}
__device__ __forceinline__ Batch decode(int n, int rank, int dst, unsigned img) {
    Batch b;
    b.stage = stage_of(n);
    if (n >= S3) {
        const int o = n - S3;
        b.type = 2; b.la = 16; b.lb = 256;
        constexpr unsigned RB = ND * 512;
        constexpr int ROWS = 8192 / RB;
        b.so = (unsigned)(o * ROWS) * (N_ * 2u) + (unsigned)rank * RB;
        if (ND == 4) { b.s0 = 1024; b.s1 = N_ * 2; b.s2 = N_ * 4; }
        else if (ND == 2) { b.s0 = N_ * 2; b.s1 = N_ * 4; b.s2 = N_ * 8; }
        else { b.s0 = 1024; b.s1 = 2048; b.s2 = N_ * 2; }
        b.dof = b.so; b.d0 = b.s0; b.d1 = b.s1; b.d2 = b.s2;
        return b;
    }
    const int g = b.stage & 7, r = b.stage >> 3;
    const int idx = n - (r == 0 ? g * NB0 : r == 1 ? S1 + g * NB1 : S2 + g * NB2);
    const int sub = idx & 3;
    int pan = idx >> 2;
    const unsigned kt_off = (unsigned)(g * 8 + rank) * 32768u;
    if (r == 0 && pan < NXP) {
        b.type = 0; b.la = KS_ * 2; b.lb = 16;
        b.so = (unsigned)(pan * 256 + sub * 64) * (KS_ * 2u) + (unsigned)g * 128u;
        b.s0 = 64; b.s1 = 16 * KS_ * 2; b.s2 = 32 * KS_ * 2;
        b.dof = img + (unsigned)pan * (unsigned)PANEL + kt_off + (unsigned)sub * 4096u;
        b.d0 = 1024; b.d1 = 2048; b.d2 = 16384;
    } else {
        const int lp = (r == 0 ? pan - NXP : pan) + r * WPR;
        const int cb = lp < ND ? dst * ND + lp : 8 * ND + (lp - ND);
        const int wn = sub >> 1, ks = sub & 1;
        b.type = 1; b.la = 16; b.lb = 8 * N_ * 2;
        b.so = (unsigned)(g * 64 + ks * 32) * (N_ * 2u) + (unsigned)(cb * 256 + wn * 128) * 2u;
        b.s0 = N_ * 2; b.s1 = N_ * 4; b.s2 = N_ * 8;
        b.dof = img + (unsigned)(NXP + lp) * (unsigned)PANEL + kt_off + (unsigned)wn * 8192u + (unsigned)ks * 1024u;
        b.d0 = 2048; b.d1 = 4096; b.d2 = 16384;
    }
    return b;
}

__device__ __forceinline__ void copier(const Params& p, LAS unsigned char* lds, unsigned call) {
    const int q = blockIdx.x & 7, j = blockIdx.x >> 3;
    const int rank = p.rank, dst = (rank + q) & 7;
    const int wid = (int)uni(threadIdx.x >> 6), lane = threadIdx.x & 63;
    char* const myws = p.ws[rank];
    char* const dws = p.ws[dst];
    const __amdgpu_buffer_rsrc_t rX = __builtin_amdgcn_make_buffer_rsrc((void*)p.x, 0, M_ * KS_ * 2, RSRC3);
    const __amdgpu_buffer_rsrc_t rW = __builtin_amdgcn_make_buffer_rsrc((void*)p.w, 0, KS_ * N_ * 2, RSRC3);
    const __amdgpu_buffer_rsrc_t rO = __builtin_amdgcn_make_buffer_rsrc((void*)p.out[rank], 0, M_ * N_ * 2, RSRC3);
    const __amdgpu_buffer_rsrc_t rDW = __builtin_amdgcn_make_buffer_rsrc((void*)dws, 0, (int)WS_NEED, RSRC3);
    const __amdgpu_buffer_rsrc_t rDO = __builtin_amdgcn_make_buffer_rsrc((void*)p.out[dst], 0, M_ * N_ * 2, RSRC3);
    const unsigned vL = (unsigned)lane * 16u;
    const unsigned img = (unsigned)OFF_IMG + (call & 1u) * (unsigned)IMG_BYTES;
    const int steps = q == 0 ? STEPS_IN : STEPS_ALL;
    LAS unsigned* bc = (LAS unsigned*)lds;
    char* const flag = dws + OFF_PROG + (unsigned)(lane & 7) * 256u + (unsigned)(rank * CPD + j) * 4u;
    const unsigned tiles_need = call * (unsigned)(NXP * ND * 8);

    auto issue_loads = [&](u32x4 (&v)[8], const Batch& b) {
        const __amdgpu_buffer_rsrc_t rs = b.type == 0 ? rX : (b.type == 1 ? rW : rO);
        const unsigned vo = (unsigned)(lane & 15) * b.la + (unsigned)(lane >> 4) * b.lb;
#pragma unroll
        for (int i = 0; i < 8; ++i)
            v[i] = __builtin_amdgcn_raw_buffer_load_b128(rs, vo, b.so + ((i & 1) ? b.s0 : 0u) + ((i & 2) ? b.s1 : 0u) + ((i & 4) ? b.s2 : 0u), AUX_SC1);
    };
    auto arrange = [&](u32x4 (&t)[8], const u32x4 (&v)[8], const Batch& b) {
        if (b.type == 1) {
#pragma unroll
            for (int c = 0; c < 8; ++c)
#pragma unroll
                for (int e = 0; e < 4; ++e)
                    t[c][e] = __builtin_amdgcn_perm(v[2 * e + 1][c >> 1], v[2 * e][c >> 1], (c & 1) ? 0x07060302u : 0x05040100u);
        } else {
#pragma unroll
            for (int c = 0; c < 8; ++c) t[c] = v[c];
        }
#pragma unroll
        for (int c = 0; c < 8; ++c) asm volatile("" : "+v"(t[c]));
    };
    auto issue_stores = [&](const u32x4 (&t)[8], const Batch& b) {
        const __amdgpu_buffer_rsrc_t rd = b.type == 2 ? rDO : rDW;
#pragma unroll
        for (int i = 0; i < 8; ++i)
            __builtin_amdgcn_raw_buffer_store_b128(t[i], rd, vL + (b.dof + ((i & 1) ? b.d0 : 0u) + ((i & 2) ? b.d1 : 0u) + ((i & 4) ? b.d2 : 0u)), 0, AUX_SYS);
    };
    auto publish = [&](int done) {
        if (wid == 0 && lane < 8) st_sys(flag, call * PSH + (unsigned)done);
    };
    constexpr int VM0 = 0x0f70, VM8_LGKM0 = 0x0078;

    u32x4 R[8], T[8];
    const int n0 = wid * CPD + j;
    Batch cur = decode(n0, rank, dst, img);
    issue_loads(R, cur);
    if (threadIdx.x == 0) bc[1] = 0u;
    __builtin_amdgcn_s_waitcnt(VM0);
    int pend = 0;
    for (int k = 0;; ++k) {
        arrange(T, R, cur);
        if (pend) publish(pend);
        if (wid == 0) STAMP(myws, 2 + k);
        const bool more = k + 1 < steps;
        bool early = more;
        if (k + 1 == STEPS_IN && more) early = uni(bc[0]) != 0u;
        const Batch nxt = decode(n0 + (more ? k + 1 : k) * CW, rank, dst, img);
        if (early) {
            unsigned d = 0;
            if (k + 2 == STEPS_IN && wid == 0) d = ld_agent(myws + OFF_DONE);
            issue_loads(R, nxt);
            issue_stores(T, cur);
            if (k + 2 == STEPS_IN && threadIdx.x == 0) bc[0] = d >= tiles_need ? 1u : 0u;
            asm volatile("" ::: "memory");
            __builtin_amdgcn_s_waitcnt(VM8_LGKM0);
            __builtin_amdgcn_s_barrier();
            asm volatile("" ::: "memory");
            pend = (k < 64 && ((PUBMASK >> k) & 1ull)) ? k : 0;
        } else {
            issue_stores(T, cur);
            asm volatile("" ::: "memory");
            __builtin_amdgcn_s_waitcnt(VM0);
            __builtin_amdgcn_s_barrier();
            asm volatile("" ::: "memory");
            if (uni(bc[1]) == 0u) publish(k + 1);
            if (!more) break;
            if (wave_wait_ge(myws + OFF_DONE, tiles_need, myws + OFF_TMO, call) == 0u && lane == 0) bc[1] = 1u;
            issue_loads(R, nxt);
            __builtin_amdgcn_s_waitcnt(VM0);
            pend = 0;
        }
        cur = nxt;
    }

    if (q == 0 && j == 0 && wid == 0) {
        int src = (lane & 31) / CPD;
        if (src == rank) src = (rank + 1) & 7;
        const char* word = myws + OFF_PROG + (unsigned)(src * CPD + (lane & (CPD - 1))) * 4u;
        const unsigned got = wave_wait_ge(word, call * PSH + (unsigned)STEPS_ALL, myws + OFF_TMO, call);
        __builtin_amdgcn_fence(__ATOMIC_ACQUIRE, "");
        if ((got == 0u || uni(ld_agent(myws + OFF_TMO)) == call) && lane == 0)
            *(unsigned short*)(p.out[rank] + (size_t)rank * ND * 512) = 0x7fc0u;
        STAMP(myws, 120);
    }
}

__device__ __forceinline__ bool unit_of(int r, int q, int i, int& rb, int& lp) {
    if (r >= NROUND) return false;
    const int half = (r + 1 < NROUND ? WPR : WLAST) / 2;
    if ((i >> 2) >= half) return false;
    rb = (q >> 1) * 4 + (i & 3);
    lp = r * WPR + ((i >> 2) * 2 + (q & 1));
    return true;
}

__device__ __forceinline__ void gemm(const Params& p, LAS unsigned char* lds, unsigned call) {
    const int tid = threadIdx.x, wid = (int)uni(tid >> 6), lane = tid & 63, wr = wid >> 2, wc = wid & 3;
    const int q = blockIdx.x & 7, gi = (blockIdx.x >> 3) - CPD;
    const int rank = p.rank;
    char* const myws = p.ws[rank];
    const char* const img = myws + OFF_IMG + (size_t)(call & 1u) * IMG_BYTES;
    const unsigned voff = (unsigned)tid * 16u;
    const unsigned ldsw = (unsigned)wid * 1024u;
    const int aoff = wr * 8192 + lane * 16, boff = wc * 4096 + lane * 16;
    LAS unsigned* bc = (LAS unsigned*)(lds + 8 * HTB);
    unsigned known = 0; bool bad = false;
    constexpr size_t kstep = 32768, hstep = 16384;
    constexpr int nt = K_ / 64;

#define SA(b, h) (((b) * 2 + (h)) * HTB)
#define SB(b, h) ((4 + (b) * 2 + (h)) * HTB)
#define STAGE(bufoff, gbase) do { _Pragma("unroll") for (int _i = 0; _i < 2; ++_i) \
        __builtin_amdgcn_global_load_lds((const unsigned*)((gbase) + voff + _i * 8192), (LAS unsigned*)(lds + (bufoff) + ldsw + _i * 8192), 16, 0, 0); } while (0)
#define LDA(dst, b, h) do { _Pragma("unroll") for (int m = 0; m < 4; ++m) _Pragma("unroll") for (int k = 0; k < 2; ++k) dst[m][k] = *(const LAS bf16x8*)(lds + SA(b, h) + aoff + m * 2048 + k * 1024); } while (0)
#define LDB(dst, b, h) do { _Pragma("unroll") for (int n = 0; n < 2; ++n) _Pragma("unroll") for (int k = 0; k < 2; ++k) dst[n][k] = *(const LAS bf16x8*)(lds + SB(b, h) + boff + n * 2048 + k * 1024); } while (0)
#define MMA(ai, bj, At, Bt) do { __builtin_amdgcn_s_setprio(1); _Pragma("unroll") for (int m = 0; m < 4; ++m) _Pragma("unroll") for (int n = 0; n < 2; ++n) _Pragma("unroll") for (int k = 0; k < 2; ++k) \
        acc[ai][bj][m][n] = __builtin_amdgcn_mfma_f32_16x16x32_bf16(Bt[n][k], At[m][k], acc[ai][bj][m][n], 0, 0, 0); __builtin_amdgcn_s_setprio(0); } while (0)
#define WAIT_V(n) asm volatile("s_waitcnt vmcnt(" #n ")" ::: "memory")
#define WAIT_L(n) asm volatile("s_waitcnt lgkmcnt(" #n ")" ::: "memory")
#define BAR __builtin_amdgcn_s_barrier()
#define SCHED __builtin_amdgcn_sched_barrier(0)
#define READY(st) do { const unsigned _need = call * PSH + (unsigned)steps_for(st); if (_need > known) { \
        if (wid == 0) { \
            unsigned _got = wave_wait_ge(myws + OFF_PROG + (unsigned)q * 256u + (unsigned)(lane & 31) * 4u, _need, myws + OFF_TMO, call); \
            __builtin_amdgcn_fence(__ATOMIC_ACQUIRE, ""); \
            if (lane == 0) bc[0] = _got; \
            asm volatile("s_waitcnt vmcnt(0) lgkmcnt(0)" ::: "memory"); \
            STAMP(myws, 2 + (st)); \
        } \
        asm volatile("" ::: "memory"); BAR; asm volatile("" ::: "memory"); \
        known = uni(bc[0]); if (known == 0u) { bad = true; known = 0xffffffffu; } \
        asm volatile("s_waitcnt lgkmcnt(0)" ::: "memory"); } } while (0)

    int rb, lp, nrb, nlp, ui = 0;
    if (!unit_of(0, q, gi, rb, lp)) return;
    f32x4 acc[2][2][4][2];
#pragma unroll
    for (int a = 0; a < 2; ++a)
#pragma unroll
        for (int b = 0; b < 2; ++b)
#pragma unroll
            for (int m = 0; m < 4; ++m)
#pragma unroll
                for (int n = 0; n < 2; ++n) acc[a][b][m][n] = (f32x4){0.f, 0.f, 0.f, 0.f};
    bf16x8 At[4][2], B0[2][2], B1[2][2];
    const char* cA = img + (size_t)(NXP + lp) * PANEL;
    const char* cB = img + (size_t)rb * PANEL;
    READY(0);
    STAGE(SB(0, 0), cB); STAGE(SA(0, 0), cA); STAGE(SB(0, 1), cB + hstep); STAGE(SA(0, 1), cA + hstep);
    if (wr == 1) BAR;
    WAIT_V(4); BAR;
    STAGE(SB(1, 0), cB + kstep); STAGE(SA(1, 0), cA + kstep); STAGE(SB(1, 1), cB + hstep + kstep);
    WAIT_V(6); BAR;
    for (;;) {
        const bool has_next = unit_of(ui + 1, q, gi, nrb, nlp);
        const char* nA = has_next ? img + (size_t)(NXP + nlp) * PANEL : cA;
        const char* nB = has_next ? img + (size_t)nrb * PANEL : cB;
        for (int t = 0; t < nt; t += 2) {
            const bool last = (t == nt - 2);
            const char* a1 = cA + (size_t)(t + 1) * kstep;
            const char* a2 = last ? nA : cA + (size_t)(t + 2) * kstep; const char* b2 = last ? nB : cB + (size_t)(t + 2) * kstep;
            const char* a3 = a2 + kstep; const char* b3 = b2 + kstep;
            if ((t & 7) == 6 && (!last || has_next)) READY(ui * 8 + ((t + 2) >> 3));
            LDB(B0, 0, 0); SCHED; LDA(At, 0, 0); STAGE(SA(1, 1), a1 + hstep);
            WAIT_L(8); BAR; WAIT_L(0); MMA(0, 0, At, B0); BAR; SCHED;
            LDB(B1, 0, 1); STAGE(SB(0, 0), b2);
            BAR; WAIT_L(0); MMA(0, 1, At, B1); BAR;
            LDA(At, 0, 1); STAGE(SA(0, 0), a2);
            BAR; WAIT_L(0); MMA(1, 0, At, B0); BAR; SCHED;
            STAGE(SB(0, 1), b2 + hstep);
            WAIT_V(6); BAR; MMA(1, 1, At, B1); BAR;
            LDB(B0, 1, 0); SCHED; LDA(At, 1, 0); STAGE(SA(0, 1), a2 + hstep);
            WAIT_L(8); BAR; WAIT_L(0); MMA(0, 0, At, B0); BAR; SCHED;
            LDB(B1, 1, 1); STAGE(SB(1, 0), b3);
            BAR; WAIT_L(0); MMA(0, 1, At, B1); BAR;
            LDA(At, 1, 1); STAGE(SA(1, 0), a3);
            BAR; WAIT_L(0); MMA(1, 0, At, B0); BAR; SCHED;
            STAGE(SB(1, 1), b3 + hstep);
            WAIT_V(6); BAR; MMA(1, 1, At, B1); BAR;
        }
        if (wid == 0) STAMP(myws, 32 + 2 * ui);
        {
            int tz = threadIdx.x; asm volatile("" : "+v"(tz));
            const int fr = tz & 15, fq = (tz >> 4) & 3;
            const bool dist = lp < ND;
            const int cb = dist ? rank * ND + lp : 8 * ND + (lp - ND);
            const __amdgpu_buffer_rsrc_t rO = __builtin_amdgcn_make_buffer_rsrc((void*)p.out[rank], 0, M_ * N_ * 2, RSRC3);
            const unsigned so = (unsigned)(rb * 256 + wc * 64) * (N_ * 2u) + (unsigned)(cb * 256 + wr * 128) * 2u;
            const unsigned vo = (unsigned)(4 * fq) * (N_ * 2u) + (unsigned)fr * 16u + so;
#pragma unroll
            for (int bj = 0; bj < 2; ++bj)
#pragma unroll
                for (int n = 0; n < 2; ++n)
#pragma unroll
                    for (int r = 0; r < 4; ++r) {
                        u32x4 o;
                        o[0] = cvt_pk_bf16(gelu(acc[0][bj][0][n][r]), gelu(acc[0][bj][1][n][r]));
                        o[1] = cvt_pk_bf16(gelu(acc[0][bj][2][n][r]), gelu(acc[0][bj][3][n][r]));
                        o[2] = cvt_pk_bf16(gelu(acc[1][bj][0][n][r]), gelu(acc[1][bj][1][n][r]));
                        o[3] = cvt_pk_bf16(gelu(acc[1][bj][2][n][r]), gelu(acc[1][bj][3][n][r]));
                        if (bad) o = (u32x4){0x7fc07fc0u, 0x7fc07fc0u, 0x7fc07fc0u, 0x7fc07fc0u};
                        __builtin_amdgcn_raw_buffer_store_b128(o, rO, vo + (unsigned)(bj * 32 + n * 16 + r) * (N_ * 2u), 0, AUX_SC1);
                    }
            if (dist) {
                WAIT_V(0);
                if (lane == 0) __hip_atomic_fetch_add((gu32*)(myws + OFF_DONE), 1u, __ATOMIC_RELAXED, __HIP_MEMORY_SCOPE_AGENT);
            }
        }
        if (wid == 0) STAMP(myws, 33 + 2 * ui);
        if (!has_next) break;
#pragma unroll
        for (int a = 0; a < 2; ++a)
#pragma unroll
            for (int b = 0; b < 2; ++b)
#pragma unroll
                for (int m = 0; m < 4; ++m)
#pragma unroll
                    for (int n = 0; n < 2; ++n) acc[a][b][m][n] = (f32x4){0.f, 0.f, 0.f, 0.f};
        rb = nrb; lp = nlp; cA = nA; cB = nB; ++ui;
    }
    WAIT_V(0);
    if (wr == 0) BAR;
#undef SA
#undef SB
#undef STAGE
#undef LDA
#undef LDB
#undef MMA
#undef READY
}

__global__ __launch_bounds__(512, 2) void fused_gemm_ar(const Params p) {
    __shared__ __attribute__((aligned(16))) unsigned char shm[LDS_BYTES];
    LAS unsigned char* lds = (LAS unsigned char*)shm;
    char* const myws = p.ws[p.rank];
    char* const cw = myws + OFF_CALL + blockIdx.x * 4u;
    if (threadIdx.x == 0) STAMP(myws, 0);
    const unsigned call = uni(ld_agent(cw)) + 1u;
    if (blockIdx.x < NCOPY) {
        copier(p, lds, call);
    } else {
        gemm(p, lds, call);
    }
    asm volatile("s_waitcnt vmcnt(0) lgkmcnt(0)" ::: "memory");
    __builtin_amdgcn_s_barrier();
    if (threadIdx.x == 0) { st_agent(cw, call); STAMP(myws, 1); }
}

}

extern "C" void kernel_launch(int rank, int nranks, ncclComm_t comm, hipStream_t stream,
                              void* const* d_in, const int* in_sizes, int n_in,
                              void* d_out, int out_size, const int* in_dtypes, int out_dtype,
                              void* const* d_out_all, void* const* d_ws_all, size_t ws_bytes) {
    (void)comm; (void)in_dtypes; (void)out_dtype; (void)d_out;
    if (nranks != P_ || (unsigned)rank >= (unsigned)P_ || n_in != 2 || in_sizes[0] != M_ * KS_ || in_sizes[1] != KS_ * N_ ||
        out_size != M_ * N_ || ws_bytes < WS_NEED) {
        fprintf(stderr, "kernel_launch: unexpected problem shape\n");
        return;
    }
    Params p;
    p.x = (const char*)d_in[0]; p.w = (const char*)d_in[1]; p.rank = rank;
    for (int i = 0; i < P_; ++i) { p.out[i] = (char*)d_out_all[i]; p.ws[i] = (char*)d_ws_all[i]; }
    hipLaunchKernelGGL(fused_gemm_ar, dim3(NWG), dim3(512), 0, stream, p);
    HIP_CHECK(hipGetLastError());
}
```

```cpp
#include <hip/hip_runtime.h>
#include <rccl/rccl.h>
#include <cstdio>
#include <cstdint>

#define HIP_CHECK(expr) do { hipError_t _e = (expr); if (_e != hipSuccess) { \
    fprintf(stderr, "HIP error %s at %s:%d\n", hipGetErrorString(_e), __FILE__, __LINE__); \
    return; } } while (0)

namespace {

#define LAS __attribute__((address_space(3)))
typedef short bf16x8 __attribute__((ext_vector_type(8)));
typedef float f32x4 __attribute__((ext_vector_type(4)));
typedef unsigned u32x4 __attribute__((ext_vector_type(4)));
typedef __attribute__((address_space(1))) unsigned gu32;

constexpr int M_ = 4096, K_ = 4096, N_ = 16384, P_ = 8, KS_ = K_ / P_;
#define ND_ 4
#define CPD_ 4
constexpr int ND = ND_;
constexpr int NCOM = 64 - 8 * ND;
constexpr int NWP = ND + NCOM;
constexpr int NXP = M_ / 256;
constexpr int CPD = CPD_;
constexpr int NCOPY = 8 * CPD;
constexpr int NWG = 256;
constexpr int GPX = NWG / 8 - CPD;
constexpr int NROUND = (4 * NWP + 2 * GPX - 1) / (2 * GPX);
static_assert(NROUND <= 3 && 4 * ND <= 2 * GPX && CPD <= 4, "schedule");
constexpr int CW = CPD * 8;
constexpr int first_wp(int r) { return (r * GPX + 1) / 2 < NWP ? (r * GPX + 1) / 2 : NWP; }
constexpr int F1 = first_wp(1), F2 = first_wp(2);
constexpr int NS0 = NXP + F1, NS1 = F2 - F1, NS2 = NWP - F2;
constexpr int SB1 = 8 * NS0, SB2 = SB1 + 8 * NS1, SB3 = SB2 + 8 * NS2;
constexpr int NBI = 4 * SB3;
constexpr int NBO = ND > 2 ? M_ / 4 : M_ / 8;
constexpr int STEPS_IN = (NBI + CW - 1) / CW, STEPS_ALL = (NBI + NBO + CW - 1) / CW;
constexpr int STEP_OUT0 = NBI / CW;
constexpr unsigned PSH = 256;
static_assert(STEPS_ALL < (int)PSH && ND >= 2 && ND <= 4 && NS1 > 0 && NS2 > 0, "steps");
__host__ __device__ constexpr int steps_for(int st) {
    return (4 * (st < 8 ? (st + 1) * NS0 : st < 16 ? SB1 + (st - 7) * NS1 : SB2 + (st - 15) * NS2) + CW - 1) / CW;
}

constexpr size_t OFF_PROG = 0;
constexpr size_t OFF_TMO = 4096;
constexpr size_t OFF_DONE = 4096 + 256;
constexpr size_t OFF_CALL = 8192;
constexpr size_t OFF_DBG = 65536;
constexpr size_t OFF_IMG = 1u << 20;
constexpr size_t REGION = (size_t)SB3 * 32768;
constexpr size_t IMG_BYTES = 8 * REGION;
constexpr size_t WS_NEED = OFF_IMG + 2 * IMG_BYTES;
__host__ __device__ constexpr unsigned batch_off(unsigned n) { return (n >> 5) * 262144u + (n & 31u) * 2048u; }
constexpr unsigned long long DEADLINE_TICKS = 100000000ull;

constexpr int HTB = 16384;
constexpr int LDS_BYTES = 8 * HTB + 256;
constexpr int AUX_SC1 = 16, AUX_SYS = 17;
constexpr int RSRC3 = 0x00020000;

struct Params {
    const char* x; const char* w;
    char* out[P_]; char* ws[P_];
    int rank;
    unsigned dmap;
};

__device__ __forceinline__ unsigned ld_sys(const char* p) { return __hip_atomic_load((gu32*)p, __ATOMIC_RELAXED, __HIP_MEMORY_SCOPE_SYSTEM); }
__device__ __forceinline__ unsigned ld_agent(const char* p) { return __hip_atomic_load((gu32*)p, __ATOMIC_RELAXED, __HIP_MEMORY_SCOPE_AGENT); }
__device__ __forceinline__ void st_sys(char* p, unsigned v) { __hip_atomic_store((gu32*)p, v, __ATOMIC_RELAXED, __HIP_MEMORY_SCOPE_SYSTEM); }
__device__ __forceinline__ void st_agent(char* p, unsigned v) { __hip_atomic_store((gu32*)p, v, __ATOMIC_RELAXED, __HIP_MEMORY_SCOPE_AGENT); }
__device__ __forceinline__ unsigned uni(unsigned v) { return (unsigned)__builtin_amdgcn_readfirstlane((int)v); }
__device__ __forceinline__ char* uni_ptr(const char* q) {
    const unsigned long long a = (unsigned long long)q;
    return (char*)(((unsigned long long)uni((unsigned)(a >> 32)) << 32) | uni((unsigned)a));
}

#define STAMP(ws, idx) do { } while (0)

__device__ __forceinline__ unsigned wave_wait_ge(const char* lane_word, unsigned need, char* tmo, unsigned call) {
    const unsigned long long t0 = __builtin_amdgcn_s_memrealtime();
    unsigned v;
    for (;;) {
        const unsigned tw = ld_agent(tmo);
        v = ld_sys(lane_word);
        if (__all(v >= need)) break;
        if (uni(tw) == call || __builtin_amdgcn_s_memrealtime() - t0 > DEADLINE_TICKS) {
            if ((threadIdx.x & 63) == 0) st_agent(tmo, call);
            return 0u;
        }
        __builtin_amdgcn_s_sleep(8);
    }
#pragma unroll
    for (int o = 32; o >= 1; o >>= 1) { const unsigned u = (unsigned)__shfl_xor((int)v, o); v = u < v ? u : v; }
    return uni(v);
}

__device__ __forceinline__ unsigned cvt_pk_bf16(float lo, float hi) { unsigned r; asm("v_cvt_pk_bf16_f32 %0, %1, %2" : "=v"(r) : "v"(lo), "v"(hi)); return r; }
__device__ __forceinline__ float gelu(float y) {
    constexpr float k1 = -2.0f * 0.7978845608028654f * 1.4426950408889634f, k2 = k1 * 0.044715f;
    const float t = y * __builtin_fmaf(y * y, k2, k1);
    return y * __builtin_amdgcn_rcpf(1.0f + __builtin_amdgcn_exp2f(t));
}

struct Batch {
    const char* sp; char* dp;
    unsigned sn, dn;
    unsigned s1, d1;
    unsigned a0, a1, c0, c1;
    bool tr;
};
__device__ __forceinline__ int stage_of(int slot) {
    if (slot < SB1) return slot / NS0;
    if (slot < SB2) return 8 + (slot - SB1) / NS1;
    if (slot < SB3) return 16 + (slot - SB2) / NS2;
    return 24;
}

__device__ __forceinline__ void copier(const Params& p, LAS unsigned char* lds, unsigned call) {
    const int j = blockIdx.x >> 3;
    const int rank = p.rank, dst = (int)((p.dmap >> ((blockIdx.x & 7) * 4)) & 7u);
    const bool self = dst == rank;
    const int wid = (int)uni(threadIdx.x >> 6), lane = threadIdx.x & 63;
    char* const myws = p.ws[rank];
    char* const dws = p.ws[dst];
    char* const myout = uni_ptr(p.out[rank]);
    char* const dout = p.out[dst];
    constexpr unsigned OOB = 0x80000000u;
    const unsigned vL = (unsigned)lane * 16u;
    const unsigned vL1 = (ND == 3 && lane >= 32) ? OOB : vL;
    const unsigned vX = (unsigned)(lane >> 3) * (KS_ * 2u) + (unsigned)((lane & 7) ^ (lane >> 4)) * 16u;
    const unsigned vW = (unsigned)(lane & 15) * 16u + (unsigned)(lane >> 4) * (8u * N_ * 2u);
    const unsigned img = (unsigned)OFF_IMG + (call & 1u) * (unsigned)IMG_BYTES + (unsigned)rank * (unsigned)REGION;
    const int total = self ? NBI : NBI + NBO;
    const int steps = self ? STEPS_IN : STEPS_ALL;
    LAS unsigned* bc = (LAS unsigned*)lds;
    char* const flag = dws + OFF_PROG + (unsigned)(lane & 7) * 256u + (unsigned)(rank * 4 + j) * 4u;
    const unsigned tiles_need = call * (unsigned)(NXP * ND * 8);

    const char* const px = uni_ptr(p.x);
    const char* const pw = uni_ptr(p.w);
    auto decode = [&](int n) {
        Batch b;
        b.tr = false;
        if (n >= NBI) {
            constexpr unsigned O0 = ND > 2 ? 1024u : N_ * 2u, O1 = (ND > 2 ? 32u : 64u) * N_ * 2u;
            const int o = n - NBI;
            const int row = ND > 2 ? (o >> 5) * 128 + (o & 31) : (o >> 5) * 256 + (o & 31) * 2;
            const unsigned off = n < total ? (unsigned)row * (N_ * 2u) + (unsigned)rank * (ND * 512u) : (OOB >> 1);
            b.sp = myout; b.dp = dout; b.sn = b.dn = M_ * N_ * 2; b.s1 = b.d1 = O1;
            b.a0 = b.c0 = vL + off; b.a1 = b.c1 = vL1 + (off + O0);
            return b;
        }
        const int slot = n >> 2, sub = n & 3, st = stage_of(slot);
        const int g = st & 7, r = st >> 3;
        const int pan = slot - (r == 0 ? g * NS0 : r == 1 ? SB1 + g * NS1 : SB2 + g * NS2);
        b.dp = dws; b.dn = (unsigned)WS_NEED; b.d1 = 65536u;
        b.c0 = vL + (img + batch_off((unsigned)n)); b.c1 = b.c0 + 1024u;
        if (r == 0 && pan < NXP) {
            const unsigned so = (unsigned)(pan * 256 + sub * 64) * (KS_ * 2u) + (unsigned)g * 128u;
            b.sp = px; b.sn = M_ * KS_ * 2; b.s1 = 16u * KS_ * 2u;
            b.a0 = vX + so; b.a1 = (vX ^ 64u) + (so + 8u * KS_ * 2u);
        } else {
            const int lp = r == 0 ? pan - NXP : (r == 1 ? F1 + pan : F2 + pan);
            const int cb = lp < ND ? dst * ND + lp : 8 * ND + (lp - ND);
            const unsigned so = (unsigned)(g * 64 + (sub & 1) * 32) * (N_ * 2u) + (unsigned)(cb * 256 + (sub >> 1) * 128) * 2u;
            b.sp = pw; b.sn = KS_ * N_ * 2; b.s1 = N_ * 4u; b.tr = true;
            b.a0 = vW + so; b.a1 = b.a0 + N_ * 2u;
        }
        return b;
    };
    auto load2 = [&](u32x4& v0, u32x4& v1, const Batch& b, int h) {
        const __amdgpu_buffer_rsrc_t rs = __builtin_amdgcn_make_buffer_rsrc((void*)b.sp, 0, (int)uni(b.sn), RSRC3);
        v0 = __builtin_amdgcn_raw_buffer_load_b128(rs, b.a0 + (unsigned)h * b.s1, 0, AUX_SC1);
        v1 = __builtin_amdgcn_raw_buffer_load_b128(rs, b.a1 + (unsigned)h * b.s1, 0, AUX_SC1);
    };
    auto store2 = [&](const u32x4& v0, const u32x4& v1, const Batch& b, int h) {
        const __amdgpu_buffer_rsrc_t rd = __builtin_amdgcn_make_buffer_rsrc((void*)b.dp, 0, (int)uni(b.dn), RSRC3);
        __builtin_amdgcn_raw_buffer_store_b128(v0, rd, b.c0 + (unsigned)h * b.d1, 0, AUX_SYS);
        __builtin_amdgcn_raw_buffer_store_b128(v1, rd, b.c1 + (unsigned)h * b.d1, 0, AUX_SYS);
    };
    auto arrange = [&](u32x4 (&t)[8], const u32x4 (&v)[8], const Batch& b) {
        if (b.tr) {
#pragma unroll
            for (int c = 0; c < 8; ++c)
#pragma unroll
                for (int e = 0; e < 4; ++e)
                    t[c][e] = __builtin_amdgcn_perm(v[2 * e + 1][c >> 1], v[2 * e][c >> 1], (c & 1) ? 0x07060302u : 0x05040100u);
        } else {
#pragma unroll
            for (int c = 0; c < 8; ++c) t[c] = v[c];
        }
#pragma unroll
        for (int c = 0; c < 8; ++c) asm volatile("" : "+v"(t[c]));
    };
    auto publish = [&](int done) {
        if (wid == 0 && lane < 8) st_sys(flag, call * PSH + (unsigned)done);
    };
#define WIN 6
    constexpr int VM0 = 0x0070, VMW = 0x0070 + WIN;
    static_assert(WIN == 6 || WIN == 10, "window");
#define PACE(imm) do { asm volatile("" ::: "memory"); __builtin_amdgcn_s_waitcnt(imm); __builtin_amdgcn_s_barrier(); \
        asm volatile("" ::: "memory"); __builtin_amdgcn_sched_barrier(0); } while (0)

    u32x4 R[8], T[8], Pa[4], Pb[4];
    const int n0 = j * 8 + wid;
    Batch b0 = decode(n0), b1 = decode(n0 + CW);
    load2(Pa[0], Pa[1], b0, 0); load2(Pa[2], Pa[3], b0, 1); load2(R[4], R[5], b0, 2); load2(R[6], R[7], b0, 3);
    load2(Pb[0], Pb[1], b1, 0); load2(Pb[2], Pb[3], b1, 1);
    if (threadIdx.x == 0) bc[1] = 0u;
    __builtin_amdgcn_s_waitcnt(0x0074);
    int k = 0, st_prev = 0;
#define COPY_STEP(PC) { \
        R[0] = PC[0]; R[1] = PC[1]; R[2] = PC[2]; R[3] = PC[3]; \
        arrange(T, R, b0); \
        if (wid == 0) STAMP(myws, 2 + (k * CW >> 5)); \
        const bool chk = k + 2 == STEP_OUT0 && !self; \
        unsigned done = 0; \
        if (chk) done = ld_agent(myws + OFF_DONE); \
        load2(R[4], R[5], b1, 2); store2(T[0], T[1], b0, 0); \
        const Batch b2 = decode(n0 + (k + 2) * CW); \
        const int st = stage_of(k * CW >> 2); \
        PACE(VMW); \
        load2(R[6], R[7], b1, 3); store2(T[2], T[3], b0, 1); \
        PACE(VMW); \
        if (WIN == 6 && st != st_prev) publish(k); \
        if (chk && uni(done) < tiles_need) { \
            if (wave_wait_ge(myws + OFF_DONE, tiles_need, myws + OFF_TMO, call) == 0u && lane == 0) bc[1] = 1u; \
        } \
        load2(PC[0], PC[1], b2, 0); store2(T[4], T[5], b0, 2); \
        PACE(VMW); \
        if (WIN == 10 && st != st_prev) publish(k); \
        st_prev = st; \
        load2(PC[2], PC[3], b2, 1); store2(T[6], T[7], b0, 3); \
        if (++k == steps) break; \
        PACE(VMW); \
        b0 = b1; b1 = b2; }
    for (;;) {
        COPY_STEP(Pa)
        COPY_STEP(Pb)
    }
#undef COPY_STEP
    PACE(VM0);
    if (uni(bc[1]) == 0u) publish(steps);
#undef PACE

    if (self && j == 0 && wid == 0) {
        int src = (lane & 31) >> 2;
        if (src == rank) src = (rank + 1) & 7;
        const char* word = myws + OFF_PROG + (unsigned)(src * 4 + (lane & 3) % CPD) * 4u;
        const unsigned got = wave_wait_ge(word, call * PSH + (unsigned)STEPS_ALL, myws + OFF_TMO, call);
        if (got == 0u && lane == 0) *(unsigned short*)(myout + (size_t)rank * ND * 512) = 0x7fc0u;
        STAMP(myws, 120);
    }
}

struct Unit { int rb, lp, slot, ns, st; };
__device__ __forceinline__ bool unit_of(int r, int q, int i, Unit& u) {
    const int t = (2 * r + (q & 1)) * GPX + i;
    if (t >= 4 * NWP) return false;
    u.rb = (q >> 1) * 4 + (t & 3);
    u.lp = t >> 2;
    if (u.lp < F1) { u.slot = NXP + u.lp; u.ns = NS0; u.st = 0; }
    else if (u.lp < F2) { u.slot = SB1 + u.lp - F1; u.ns = NS1; u.st = 8; }
    else { u.slot = SB2 + u.lp - F2; u.ns = NS2; u.st = 16; }
    return true;
}

__device__ __forceinline__ void gemm(const Params& p, LAS unsigned char* lds, unsigned call) {
    const int tid = threadIdx.x, wid = (int)uni(tid >> 6), lane = tid & 63, wr = wid >> 2, wc = wid & 3;
    const int q = blockIdx.x & 7, gi = (blockIdx.x >> 3) - CPD;
    const int rank = p.rank;
    char* const myws = p.ws[rank];
    const char* const img = myws + OFF_IMG + (size_t)(call & 1u) * IMG_BYTES;
    const unsigned voff = (unsigned)lane * 16u + (unsigned)(wid & 1) * 2048u + (unsigned)((wid >> 1) & 1) * 1024u + (unsigned)(wid >> 2) * 65536u;
    const unsigned ldsw = (unsigned)wid * 1024u;
    const int aoff = wr * 8192 + lane * 16;
    const int boff0 = (wc >> 1) * 8192 + (wc & 1) * 1024 + ((lane >> 3) & 1) * 2048 + (lane & 7) * 128 + (((lane >> 4) ^ ((lane >> 1) & 7)) * 16);
    const int boff1 = boff0 ^ 64;
    LAS unsigned* bc = (LAS unsigned*)(lds + 8 * HTB);
    unsigned known = 0; bool bad = false;
    constexpr size_t kstep = REGION, hstep = 131072;
    constexpr int nt = K_ / 64;

#define SA(b, h) (((b) * 2 + (h)) * HTB)
#define SB(b, h) ((4 + (b) * 2 + (h)) * HTB)
#define STAGE(bufoff, gbase) do { _Pragma("unroll") for (int _i = 0; _i < 2; ++_i) \
        __builtin_amdgcn_global_load_lds((const unsigned*)((gbase) + voff + _i * 4096), (LAS unsigned*)(lds + (bufoff) + ldsw + _i * 8192), 16, 0, AUX_SYS); } while (0)
#define LDA(dst, b, h) do { _Pragma("unroll") for (int m = 0; m < 4; ++m) _Pragma("unroll") for (int k = 0; k < 2; ++k) dst[m][k] = *(const LAS bf16x8*)(lds + SA(b, h) + aoff + m * 2048 + k * 1024); } while (0)
#define LDB(dst, b, h) do { _Pragma("unroll") for (int n = 0; n < 2; ++n) _Pragma("unroll") for (int k = 0; k < 2; ++k) dst[n][k] = *(const LAS bf16x8*)(lds + SB(b, h) + (k ? boff1 : boff0) + n * 4096); } while (0)
#define MMA(ai, bj, At, Bt) do { __builtin_amdgcn_s_setprio(1); _Pragma("unroll") for (int m = 0; m < 4; ++m) _Pragma("unroll") for (int n = 0; n < 2; ++n) _Pragma("unroll") for (int k = 0; k < 2; ++k) \
        acc[ai][bj][m][n] = __builtin_amdgcn_mfma_f32_16x16x32_bf16(Bt[n][k], At[m][k], acc[ai][bj][m][n], 0, 0, 0); __builtin_amdgcn_s_setprio(0); } while (0)
#define WAIT_V(n) asm volatile("s_waitcnt vmcnt(" #n ")" ::: "memory")
#define WAIT_L(n) asm volatile("s_waitcnt lgkmcnt(" #n ")" ::: "memory")
#define BAR __builtin_amdgcn_s_barrier()
#define SCHED __builtin_amdgcn_sched_barrier(0)
#define READY(st) do { const unsigned _need = call * PSH + (unsigned)steps_for(st); if (_need > known) { \
        if (wid == 0) { \
            unsigned _got = wave_wait_ge(myws + OFF_PROG + (unsigned)q * 256u + (unsigned)((lane & 28) + (lane & 3) % CPD) * 4u, _need, myws + OFF_TMO, call); \
            if (lane == 0) bc[0] = _got; \
            asm volatile("s_waitcnt vmcnt(0) lgkmcnt(0)" ::: "memory"); \
            STAMP(myws, 2 + (st)); \
        } \
        asm volatile("" ::: "memory"); BAR; asm volatile("" ::: "memory"); \
        known = uni(bc[0]); if (known == 0u) { bad = true; known = 0xffffffffu; } \
        asm volatile("s_waitcnt lgkmcnt(0)" ::: "memory"); } } while (0)

    Unit u, nu; int ui = 0;
    if (!unit_of(0, q, gi, u)) return;
    f32x4 acc[2][2][4][2];
#pragma unroll
    for (int a = 0; a < 2; ++a)
#pragma unroll
        for (int b = 0; b < 2; ++b)
#pragma unroll
            for (int m = 0; m < 4; ++m)
#pragma unroll
                for (int n = 0; n < 2; ++n) acc[a][b][m][n] = (f32x4){0.f, 0.f, 0.f, 0.f};
    bf16x8 At[4][2], B0[2][2], B1[2][2];
    const char* cA = img + batch_off(4u * (unsigned)u.slot);
    const char* cB = img + batch_off(4u * (unsigned)u.rb);
    READY(u.st);
    STAGE(SB(0, 0), cB); STAGE(SA(0, 0), cA); STAGE(SB(0, 1), cB + hstep); STAGE(SA(0, 1), cA + hstep);
    if (wr == 1) BAR;
    WAIT_V(4); BAR;
    STAGE(SB(1, 0), cB + kstep); STAGE(SA(1, 0), cA + kstep); STAGE(SB(1, 1), cB + hstep + kstep);
    WAIT_V(6); BAR;
    for (;;) {
        const bool has_next = unit_of(ui + 1, q, gi, nu);
        if (!has_next) nu = u;
        for (int t = 0; t < nt; t += 2) {
            const bool last = (t == nt - 2);
            const char* a1 = cA + kstep;
            const char* a2 = cA + 2 * kstep; const char* b2 = cB + 2 * kstep;
            if ((t & 7) == 6) {
                const int g = (t + 2) >> 3;
                a2 = img + batch_off(4u * (unsigned)(last ? nu.slot : u.slot + g * u.ns));
                b2 = img + batch_off(4u * (unsigned)(last ? nu.rb : u.rb + g * NS0));
                READY(last ? nu.st : u.st + g);
            }
            const char* a3 = a2 + kstep; const char* b3 = b2 + kstep;
            cA = a2; cB = b2;
            LDB(B0, 0, 0); SCHED; LDA(At, 0, 0); STAGE(SA(1, 1), a1 + hstep);
            WAIT_L(8); BAR; WAIT_L(0); MMA(0, 0, At, B0); BAR; SCHED;
            LDB(B1, 0, 1); STAGE(SB(0, 0), b2);
            BAR; WAIT_L(0); MMA(0, 1, At, B1); BAR;
            LDA(At, 0, 1); STAGE(SA(0, 0), a2);
            BAR; WAIT_L(0); MMA(1, 0, At, B0); BAR; SCHED;
            STAGE(SB(0, 1), b2 + hstep);
            WAIT_V(6); BAR; MMA(1, 1, At, B1); BAR;
            LDB(B0, 1, 0); SCHED; LDA(At, 1, 0); STAGE(SA(0, 1), a2 + hstep);
            WAIT_L(8); BAR; WAIT_L(0); MMA(0, 0, At, B0); BAR; SCHED;
            LDB(B1, 1, 1); STAGE(SB(1, 0), b3);
            BAR; WAIT_L(0); MMA(0, 1, At, B1); BAR;
            LDA(At, 1, 1); STAGE(SA(1, 0), a3);
            BAR; WAIT_L(0); MMA(1, 0, At, B0); BAR; SCHED;
            STAGE(SB(1, 1), b3 + hstep);
            WAIT_V(6); BAR; MMA(1, 1, At, B1); BAR;
        }
        if (wid == 0) STAMP(myws, 32 + 2 * ui);
        {
            int tz = threadIdx.x; asm volatile("" : "+v"(tz));
            const int fr = tz & 15, fq = (tz >> 4) & 3;
            const bool dist = u.lp < ND;
            const int cb = dist ? rank * ND + u.lp : 8 * ND + (u.lp - ND);
            const __amdgpu_buffer_rsrc_t rO = __builtin_amdgcn_make_buffer_rsrc((void*)p.out[rank], 0, M_ * N_ * 2, RSRC3);
            const unsigned so = (unsigned)(u.rb * 256 + wc * 64) * (N_ * 2u) + (unsigned)(cb * 256 + wr * 128) * 2u;
            const unsigned vo = (unsigned)(4 * fq) * (N_ * 2u) + (unsigned)fr * 16u + so;
#pragma unroll
            for (int bj = 0; bj < 2; ++bj)
#pragma unroll
                for (int n = 0; n < 2; ++n)
#pragma unroll
                    for (int r = 0; r < 4; ++r) {
                        u32x4 o;
                        o[0] = cvt_pk_bf16(gelu(acc[0][bj][0][n][r]), gelu(acc[0][bj][1][n][r]));
                        o[1] = cvt_pk_bf16(gelu(acc[0][bj][2][n][r]), gelu(acc[0][bj][3][n][r]));
                        o[2] = cvt_pk_bf16(gelu(acc[1][bj][0][n][r]), gelu(acc[1][bj][1][n][r]));
                        o[3] = cvt_pk_bf16(gelu(acc[1][bj][2][n][r]), gelu(acc[1][bj][3][n][r]));
                        if (bad) o = (u32x4){0x7fc07fc0u, 0x7fc07fc0u, 0x7fc07fc0u, 0x7fc07fc0u};
                        __builtin_amdgcn_raw_buffer_store_b128(o, rO, vo + (unsigned)(bj * 32 + n * 16 + r) * (N_ * 2u), 0, AUX_SC1);
                    }
            if (dist) {
                WAIT_V(0);
                if (lane == 0) __hip_atomic_fetch_add((gu32*)(myws + OFF_DONE), 1u, __ATOMIC_RELAXED, __HIP_MEMORY_SCOPE_AGENT);
            }
        }
        if (wid == 0) STAMP(myws, 33 + 2 * ui);
        if (!has_next) break;
#pragma unroll
        for (int a = 0; a < 2; ++a)
#pragma unroll
            for (int b = 0; b < 2; ++b)
#pragma unroll
                for (int m = 0; m < 4; ++m)
#pragma unroll
                    for (int n = 0; n < 2; ++n) acc[a][b][m][n] = (f32x4){0.f, 0.f, 0.f, 0.f};
        u = nu; ++ui;
    }
    WAIT_V(0);
    if (wr == 0) BAR;
#undef SA
#undef SB
#undef STAGE
#undef LDA
#undef LDB
#undef MMA
#undef READY
}

__global__ __launch_bounds__(512, 2) void fused_gemm_ar(const Params p) {
    __shared__ __attribute__((aligned(16))) unsigned char shm[LDS_BYTES];
    LAS unsigned char* lds = (LAS unsigned char*)shm;
    char* const myws = p.ws[p.rank];
    char* const cw = myws + OFF_CALL + blockIdx.x * 4u;
    if (threadIdx.x == 0) STAMP(myws, 0);
    const unsigned call = uni(ld_agent(cw)) + 1u;
    if (blockIdx.x < NCOPY) {
        copier(p, lds, call);
    } else {
        gemm(p, lds, call);
    }
    asm volatile("s_waitcnt vmcnt(0) lgkmcnt(0)" ::: "memory");
    __builtin_amdgcn_s_barrier();
    if (threadIdx.x == 0) { st_agent(cw, call); STAMP(myws, 1); }
}

}

extern "C" void kernel_launch(int rank, int nranks, ncclComm_t comm, hipStream_t stream,
                              void* const* d_in, const int* in_sizes, int n_in,
                              void* d_out, int out_size, const int* in_dtypes, int out_dtype,
                              void* const* d_out_all, void* const* d_ws_all, size_t ws_bytes) {
    (void)comm; (void)in_dtypes; (void)out_dtype; (void)d_out;
    if (nranks != P_ || (unsigned)rank >= (unsigned)P_ || n_in != 2 || in_sizes[0] != M_ * KS_ || in_sizes[1] != KS_ * N_ ||
        out_size != M_ * N_ || ws_bytes < WS_NEED) {
        fprintf(stderr, "kernel_launch: unexpected problem shape\n");
        return;
    }
    Params p;
    p.x = (const char*)d_in[0]; p.w = (const char*)d_in[1]; p.rank = rank;
    static const unsigned dmap[P_] = {0x71546302u, 0x26307415u, 0x14572360u, 0x14570632u, 0x60237451u, 0x06235147u, 0x57410362u, 0x36201574u};
    p.dmap = dmap[rank];
    for (int i = 0; i < P_; ++i) { p.out[i] = (char*)d_out_all[i]; p.ws[i] = (char*)d_ws_all[i]; }
    hipLaunchKernelGGL(fused_gemm_ar, dim3(NWG), dim3(512), 0, stream, p);
    HIP_CHECK(hipGetLastError());
}
```

```cpp
#include <hip/hip_runtime.h>
#include <rccl/rccl.h>
#include <cstdio>
#include <cstdint>

#define HIP_CHECK(expr) do { hipError_t _e = (expr); if (_e != hipSuccess) { \
    fprintf(stderr, "HIP error %s at %s:%d\n", hipGetErrorString(_e), __FILE__, __LINE__); \
    return; } } while (0)

namespace {

#define LAS __attribute__((address_space(3)))
typedef short bf16x8 __attribute__((ext_vector_type(8)));
typedef float f32x4 __attribute__((ext_vector_type(4)));
typedef unsigned u32x4 __attribute__((ext_vector_type(4)));
typedef __attribute__((address_space(1))) unsigned gu32;

constexpr int M_ = 4096, K_ = 4096, N_ = 16384, P_ = 8, KS_ = K_ / P_;
#define ND_ 4
#define CPD_ 4
constexpr int ND = ND_;
constexpr int NCOM = 64 - 8 * ND;
constexpr int NWP = ND + NCOM;
constexpr int NXP = M_ / 256;
constexpr int CPD = CPD_;
constexpr int NCOPY = 8 * CPD;
constexpr int NWG = 256;
constexpr int GPX = NWG / 8 - CPD;
constexpr int NROUND = (4 * NWP + 2 * GPX - 1) / (2 * GPX);
static_assert(NROUND <= 3 && 4 * ND <= 2 * GPX && CPD <= 4, "schedule");
constexpr int CW = CPD * 8;
constexpr int first_wp(int r) { return (r * GPX + 1) / 2 < NWP ? (r * GPX + 1) / 2 : NWP; }
constexpr int F1 = first_wp(1), F2 = first_wp(2);
constexpr int NS0 = NXP + F1, NS1 = F2 - F1, NS2 = NWP - F2;
constexpr int SB1 = 8 * NS0, SB2 = SB1 + 8 * NS1, SB3 = SB2 + 8 * NS2;
constexpr int NBI = 4 * SB3;
constexpr int NBO = ND > 2 ? M_ / 4 : M_ / 8;
constexpr int STEPS_IN = (NBI + CW - 1) / CW, STEPS_ALL = (NBI + NBO + CW - 1) / CW;
constexpr int STEP_OUT0 = NBI / CW;
constexpr unsigned PSH = 256;
static_assert(STEPS_ALL < (int)PSH && ND >= 2 && ND <= 4 && NS1 > 0 && NS2 > 0, "steps");
__host__ __device__ constexpr int steps_for(int st) {
    return (4 * (st < 8 ? (st + 1) * NS0 : st < 16 ? SB1 + (st - 7) * NS1 : SB2 + (st - 15) * NS2) + CW - 1) / CW;
}

constexpr size_t OFF_PROG = 0;
constexpr size_t OFF_TMO = 4096;
constexpr size_t OFF_DONE = 4096 + 256;
constexpr size_t OFF_CALL = 8192;
constexpr size_t OFF_CLAIM = 16384;
constexpr size_t OFF_DBG = 65536;
constexpr size_t OFF_IMG = 1u << 20;
constexpr size_t REGION = (size_t)SB3 * 32768;
constexpr size_t IMG_BYTES = 8 * REGION;
constexpr size_t WS_NEED = OFF_IMG + 2 * IMG_BYTES;
__host__ __device__ constexpr unsigned batch_off(unsigned n) { return (n >> 5) * 262144u + (n & 31u) * 2048u; }
constexpr unsigned long long DEADLINE_TICKS = 100000000ull;

constexpr int HTB = 16384;
constexpr int LDS_BYTES = 8 * HTB + 256;
constexpr int AUX_SC1 = 16, AUX_SYS = 17;
constexpr int RSRC3 = 0x00020000;

struct Params {
    const char* x; const char* w;
    char* out[P_]; char* ws[P_];
    int rank;
    unsigned dmap;
};

__device__ __forceinline__ unsigned ld_sys(const char* p) { return __hip_atomic_load((gu32*)p, __ATOMIC_RELAXED, __HIP_MEMORY_SCOPE_SYSTEM); }
__device__ __forceinline__ unsigned ld_agent(const char* p) { return __hip_atomic_load((gu32*)p, __ATOMIC_RELAXED, __HIP_MEMORY_SCOPE_AGENT); }
__device__ __forceinline__ void st_sys(char* p, unsigned v) { __hip_atomic_store((gu32*)p, v, __ATOMIC_RELAXED, __HIP_MEMORY_SCOPE_SYSTEM); }
__device__ __forceinline__ void st_agent(char* p, unsigned v) { __hip_atomic_store((gu32*)p, v, __ATOMIC_RELAXED, __HIP_MEMORY_SCOPE_AGENT); }
__device__ __forceinline__ unsigned uni(unsigned v) { return (unsigned)__builtin_amdgcn_readfirstlane((int)v); }
__device__ __forceinline__ char* uni_ptr(const char* q) {
    const unsigned long long a = (unsigned long long)q;
    return (char*)(((unsigned long long)uni((unsigned)(a >> 32)) << 32) | uni((unsigned)a));
}

#define STAMP(ws, idx) do { } while (0)

__device__ __forceinline__ unsigned wave_wait_ge(const char* lane_word, unsigned need, char* tmo, unsigned call) {
    const unsigned long long t0 = __builtin_amdgcn_s_memrealtime();
    unsigned v;
    for (;;) {
        const unsigned tw = ld_agent(tmo);
        v = ld_sys(lane_word);
        if (__all(v >= need)) break;
        if (uni(tw) == call || __builtin_amdgcn_s_memrealtime() - t0 > DEADLINE_TICKS) {
            if ((threadIdx.x & 63) == 0) st_agent(tmo, call);
            return 0u;
        }
        __builtin_amdgcn_s_sleep(8);
    }
#pragma unroll
    for (int o = 32; o >= 1; o >>= 1) { const unsigned u = (unsigned)__shfl_xor((int)v, o); v = u < v ? u : v; }
    return uni(v);
}

__device__ __forceinline__ unsigned cvt_pk_bf16(float lo, float hi) { unsigned r; asm("v_cvt_pk_bf16_f32 %0, %1, %2" : "=v"(r) : "v"(lo), "v"(hi)); return r; }
__device__ __forceinline__ float gelu(float y) {
    constexpr float k1 = -2.0f * 0.7978845608028654f * 1.4426950408889634f, k2 = k1 * 0.044715f;
    const float t = y * __builtin_fmaf(y * y, k2, k1);
    return y * __builtin_amdgcn_rcpf(1.0f + __builtin_amdgcn_exp2f(t));
}

struct Batch {
    const char* sp; char* dp;
    unsigned sn, dn;
    unsigned s1, d1;
    unsigned a0, a1, c0, c1;
    bool tr;
};
__device__ __forceinline__ int stage_of(int slot) {
    if (slot < SB1) return slot / NS0;
    if (slot < SB2) return 8 + (slot - SB1) / NS1;
    if (slot < SB3) return 16 + (slot - SB2) / NS2;
    return 24;
}

__device__ __forceinline__ unsigned copier(const Params& p, LAS unsigned char* lds, const char* cw) {
    const int j = blockIdx.x >> 3;
    const int rank = p.rank;
    const int wid = (int)uni(threadIdx.x >> 6), lane = threadIdx.x & 63;
    char* const myws = p.ws[rank];
    char* const myout = uni_ptr(p.out[rank]);
    constexpr unsigned OOB = 0x80000000u;
    const unsigned vL = (unsigned)lane * 16u;
    const unsigned vL1 = (ND == 3 && lane >= 32) ? OOB : vL;
    const unsigned vX = (unsigned)(lane >> 3) * (KS_ * 2u) + (unsigned)((lane & 7) ^ (lane >> 4)) * 16u;
    const unsigned vW = (unsigned)(lane & 15) * 16u + (unsigned)(lane >> 4) * (8u * N_ * 2u);
    LAS unsigned* bc = (LAS unsigned*)lds;
    int dst = 0, total = NBI; unsigned img = 0; char* dws = myws; char* dout = myout;

    const char* const px = uni_ptr(p.x);
    const char* const pw = uni_ptr(p.w);
    auto decode = [&](int n) {
        Batch b;
        b.tr = false;
        if (n >= NBI) {
            constexpr unsigned O0 = ND > 2 ? 1024u : N_ * 2u, O1 = (ND > 2 ? 32u : 64u) * N_ * 2u;
            const int o = n - NBI;
            const int row = ND > 2 ? (o >> 5) * 128 + (o & 31) : (o >> 5) * 256 + (o & 31) * 2;
            const unsigned off = n < total ? (unsigned)row * (N_ * 2u) + (unsigned)rank * (ND * 512u) : (OOB >> 1);
            b.sp = myout; b.dp = dout; b.sn = b.dn = M_ * N_ * 2; b.s1 = b.d1 = O1;
            b.a0 = b.c0 = vL + off; b.a1 = b.c1 = vL1 + (off + O0);
            return b;
        }
        const int slot = n >> 2, sub = n & 3, st = stage_of(slot);
        const int g = st & 7, r = st >> 3;
        const int pan = slot - (r == 0 ? g * NS0 : r == 1 ? SB1 + g * NS1 : SB2 + g * NS2);
        b.dp = dws; b.dn = (unsigned)WS_NEED; b.d1 = 65536u;
        b.c0 = vL + (img + batch_off((unsigned)n)); b.c1 = b.c0 + 1024u;
        if (r == 0 && pan < NXP) {
            const unsigned so = (unsigned)(pan * 256 + sub * 64) * (KS_ * 2u) + (unsigned)g * 128u;
            b.sp = px; b.sn = M_ * KS_ * 2; b.s1 = 16u * KS_ * 2u;
            b.a0 = vX + so; b.a1 = (vX ^ 64u) + (so + 8u * KS_ * 2u);
        } else {
            const int lp = r == 0 ? pan - NXP : (r == 1 ? F1 + pan : F2 + pan);
            const int cb = lp < ND ? dst * ND + lp : 8 * ND + (lp - ND);
            const unsigned so = (unsigned)(g * 64 + (sub & 1) * 32) * (N_ * 2u) + (unsigned)(cb * 256 + (sub >> 1) * 128) * 2u;
            b.sp = pw; b.sn = KS_ * N_ * 2; b.s1 = N_ * 4u; b.tr = true;
            b.a0 = vW + so; b.a1 = b.a0 + N_ * 2u;
        }
        return b;
    };
    auto load2 = [&](u32x4& v0, u32x4& v1, const Batch& b, int h) {
        const __amdgpu_buffer_rsrc_t rs = __builtin_amdgcn_make_buffer_rsrc((void*)b.sp, 0, (int)uni(b.sn), RSRC3);
        v0 = __builtin_amdgcn_raw_buffer_load_b128(rs, b.a0 + (unsigned)h * b.s1, 0, AUX_SC1);
        v1 = __builtin_amdgcn_raw_buffer_load_b128(rs, b.a1 + (unsigned)h * b.s1, 0, AUX_SC1);
    };
    auto store2 = [&](const u32x4& v0, const u32x4& v1, const Batch& b, int h) {
        const __amdgpu_buffer_rsrc_t rd = __builtin_amdgcn_make_buffer_rsrc((void*)b.dp, 0, (int)uni(b.dn), RSRC3);
        __builtin_amdgcn_raw_buffer_store_b128(v0, rd, b.c0 + (unsigned)h * b.d1, 0, AUX_SYS);
        __builtin_amdgcn_raw_buffer_store_b128(v1, rd, b.c1 + (unsigned)h * b.d1, 0, AUX_SYS);
    };
    auto arrange = [&](u32x4 (&t)[8], const u32x4 (&v)[8], const Batch& b) {
        if (b.tr) {
#pragma unroll
            for (int c = 0; c < 8; ++c)
#pragma unroll
                for (int e = 0; e < 4; ++e)
                    t[c][e] = __builtin_amdgcn_perm(v[2 * e + 1][c >> 1], v[2 * e][c >> 1], (c & 1) ? 0x07060302u : 0x05040100u);
        } else {
#pragma unroll
            for (int c = 0; c < 8; ++c) t[c] = v[c];
        }
#pragma unroll
        for (int c = 0; c < 8; ++c) asm volatile("" : "+v"(t[c]));
    };
#define WIN 6
    constexpr int VM0 = 0x0070, VMW = 0x0070 + WIN;
    static_assert(WIN == 6 || WIN == 10, "window");
#define PACE(imm) do { asm volatile("" ::: "memory"); __builtin_amdgcn_s_waitcnt(imm); __builtin_amdgcn_s_barrier(); \
        asm volatile("" ::: "memory"); __builtin_amdgcn_sched_barrier(0); } while (0)

    u32x4 R[8], T[8], Pa[4], Pb[4];
    const int n0 = j * 8 + wid;
    static_assert(2 * CW <= 4 * NXP, "the first two steps are x batches: their sources do not depend on the destination");
    Batch b0 = decode(n0), b1 = decode(n0 + CW);
    __builtin_amdgcn_s_waitcnt(VM0);
    const unsigned callv = ld_agent(cw);
    load2(Pa[0], Pa[1], b0, 0); load2(Pa[2], Pa[3], b0, 1); load2(R[4], R[5], b0, 2); load2(R[6], R[7], b0, 3);
    load2(Pb[0], Pb[1], b1, 0); load2(Pb[2], Pb[3], b1, 1);
    const unsigned call = uni(callv) + 1u;
    if (threadIdx.x == 0) {
        const unsigned xcc = __builtin_amdgcn_s_getreg((20 << 0) | (0 << 6) | ((4 - 1) << 11)) & 7u;
        unsigned d = (p.dmap >> (xcc * 4u)) & 7u;
        for (int i = 0; i < 7; ++i) {
            if (__hip_atomic_fetch_max((gu32*)(myws + OFF_CLAIM) + (j * 8 + d) * 32, call, __ATOMIC_RELAXED, __HIP_MEMORY_SCOPE_AGENT) < call) break;
            d = (d + 1u) & 7u;
        }
        bc[0] = d;
        bc[1] = 0u;
    }
    PACE(0x0074);
    dst = (int)(uni(bc[0]) & 7u);
    const bool self = dst == rank;
    dws = uni_ptr(p.ws[dst]); dout = uni_ptr(p.out[dst]);
    img = (unsigned)OFF_IMG + (call & 1u) * (unsigned)IMG_BYTES + (unsigned)rank * (unsigned)REGION;
    total = self ? NBI : NBI + NBO;
    b0.dp = b1.dp = dws; b0.c0 += img; b0.c1 += img; b1.c0 += img; b1.c1 += img;
    const int steps = self ? STEPS_IN : STEPS_ALL;
    char* const flag = dws + OFF_PROG + (unsigned)(lane & 7) * 256u + (unsigned)(rank * 4 + j) * 4u;
    const unsigned tiles_need = call * (unsigned)(NXP * ND * 8);
    auto publish = [&](int done) {
        if (wid == 0 && lane < 8) st_sys(flag, call * PSH + (unsigned)done);
    };
    int k = 0, st_prev = 0;
#define COPY_STEP(PC) { \
        R[0] = PC[0]; R[1] = PC[1]; R[2] = PC[2]; R[3] = PC[3]; \
        arrange(T, R, b0); \
        if (wid == 0) STAMP(myws, 2 + (k * CW >> 5)); \
        const bool chk = k + 2 == STEP_OUT0 && !self; \
        unsigned done = 0; \
        if (chk) done = ld_agent(myws + OFF_DONE); \
        load2(R[4], R[5], b1, 2); store2(T[0], T[1], b0, 0); \
        const Batch b2 = decode(n0 + (k + 2) * CW); \
        const int st = stage_of(k * CW >> 2); \
        PACE(VMW); \
        load2(R[6], R[7], b1, 3); store2(T[2], T[3], b0, 1); \
        PACE(VMW); \
        if (WIN == 6 && st != st_prev) publish(k); \
        if (chk && uni(done) < tiles_need) { \
            if (wave_wait_ge(myws + OFF_DONE, tiles_need, myws + OFF_TMO, call) == 0u && lane == 0) bc[1] = 1u; \
        } \
        load2(PC[0], PC[1], b2, 0); store2(T[4], T[5], b0, 2); \
        PACE(VMW); \
        if (WIN == 10 && st != st_prev) publish(k); \
        st_prev = st; \
        load2(PC[2], PC[3], b2, 1); store2(T[6], T[7], b0, 3); \
        if (++k == steps) break; \
        PACE(VMW); \
        b0 = b1; b1 = b2; }
    for (;;) {
        COPY_STEP(Pa)
        COPY_STEP(Pb)
    }
#undef COPY_STEP
    PACE(VM0);
    if (uni(bc[1]) == 0u) publish(steps);
#undef PACE

    if (self && j == 0 && wid == 0) {
        int src = (lane & 31) >> 2;
        if (src == rank) src = (rank + 1) & 7;
        const char* word = myws + OFF_PROG + (unsigned)(src * 4 + (lane & 3) % CPD) * 4u;
        const unsigned got = wave_wait_ge(word, call * PSH + (unsigned)STEPS_ALL, myws + OFF_TMO, call);
        if (got == 0u && lane == 0) *(unsigned short*)(myout + (size_t)rank * ND * 512) = 0x7fc0u;
        STAMP(myws, 120);
    }
    return call;
}

struct Unit { int rb, lp, slot, ns, st; };
__device__ __forceinline__ bool unit_of(int r, int q, int i, Unit& u) {
    const int t = (2 * r + (q & 1)) * GPX + i;
    if (t >= 4 * NWP) return false;
    u.rb = (q >> 1) * 4 + (t & 3);
    u.lp = t >> 2;
    if (u.lp < F1) { u.slot = NXP + u.lp; u.ns = NS0; u.st = 0; }
    else if (u.lp < F2) { u.slot = SB1 + u.lp - F1; u.ns = NS1; u.st = 8; }
    else { u.slot = SB2 + u.lp - F2; u.ns = NS2; u.st = 16; }
    return true;
}

__device__ __forceinline__ void gemm(const Params& p, LAS unsigned char* lds, unsigned call) {
    const int tid = threadIdx.x, wid = (int)uni(tid >> 6), lane = tid & 63, wr = wid >> 2, wc = wid & 3;
    const int q = blockIdx.x & 7, gi = (blockIdx.x >> 3) - CPD;
    const int rank = p.rank;
    char* const myws = p.ws[rank];
    const char* const img = myws + OFF_IMG + (size_t)(call & 1u) * IMG_BYTES;
    const unsigned voff = (unsigned)lane * 16u + (unsigned)(wid & 1) * 2048u + (unsigned)((wid >> 1) & 1) * 1024u + (unsigned)(wid >> 2) * 65536u;
    const unsigned ldsw = (unsigned)wid * 1024u;
    const int aoff = wr * 8192 + lane * 16;
    const int boff0 = (wc >> 1) * 8192 + (wc & 1) * 1024 + ((lane >> 3) & 1) * 2048 + (lane & 7) * 128 + (((lane >> 4) ^ ((lane >> 1) & 7)) * 16);
    const int boff1 = boff0 ^ 64;
    LAS unsigned* bc = (LAS unsigned*)(lds + 8 * HTB);
    unsigned known = 0; bool bad = false;
    constexpr size_t kstep = REGION, hstep = 131072;
    constexpr int nt = K_ / 64;

#define SA(b, h) (((b) * 2 + (h)) * HTB)
#define SB(b, h) ((4 + (b) * 2 + (h)) * HTB)
#define STAGE(bufoff, gbase) do { _Pragma("unroll") for (int _i = 0; _i < 2; ++_i) \
        __builtin_amdgcn_global_load_lds((const unsigned*)((gbase) + voff + _i * 4096), (LAS unsigned*)(lds + (bufoff) + ldsw + _i * 8192), 16, 0, AUX_SYS); } while (0)
#define LDA(dst, b, h) do { _Pragma("unroll") for (int m = 0; m < 4; ++m) _Pragma("unroll") for (int k = 0; k < 2; ++k) dst[m][k] = *(const LAS bf16x8*)(lds + SA(b, h) + aoff + m * 2048 + k * 1024); } while (0)
#define LDB(dst, b, h) do { _Pragma("unroll") for (int n = 0; n < 2; ++n) _Pragma("unroll") for (int k = 0; k < 2; ++k) dst[n][k] = *(const LAS bf16x8*)(lds + SB(b, h) + (k ? boff1 : boff0) + n * 4096); } while (0)
#define MMA(ai, bj, At, Bt) do { __builtin_amdgcn_s_setprio(1); _Pragma("unroll") for (int m = 0; m < 4; ++m) _Pragma("unroll") for (int n = 0; n < 2; ++n) _Pragma("unroll") for (int k = 0; k < 2; ++k) \
        acc[ai][bj][m][n] = __builtin_amdgcn_mfma_f32_16x16x32_bf16(Bt[n][k], At[m][k], acc[ai][bj][m][n], 0, 0, 0); __builtin_amdgcn_s_setprio(0); } while (0)
#define WAIT_V(n) asm volatile("s_waitcnt vmcnt(" #n ")" ::: "memory")
#define WAIT_L(n) asm volatile("s_waitcnt lgkmcnt(" #n ")" ::: "memory")
#define BAR __builtin_amdgcn_s_barrier()
#define SCHED __builtin_amdgcn_sched_barrier(0)
#define READY(st) do { const unsigned _need = call * PSH + (unsigned)steps_for(st); if (_need > known) { \
        if (wid == 0) { \
            unsigned _got = wave_wait_ge(myws + OFF_PROG + (unsigned)q * 256u + (unsigned)((lane & 28) + (lane & 3) % CPD) * 4u, _need, myws + OFF_TMO, call); \
            if (lane == 0) bc[0] = _got; \
            asm volatile("s_waitcnt vmcnt(0) lgkmcnt(0)" ::: "memory"); \
            STAMP(myws, 2 + (st)); \
        } \
        asm volatile("" ::: "memory"); BAR; asm volatile("" ::: "memory"); \
        known = uni(bc[0]); if (known == 0u) { bad = true; known = 0xffffffffu; } \
        asm volatile("s_waitcnt lgkmcnt(0)" ::: "memory"); } } while (0)

    Unit u, nu; int ui = 0;
    if (!unit_of(0, q, gi, u)) return;
    f32x4 acc[2][2][4][2];
#pragma unroll
    for (int a = 0; a < 2; ++a)
#pragma unroll
        for (int b = 0; b < 2; ++b)
#pragma unroll
            for (int m = 0; m < 4; ++m)
#pragma unroll
                for (int n = 0; n < 2; ++n) acc[a][b][m][n] = (f32x4){0.f, 0.f, 0.f, 0.f};
    bf16x8 At[4][2], B0[2][2], B1[2][2];
    const char* cA = img + batch_off(4u * (unsigned)u.slot);
    const char* cB = img + batch_off(4u * (unsigned)u.rb);
    READY(u.st);
    STAGE(SB(0, 0), cB); STAGE(SA(0, 0), cA); STAGE(SB(0, 1), cB + hstep); STAGE(SA(0, 1), cA + hstep);
    if (wr == 1) BAR;
    WAIT_V(4); BAR;
    STAGE(SB(1, 0), cB + kstep); STAGE(SA(1, 0), cA + kstep); STAGE(SB(1, 1), cB + hstep + kstep);
    WAIT_V(6); BAR;
    for (;;) {
        const bool has_next = unit_of(ui + 1, q, gi, nu);
        if (!has_next) nu = u;
        for (int t = 0; t < nt; t += 2) {
            const bool last = (t == nt - 2);
            const char* a1 = cA + kstep;
            const char* a2 = cA + 2 * kstep; const char* b2 = cB + 2 * kstep;
            if ((t & 7) == 6) {
                const int g = (t + 2) >> 3;
                a2 = img + batch_off(4u * (unsigned)(last ? nu.slot : u.slot + g * u.ns));
                b2 = img + batch_off(4u * (unsigned)(last ? nu.rb : u.rb + g * NS0));
                READY(last ? nu.st : u.st + g);
            }
            const char* a3 = a2 + kstep; const char* b3 = b2 + kstep;
            cA = a2; cB = b2;
            LDB(B0, 0, 0); SCHED; LDA(At, 0, 0); STAGE(SA(1, 1), a1 + hstep);
            WAIT_L(8); BAR; WAIT_L(0); MMA(0, 0, At, B0); BAR; SCHED;
            LDB(B1, 0, 1); STAGE(SB(0, 0), b2);
            BAR; WAIT_L(0); MMA(0, 1, At, B1); BAR;
            LDA(At, 0, 1); STAGE(SA(0, 0), a2);
            BAR; WAIT_L(0); MMA(1, 0, At, B0); BAR; SCHED;
            STAGE(SB(0, 1), b2 + hstep);
            WAIT_V(6); BAR; MMA(1, 1, At, B1); BAR;
            LDB(B0, 1, 0); SCHED; LDA(At, 1, 0); STAGE(SA(0, 1), a2 + hstep);
            WAIT_L(8); BAR; WAIT_L(0); MMA(0, 0, At, B0); BAR; SCHED;
            LDB(B1, 1, 1); STAGE(SB(1, 0), b3);
            BAR; WAIT_L(0); MMA(0, 1, At, B1); BAR;
            LDA(At, 1, 1); STAGE(SA(1, 0), a3);
            BAR; WAIT_L(0); MMA(1, 0, At, B0); BAR; SCHED;
            STAGE(SB(1, 1), b3 + hstep);
            WAIT_V(6); BAR; MMA(1, 1, At, B1); BAR;
        }
        if (wid == 0) STAMP(myws, 32 + 2 * ui);
        {
            int tz = threadIdx.x; asm volatile("" : "+v"(tz));
            const int fr = tz & 15, fq = (tz >> 4) & 3;
            const bool dist = u.lp < ND;
            const int cb = dist ? rank * ND + u.lp : 8 * ND + (u.lp - ND);
            const __amdgpu_buffer_rsrc_t rO = __builtin_amdgcn_make_buffer_rsrc((void*)p.out[rank], 0, M_ * N_ * 2, RSRC3);
            const unsigned so = (unsigned)(u.rb * 256 + wc * 64) * (N_ * 2u) + (unsigned)(cb * 256 + wr * 128) * 2u;
            const unsigned vo = (unsigned)(4 * fq) * (N_ * 2u) + (unsigned)fr * 16u + so;
#pragma unroll
            for (int bj = 0; bj < 2; ++bj)
#pragma unroll
                for (int n = 0; n < 2; ++n)
#pragma unroll
                    for (int r = 0; r < 4; ++r) {
                        u32x4 o;
                        o[0] = cvt_pk_bf16(gelu(acc[0][bj][0][n][r]), gelu(acc[0][bj][1][n][r]));
                        o[1] = cvt_pk_bf16(gelu(acc[0][bj][2][n][r]), gelu(acc[0][bj][3][n][r]));
                        o[2] = cvt_pk_bf16(gelu(acc[1][bj][0][n][r]), gelu(acc[1][bj][1][n][r]));
                        o[3] = cvt_pk_bf16(gelu(acc[1][bj][2][n][r]), gelu(acc[1][bj][3][n][r]));
                        if (bad) o = (u32x4){0x7fc07fc0u, 0x7fc07fc0u, 0x7fc07fc0u, 0x7fc07fc0u};
                        __builtin_amdgcn_raw_buffer_store_b128(o, rO, vo + (unsigned)(bj * 32 + n * 16 + r) * (N_ * 2u), 0, AUX_SC1);
                    }
            if (dist) {
                WAIT_V(0);
                if (lane == 0) __hip_atomic_fetch_add((gu32*)(myws + OFF_DONE), 1u, __ATOMIC_RELAXED, __HIP_MEMORY_SCOPE_AGENT);
            }
        }
        if (wid == 0) STAMP(myws, 33 + 2 * ui);
        if (!has_next) break;
#pragma unroll
        for (int a = 0; a < 2; ++a)
#pragma unroll
            for (int b = 0; b < 2; ++b)
#pragma unroll
                for (int m = 0; m < 4; ++m)
#pragma unroll
                    for (int n = 0; n < 2; ++n) acc[a][b][m][n] = (f32x4){0.f, 0.f, 0.f, 0.f};
        u = nu; ++ui;
    }
    WAIT_V(0);
    if (wr == 0) BAR;
#undef SA
#undef SB
#undef STAGE
#undef LDA
#undef LDB
#undef MMA
#undef READY
}

__global__ __launch_bounds__(512, 2) void fused_gemm_ar(const Params p) {
    __shared__ __attribute__((aligned(16))) unsigned char shm[LDS_BYTES];
    LAS unsigned char* lds = (LAS unsigned char*)shm;
    char* const myws = p.ws[p.rank];
    char* const cw = myws + OFF_CALL + blockIdx.x * 4u;
    if (threadIdx.x == 0) STAMP(myws, 0);
    unsigned call;
    if (blockIdx.x < NCOPY) {
        call = copier(p, lds, cw);
    } else {
        call = uni(ld_agent(cw)) + 1u;
        gemm(p, lds, call);
    }
    asm volatile("s_waitcnt vmcnt(0) lgkmcnt(0)" ::: "memory");
    __builtin_amdgcn_s_barrier();
    if (threadIdx.x == 0) { st_agent(cw, call); STAMP(myws, 1); }
}

}

extern "C" void kernel_launch(int rank, int nranks, ncclComm_t comm, hipStream_t stream,
                              void* const* d_in, const int* in_sizes, int n_in,
                              void* d_out, int out_size, const int* in_dtypes, int out_dtype,
                              void* const* d_out_all, void* const* d_ws_all, size_t ws_bytes) {
    (void)comm; (void)in_dtypes; (void)out_dtype; (void)d_out;
    if (nranks != P_ || (unsigned)rank >= (unsigned)P_ || n_in != 2 || in_sizes[0] != M_ * KS_ || in_sizes[1] != KS_ * N_ ||
        out_size != M_ * N_ || ws_bytes < WS_NEED) {
        fprintf(stderr, "kernel_launch: unexpected problem shape\n");
        return;
    }
    Params p;
    p.x = (const char*)d_in[0]; p.w = (const char*)d_in[1]; p.rank = rank;
    static const unsigned dmap[P_] = {0x71546302u, 0x26307415u, 0x14572360u, 0x14570632u, 0x60237451u, 0x06235147u, 0x57410362u, 0x36201574u};
    p.dmap = dmap[rank];
    for (int i = 0; i < P_; ++i) { p.out[i] = (char*)d_out_all[i]; p.ws[i] = (char*)d_ws_all[i]; }
    hipLaunchKernelGGL(fused_gemm_ar, dim3(NWG), dim3(512), 0, stream, p);
    HIP_CHECK(hipGetLastError());
}
```

```cpp
#include <hip/hip_runtime.h>
#include <rccl/rccl.h>
#include <cstdio>
#include <cstdint>

#define HIP_CHECK(expr) do { hipError_t _e = (expr); if (_e != hipSuccess) { \
    fprintf(stderr, "HIP error %s at %s:%d\n", hipGetErrorString(_e), __FILE__, __LINE__); \
    return; } } while (0)

namespace {

#define LAS __attribute__((address_space(3)))
typedef short bf16x8 __attribute__((ext_vector_type(8)));
typedef float f32x4 __attribute__((ext_vector_type(4)));
typedef unsigned u32x4 __attribute__((ext_vector_type(4)));
typedef __attribute__((address_space(1))) unsigned gu32;

constexpr int M_ = 4096, K_ = 4096, N_ = 16384, P_ = 8, KS_ = K_ / P_;
#define ND_ 4
#define CPD_ 4
constexpr int ND = ND_;
constexpr int NCOM = 64 - 8 * ND;
constexpr int NWP = ND + NCOM;
constexpr int NXP = M_ / 256;
constexpr int CPD = CPD_;
constexpr int NCOPY = 8 * CPD;
constexpr int NWG = 256;
constexpr int GPX = NWG / 8 - CPD;
constexpr int NROUND = (4 * NWP + 2 * GPX - 1) / (2 * GPX);
static_assert(NROUND <= 3 && 4 * ND <= 2 * GPX && CPD <= 4, "schedule");
constexpr int CW = CPD * 8;
constexpr int first_wp(int r) { return (r * GPX + 1) / 2 < NWP ? (r * GPX + 1) / 2 : NWP; }
constexpr int F1 = first_wp(1), F2 = first_wp(2);
constexpr int NS0 = NXP + F1, NS1 = F2 - F1, NS2 = NWP - F2;
constexpr int SB1 = 8 * NS0, SB2 = SB1 + 8 * NS1, SB3 = SB2 + 8 * NS2;
constexpr int NBI = 4 * SB3;
constexpr int NBO = M_ * ND / 16;
constexpr int STEPS_IN = (NBI + CW - 1) / CW, STEPS_ALL = (NBI + NBO + CW - 1) / CW;
constexpr int STEP_OUT0 = NBI / CW;
constexpr unsigned PSH = 256;
static_assert(STEPS_ALL < (int)PSH && ND >= 2 && ND <= 4 && NS1 > 0 && NS2 > 0, "steps");
__host__ __device__ constexpr int steps_for(int st) {
    return (4 * (st < 8 ? (st + 1) * NS0 : st < 16 ? SB1 + (st - 7) * NS1 : SB2 + (st - 15) * NS2) + CW - 1) / CW;
}

constexpr size_t OFF_PROG = 0;
constexpr size_t OFF_TMO = 4096;
constexpr size_t OFF_DONE = 4096 + 256;
constexpr size_t OFF_CALL = 8192;
constexpr size_t OFF_CLAIM = 16384;
constexpr size_t OFF_DBG = 65536;
constexpr size_t OFF_IMG = 1u << 20;
constexpr size_t REGION = (size_t)SB3 * 32768;
constexpr size_t IMG_BYTES = 8 * REGION;
constexpr size_t WS_NEED = OFF_IMG + 2 * IMG_BYTES;
__host__ __device__ constexpr unsigned batch_off(unsigned n) { return (n >> 5) * 262144u + (n & 31u) * 2048u; }
constexpr unsigned long long DEADLINE_TICKS = 100000000ull;

constexpr int HTB = 16384;
constexpr int LDS_BYTES = 8 * HTB + 256;
constexpr int AUX_SC1 = 16, AUX_SYS = 17;
constexpr int RSRC3 = 0x00020000;

struct Params {
    const char* x; const char* w;
    char* out[P_]; char* ws[P_];
    int rank;
    unsigned dmap;
};

__device__ __forceinline__ unsigned ld_sys(const char* p) { return __hip_atomic_load((gu32*)p, __ATOMIC_RELAXED, __HIP_MEMORY_SCOPE_SYSTEM); }
__device__ __forceinline__ unsigned ld_agent(const char* p) { return __hip_atomic_load((gu32*)p, __ATOMIC_RELAXED, __HIP_MEMORY_SCOPE_AGENT); }
__device__ __forceinline__ void st_sys(char* p, unsigned v) { __hip_atomic_store((gu32*)p, v, __ATOMIC_RELAXED, __HIP_MEMORY_SCOPE_SYSTEM); }
__device__ __forceinline__ void st_agent(char* p, unsigned v) { __hip_atomic_store((gu32*)p, v, __ATOMIC_RELAXED, __HIP_MEMORY_SCOPE_AGENT); }
__device__ __forceinline__ unsigned uni(unsigned v) { return (unsigned)__builtin_amdgcn_readfirstlane((int)v); }
__device__ __forceinline__ char* uni_ptr(const char* q) {
    const unsigned long long a = (unsigned long long)q;
    return (char*)(((unsigned long long)uni((unsigned)(a >> 32)) << 32) | uni((unsigned)a));
}

#define STAMP(ws, idx) do { } while (0)

__device__ __forceinline__ unsigned wave_wait_ge(const char* lane_word, unsigned need, char* tmo, unsigned call) {
    const unsigned long long t0 = __builtin_amdgcn_s_memrealtime();
    unsigned v;
    for (;;) {
        const unsigned tw = ld_agent(tmo);
        v = ld_sys(lane_word);
        if (__all(v >= need)) break;
        if (uni(tw) == call || __builtin_amdgcn_s_memrealtime() - t0 > DEADLINE_TICKS) {
            if ((threadIdx.x & 63) == 0) st_agent(tmo, call);
            return 0u;
        }
        __builtin_amdgcn_s_sleep(8);
    }
#pragma unroll
    for (int o = 32; o >= 1; o >>= 1) { const unsigned u = (unsigned)__shfl_xor((int)v, o); v = u < v ? u : v; }
    return uni(v);
}

__device__ __forceinline__ unsigned cvt_pk_bf16(float lo, float hi) { unsigned r; asm("v_cvt_pk_bf16_f32 %0, %1, %2" : "=v"(r) : "v"(lo), "v"(hi)); return r; }
__device__ __forceinline__ float gelu(float y) {
    constexpr float k1 = -2.0f * 0.7978845608028654f * 1.4426950408889634f, k2 = k1 * 0.044715f;
    const float t = y * __builtin_fmaf(y * y, k2, k1);
    return y * __builtin_amdgcn_rcpf(1.0f + __builtin_amdgcn_exp2f(t));
}

struct Batch {
    const char* sp; char* dp;
    unsigned sn, dn;
    unsigned s1, d1;
    unsigned a0, a1, c0, c1;
    bool tr;
};
__device__ __forceinline__ int stage_of(int slot) {
    if (slot < SB1) return slot / NS0;
    if (slot < SB2) return 8 + (slot - SB1) / NS1;
    if (slot < SB3) return 16 + (slot - SB2) / NS2;
    return 24;
}

__device__ __forceinline__ unsigned copier(const Params& p, LAS unsigned char* lds, const char* cw) {
    const int j = blockIdx.x >> 3;
    const int rank = p.rank;
    const int wid = (int)uni(threadIdx.x >> 6), lane = threadIdx.x & 63;
    char* const myws = p.ws[rank];
    char* const myout = uni_ptr(p.out[rank]);
    constexpr unsigned OOB = 0x80000000u;
    const unsigned vL = (unsigned)lane * 16u;
    const unsigned vL1 = lane < 32 ? vL + 1024u : vL + (N_ * 2u - 512u);
    const unsigned vX = (unsigned)(lane >> 3) * (KS_ * 2u) + (unsigned)((lane & 7) ^ (lane >> 4)) * 16u;
    const unsigned vW = (unsigned)(lane & 15) * 16u + (unsigned)(lane >> 4) * (8u * N_ * 2u);
    LAS unsigned* bc = (LAS unsigned*)lds;
    int dst = 0, total = NBI; unsigned img = 0; char* dws = myws; char* dout = myout;

    const char* const px = uni_ptr(p.x);
    const char* const pw = uni_ptr(p.w);
    auto decode = [&](int n) {
        Batch b;
        b.tr = false;
        if (n >= NBI) {
            constexpr unsigned RP = N_ * 2u;
            const int o = n - NBI, g = o / (8 * ND), w = o % (8 * ND);
            const int row = ND == 4 ? g * 128 + w : (ND == 3 ? g * 128 + w / 3 * 4 : g * 128 + w * 2);
            const unsigned off = n < total ? (unsigned)row * RP + (unsigned)rank * (ND * 512u) : (OOB >> 1);
            b.sp = myout; b.dp = dout; b.sn = b.dn = M_ * N_ * 2; b.s1 = b.d1 = 32u * RP;
            if (ND == 3) {
                const int ty = w % 3;
                b.a0 = (ty == 2 ? vL1 : vL) + (off + (ty == 0 ? 0u : ty == 1 ? RP + 512u : 2u * RP));
                b.a1 = (ty == 0 ? vL1 : vL) + (off + (ty == 0 ? 0u : ty == 1 ? 2u * RP : 3u * RP + 512u));
            } else {
                b.a0 = vL + off; b.a1 = vL + (off + (ND == 4 ? 1024u : RP));
            }
            b.c0 = b.a0; b.c1 = b.a1;
            return b;
        }
        const int slot = n >> 2, sub = n & 3, st = stage_of(slot);
        const int g = st & 7, r = st >> 3;
        const int pan = slot - (r == 0 ? g * NS0 : r == 1 ? SB1 + g * NS1 : SB2 + g * NS2);
        b.dp = dws; b.dn = (unsigned)WS_NEED; b.d1 = 65536u;
        b.c0 = vL + (img + batch_off((unsigned)n)); b.c1 = b.c0 + 1024u;
        if (r == 0 && pan < NXP) {
            const unsigned so = (unsigned)(pan * 256 + sub * 64) * (KS_ * 2u) + (unsigned)g * 128u;
            b.sp = px; b.sn = M_ * KS_ * 2; b.s1 = 16u * KS_ * 2u;
            b.a0 = vX + so; b.a1 = (vX ^ 64u) + (so + 8u * KS_ * 2u);
        } else {
            const int lp = r == 0 ? pan - NXP : (r == 1 ? F1 + pan : F2 + pan);
            const int cb = lp < ND ? dst * ND + lp : 8 * ND + (lp - ND);
            const unsigned so = (unsigned)(g * 64 + (sub & 1) * 32) * (N_ * 2u) + (unsigned)(cb * 256 + (sub >> 1) * 128) * 2u;
            b.sp = pw; b.sn = KS_ * N_ * 2; b.s1 = N_ * 4u; b.tr = true;
            b.a0 = vW + so; b.a1 = b.a0 + N_ * 2u;
        }
        return b;
    };
    auto load2 = [&](u32x4& v0, u32x4& v1, const Batch& b, int h) {
        const __amdgpu_buffer_rsrc_t rs = __builtin_amdgcn_make_buffer_rsrc((void*)b.sp, 0, (int)uni(b.sn), RSRC3);
        v0 = __builtin_amdgcn_raw_buffer_load_b128(rs, b.a0 + (unsigned)h * b.s1, 0, AUX_SC1);
        v1 = __builtin_amdgcn_raw_buffer_load_b128(rs, b.a1 + (unsigned)h * b.s1, 0, AUX_SC1);
    };
    auto store2 = [&](const u32x4& v0, const u32x4& v1, const Batch& b, int h) {
        const __amdgpu_buffer_rsrc_t rd = __builtin_amdgcn_make_buffer_rsrc((void*)b.dp, 0, (int)uni(b.dn), RSRC3);
        __builtin_amdgcn_raw_buffer_store_b128(v0, rd, b.c0 + (unsigned)h * b.d1, 0, AUX_SYS);
        __builtin_amdgcn_raw_buffer_store_b128(v1, rd, b.c1 + (unsigned)h * b.d1, 0, AUX_SYS);
    };
    auto arrange = [&](u32x4 (&t)[8], const u32x4 (&v)[8], const Batch& b) {
        if (b.tr) {
#pragma unroll
            for (int c = 0; c < 8; ++c)
#pragma unroll
                for (int e = 0; e < 4; ++e)
                    t[c][e] = __builtin_amdgcn_perm(v[2 * e + 1][c >> 1], v[2 * e][c >> 1], (c & 1) ? 0x07060302u : 0x05040100u);
        } else {
#pragma unroll
            for (int c = 0; c < 8; ++c) t[c] = v[c];
        }
#pragma unroll
        for (int c = 0; c < 8; ++c) asm volatile("" : "+v"(t[c]));
    };
#define WIN 6
    constexpr int VM0 = 0x0070, VMW = 0x0070 + WIN;
    static_assert(WIN == 6 || WIN == 10, "window");
#define PACE(imm) do { asm volatile("" ::: "memory"); __builtin_amdgcn_s_waitcnt(imm); __builtin_amdgcn_s_barrier(); \
        asm volatile("" ::: "memory"); __builtin_amdgcn_sched_barrier(0); } while (0)

    u32x4 R[8], T[8], Pa[4], Pb[4];
    const int n0 = j * 8 + wid;
    static_assert(2 * CW <= 4 * NXP, "the first two steps are x batches: their sources do not depend on the destination");
    Batch b0 = decode(n0), b1 = decode(n0 + CW);
    __builtin_amdgcn_s_waitcnt(VM0);
    const unsigned callv = ld_agent(cw);
    load2(Pa[0], Pa[1], b0, 0); load2(Pa[2], Pa[3], b0, 1); load2(R[4], R[5], b0, 2); load2(R[6], R[7], b0, 3);
    load2(Pb[0], Pb[1], b1, 0); load2(Pb[2], Pb[3], b1, 1);
    const unsigned call = uni(callv) + 1u;
    if (threadIdx.x == 0) {
        const unsigned xcc = __builtin_amdgcn_s_getreg((20 << 0) | (0 << 6) | ((4 - 1) << 11)) & 7u;
        unsigned d = (p.dmap >> (xcc * 4u)) & 7u;
        for (int i = 0; i < 7; ++i) {
            if (__hip_atomic_fetch_max((gu32*)(myws + OFF_CLAIM) + (j * 8 + d) * 32, call, __ATOMIC_RELAXED, __HIP_MEMORY_SCOPE_AGENT) < call) break;
            d = (d + 1u) & 7u;
        }
        bc[0] = d;
        bc[1] = 0u;
    }
    PACE(0x0074);
    dst = (int)(uni(bc[0]) & 7u);
    const bool self = dst == rank;
    dws = uni_ptr(p.ws[dst]); dout = uni_ptr(p.out[dst]);
    img = (unsigned)OFF_IMG + (call & 1u) * (unsigned)IMG_BYTES + (unsigned)rank * (unsigned)REGION;
    total = self ? NBI : NBI + NBO;
    b0.dp = b1.dp = dws; b0.c0 += img; b0.c1 += img; b1.c0 += img; b1.c1 += img;
    const int steps = self ? STEPS_IN : STEPS_ALL;
    char* const flag = dws + OFF_PROG + (unsigned)(lane & 7) * 256u + (unsigned)(rank * 4 + j) * 4u;
    const unsigned tiles_need = call * (unsigned)(NXP * ND * 8);
    auto publish = [&](int done) {
        if (wid == 0 && lane < 8) st_sys(flag, call * PSH + (unsigned)done);
    };
    int k = 0, st_prev = 0;
#define COPY_STEP(PC) { \
        R[0] = PC[0]; R[1] = PC[1]; R[2] = PC[2]; R[3] = PC[3]; \
        arrange(T, R, b0); \
        if (wid == 0) { STAMP(myws, 2 + (k * CW >> 5)); if (k == STEP_OUT0) STAMP(myws, 121); } \
        const bool chk = k + 2 == STEP_OUT0 && !self; \
        unsigned done = 0; \
        if (chk) done = ld_agent(myws + OFF_DONE); \
        load2(R[4], R[5], b1, 2); store2(T[0], T[1], b0, 0); \
        const Batch b2 = decode(n0 + (k + 2) * CW); \
        const int st = stage_of(k * CW >> 2); \
        PACE(VMW); \
        load2(R[6], R[7], b1, 3); store2(T[2], T[3], b0, 1); \
        PACE(VMW); \
        if (WIN == 6 && st != st_prev) publish(k); \
        if (chk && uni(done) < tiles_need) { \
            if (wave_wait_ge(myws + OFF_DONE, tiles_need, myws + OFF_TMO, call) == 0u && lane == 0) bc[1] = 1u; \
        } \
        load2(PC[0], PC[1], b2, 0); store2(T[4], T[5], b0, 2); \
        PACE(VMW); \
        if (WIN == 10 && st != st_prev) publish(k); \
        st_prev = st; \
        load2(PC[2], PC[3], b2, 1); store2(T[6], T[7], b0, 3); \
        if (++k == steps) break; \
        PACE(VMW); \
        b0 = b1; b1 = b2; }
    for (;;) {
        COPY_STEP(Pa)
        COPY_STEP(Pb)
    }
#undef COPY_STEP
    PACE(VM0);
    if (uni(bc[1]) == 0u) publish(steps);
#undef PACE

    if (self && j == 0 && wid == 0) {
        int src = (lane & 31) >> 2;
        if (src == rank) src = (rank + 1) & 7;
        const char* word = myws + OFF_PROG + (unsigned)(src * 4 + (lane & 3) % CPD) * 4u;
        const unsigned got = wave_wait_ge(word, call * PSH + (unsigned)STEPS_ALL, myws + OFF_TMO, call);
        if (got == 0u && lane == 0) *(unsigned short*)(myout + (size_t)rank * ND * 512) = 0x7fc0u;
        STAMP(myws, 120);
    }
    return call;
}

struct Unit { int rb, lp, slot, ns, st; };
__device__ __forceinline__ bool unit_of(int r, int q, int i, Unit& u) {
    const int t = (2 * r + (q & 1)) * GPX + i;
    if (t >= 4 * NWP) return false;
    u.rb = (q >> 1) * 4 + (t & 3);
    u.lp = t >> 2;
    if (u.lp < F1) { u.slot = NXP + u.lp; u.ns = NS0; u.st = 0; }
    else if (u.lp < F2) { u.slot = SB1 + u.lp - F1; u.ns = NS1; u.st = 8; }
    else { u.slot = SB2 + u.lp - F2; u.ns = NS2; u.st = 16; }
    return true;
}

__device__ __forceinline__ void gemm(const Params& p, LAS unsigned char* lds, unsigned call) {
    const int tid = threadIdx.x, wid = (int)uni(tid >> 6), lane = tid & 63, wr = wid >> 2, wc = wid & 3;
    const int q = blockIdx.x & 7, gi = (blockIdx.x >> 3) - CPD;
    const int rank = p.rank;
    char* const myws = p.ws[rank];
    const char* const img = myws + OFF_IMG + (size_t)(call & 1u) * IMG_BYTES;
    const unsigned voff = (unsigned)lane * 16u + (unsigned)(wid & 1) * 2048u + (unsigned)((wid >> 1) & 1) * 1024u + (unsigned)(wid >> 2) * 65536u;
    const unsigned ldsw = (unsigned)wid * 1024u;
    const int aoff = wr * 8192 + lane * 16;
    const int boff0 = (wc >> 1) * 8192 + (wc & 1) * 1024 + ((lane >> 3) & 1) * 2048 + (lane & 7) * 128 + (((lane >> 4) ^ ((lane >> 1) & 7)) * 16);
    const int boff1 = boff0 ^ 64;
    LAS unsigned* bc = (LAS unsigned*)(lds + 8 * HTB);
    unsigned known = 0; bool bad = false;
    constexpr size_t kstep = REGION, hstep = 131072;
    constexpr int nt = K_ / 64;

#define SA(b, h) (((b) * 2 + (h)) * HTB)
#define SB(b, h) ((4 + (b) * 2 + (h)) * HTB)
#define STAGE(bufoff, gbase) do { _Pragma("unroll") for (int _i = 0; _i < 2; ++_i) \
        __builtin_amdgcn_global_load_lds((const unsigned*)((gbase) + voff + _i * 4096), (LAS unsigned*)(lds + (bufoff) + ldsw + _i * 8192), 16, 0, AUX_SYS); } while (0)
#define LDA(dst, b, h) do { _Pragma("unroll") for (int m = 0; m < 4; ++m) _Pragma("unroll") for (int k = 0; k < 2; ++k) dst[m][k] = *(const LAS bf16x8*)(lds + SA(b, h) + aoff + m * 2048 + k * 1024); } while (0)
#define LDB(dst, b, h) do { _Pragma("unroll") for (int n = 0; n < 2; ++n) _Pragma("unroll") for (int k = 0; k < 2; ++k) dst[n][k] = *(const LAS bf16x8*)(lds + SB(b, h) + (k ? boff1 : boff0) + n * 4096); } while (0)
#define MMA(ai, bj, At, Bt) do { __builtin_amdgcn_s_setprio(1); _Pragma("unroll") for (int m = 0; m < 4; ++m) _Pragma("unroll") for (int n = 0; n < 2; ++n) _Pragma("unroll") for (int k = 0; k < 2; ++k) \
        acc[ai][bj][m][n] = __builtin_amdgcn_mfma_f32_16x16x32_bf16(Bt[n][k], At[m][k], acc[ai][bj][m][n], 0, 0, 0); __builtin_amdgcn_s_setprio(0); } while (0)
#define WAIT_V(n) asm volatile("s_waitcnt vmcnt(" #n ")" ::: "memory")
#define WAIT_L(n) asm volatile("s_waitcnt lgkmcnt(" #n ")" ::: "memory")
#define BAR __builtin_amdgcn_s_barrier()
#define SCHED __builtin_amdgcn_sched_barrier(0)
#define READY(st) do { const unsigned _need = call * PSH + (unsigned)steps_for(st); if (_need > known) { \
        if (wid == 0) { \
            unsigned _got = wave_wait_ge(myws + OFF_PROG + (unsigned)q * 256u + (unsigned)((lane & 28) + (lane & 3) % CPD) * 4u, _need, myws + OFF_TMO, call); \
            if (lane == 0) bc[0] = _got; \
            asm volatile("s_waitcnt vmcnt(0) lgkmcnt(0)" ::: "memory"); \
            STAMP(myws, 2 + (st)); \
        } \
        asm volatile("" ::: "memory"); BAR; asm volatile("" ::: "memory"); \
        known = uni(bc[0]); if (known == 0u) { bad = true; known = 0xffffffffu; } \
        asm volatile("s_waitcnt lgkmcnt(0)" ::: "memory"); } } while (0)

    Unit u, nu; int ui = 0;
    if (!unit_of(0, q, gi, u)) return;
    f32x4 acc[2][2][4][2];
#pragma unroll
    for (int a = 0; a < 2; ++a)
#pragma unroll
        for (int b = 0; b < 2; ++b)
#pragma unroll
            for (int m = 0; m < 4; ++m)
#pragma unroll
                for (int n = 0; n < 2; ++n) acc[a][b][m][n] = (f32x4){0.f, 0.f, 0.f, 0.f};
    bf16x8 At[4][2], B0[2][2], B1[2][2];
    const char* cA = img + batch_off(4u * (unsigned)u.slot);
    const char* cB = img + batch_off(4u * (unsigned)u.rb);
    READY(u.st);
    STAGE(SB(0, 0), cB); STAGE(SA(0, 0), cA); STAGE(SB(0, 1), cB + hstep); STAGE(SA(0, 1), cA + hstep);
    if (wr == 1) BAR;
    WAIT_V(4); BAR;
    STAGE(SB(1, 0), cB + kstep); STAGE(SA(1, 0), cA + kstep); STAGE(SB(1, 1), cB + hstep + kstep);
    WAIT_V(6); BAR;
    for (;;) {
        const bool has_next = unit_of(ui + 1, q, gi, nu);
        if (!has_next) nu = u;
        for (int t = 0; t < nt; t += 2) {
            const bool last = (t == nt - 2);
            const char* a1 = cA + kstep;
            const char* a2 = cA + 2 * kstep; const char* b2 = cB + 2 * kstep;
            if ((t & 7) == 6) {
                const int g = (t + 2) >> 3;
                a2 = img + batch_off(4u * (unsigned)(last ? nu.slot : u.slot + g * u.ns));
                b2 = img + batch_off(4u * (unsigned)(last ? nu.rb : u.rb + g * NS0));
                READY(last ? nu.st : u.st + g);
            }
            const char* a3 = a2 + kstep; const char* b3 = b2 + kstep;
            cA = a2; cB = b2;
            LDB(B0, 0, 0); SCHED; LDA(At, 0, 0); STAGE(SA(1, 1), a1 + hstep);
            WAIT_L(8); BAR; WAIT_L(0); MMA(0, 0, At, B0); BAR; SCHED;
            LDB(B1, 0, 1); STAGE(SB(0, 0), b2);
            BAR; WAIT_L(0); MMA(0, 1, At, B1); BAR;
            LDA(At, 0, 1); STAGE(SA(0, 0), a2);
            BAR; WAIT_L(0); MMA(1, 0, At, B0); BAR; SCHED;
            STAGE(SB(0, 1), b2 + hstep);
            WAIT_V(6); BAR; MMA(1, 1, At, B1); BAR;
            LDB(B0, 1, 0); SCHED; LDA(At, 1, 0); STAGE(SA(0, 1), a2 + hstep);
            WAIT_L(8); BAR; WAIT_L(0); MMA(0, 0, At, B0); BAR; SCHED;
            LDB(B1, 1, 1); STAGE(SB(1, 0), b3);
            BAR; WAIT_L(0); MMA(0, 1, At, B1); BAR;
            LDA(At, 1, 1); STAGE(SA(1, 0), a3);
            BAR; WAIT_L(0); MMA(1, 0, At, B0); BAR; SCHED;
            STAGE(SB(1, 1), b3 + hstep);
            WAIT_V(6); BAR; MMA(1, 1, At, B1); BAR;
        }
        if (wid == 0) STAMP(myws, 32 + 2 * ui);
        {
            int tz = threadIdx.x; asm volatile("" : "+v"(tz));
            const int fr = tz & 15, fq = (tz >> 4) & 3;
            const bool dist = u.lp < ND;
            const int cb = dist ? rank * ND + u.lp : 8 * ND + (u.lp - ND);
            const __amdgpu_buffer_rsrc_t rO = __builtin_amdgcn_make_buffer_rsrc((void*)p.out[rank], 0, M_ * N_ * 2, RSRC3);
            const unsigned so = (unsigned)(u.rb * 256 + wc * 64) * (N_ * 2u) + (unsigned)(cb * 256 + wr * 128) * 2u;
            const unsigned vo = (unsigned)(4 * fq) * (N_ * 2u) + (unsigned)fr * 16u + so;
#pragma unroll
            for (int bj = 0; bj < 2; ++bj)
#pragma unroll
                for (int n = 0; n < 2; ++n)
#pragma unroll
                    for (int r = 0; r < 4; ++r) {
                        u32x4 o;
                        o[0] = cvt_pk_bf16(gelu(acc[0][bj][0][n][r]), gelu(acc[0][bj][1][n][r]));
                        o[1] = cvt_pk_bf16(gelu(acc[0][bj][2][n][r]), gelu(acc[0][bj][3][n][r]));
                        o[2] = cvt_pk_bf16(gelu(acc[1][bj][0][n][r]), gelu(acc[1][bj][1][n][r]));
                        o[3] = cvt_pk_bf16(gelu(acc[1][bj][2][n][r]), gelu(acc[1][bj][3][n][r]));
                        if (bad) o = (u32x4){0x7fc07fc0u, 0x7fc07fc0u, 0x7fc07fc0u, 0x7fc07fc0u};
                        __builtin_amdgcn_raw_buffer_store_b128(o, rO, vo + (unsigned)(bj * 32 + n * 16 + r) * (N_ * 2u), 0, AUX_SC1);
                    }
            if (dist) {
                WAIT_V(0);
                if (lane == 0) __hip_atomic_fetch_add((gu32*)(myws + OFF_DONE), 1u, __ATOMIC_RELAXED, __HIP_MEMORY_SCOPE_AGENT);
            }
        }
        if (wid == 0) STAMP(myws, 33 + 2 * ui);
        if (!has_next) break;
#pragma unroll
        for (int a = 0; a < 2; ++a)
#pragma unroll
            for (int b = 0; b < 2; ++b)
#pragma unroll
                for (int m = 0; m < 4; ++m)
#pragma unroll
                    for (int n = 0; n < 2; ++n) acc[a][b][m][n] = (f32x4){0.f, 0.f, 0.f, 0.f};
        u = nu; ++ui;
    }
    WAIT_V(0);
    if (wr == 0) BAR;
#undef SA
#undef SB
#undef STAGE
#undef LDA
#undef LDB
#undef MMA
#undef READY
}

__global__ __launch_bounds__(512, 2) void fused_gemm_ar(const Params p) {
    __shared__ __attribute__((aligned(16))) unsigned char shm[LDS_BYTES];
    LAS unsigned char* lds = (LAS unsigned char*)shm;
    char* const myws = p.ws[p.rank];
    char* const cw = myws + OFF_CALL + blockIdx.x * 4u;
    if (threadIdx.x == 0) STAMP(myws, 0);
    unsigned call;
    if (blockIdx.x < NCOPY) {
        call = copier(p, lds, cw);
    } else {
        call = uni(ld_agent(cw)) + 1u;
        gemm(p, lds, call);
    }
    asm volatile("s_waitcnt vmcnt(0) lgkmcnt(0)" ::: "memory");
    __builtin_amdgcn_s_barrier();
    if (threadIdx.x == 0) { st_agent(cw, call); STAMP(myws, 1); }
}

}

extern "C" void kernel_launch(int rank, int nranks, ncclComm_t comm, hipStream_t stream,
                              void* const* d_in, const int* in_sizes, int n_in,
                              void* d_out, int out_size, const int* in_dtypes, int out_dtype,
                              void* const* d_out_all, void* const* d_ws_all, size_t ws_bytes) {
    (void)comm; (void)in_dtypes; (void)out_dtype; (void)d_out;
    if (nranks != P_ || (unsigned)rank >= (unsigned)P_ || n_in != 2 || in_sizes[0] != M_ * KS_ || in_sizes[1] != KS_ * N_ ||
        out_size != M_ * N_ || ws_bytes < WS_NEED) {
        fprintf(stderr, "kernel_launch: unexpected problem shape\n");
        return;
    }
    Params p;
    p.x = (const char*)d_in[0]; p.w = (const char*)d_in[1]; p.rank = rank;
    static const unsigned dmap[P_] = {0x71546302u, 0x26307415u, 0x14572360u, 0x14570632u, 0x60237451u, 0x06235147u, 0x57410362u, 0x36201574u};
    p.dmap = dmap[rank];
    for (int i = 0; i < P_; ++i) { p.out[i] = (char*)d_out_all[i]; p.ws[i] = (char*)d_ws_all[i]; }
    hipLaunchKernelGGL(fused_gemm_ar, dim3(NWG), dim3(512), 0, stream, p);
    HIP_CHECK(hipGetLastError());
}
```

```cpp
#include <hip/hip_runtime.h>
#include <rccl/rccl.h>
#include <cstdio>
#include <cstdint>

#define HIP_CHECK(expr) do { hipError_t _e = (expr); if (_e != hipSuccess) { \
    fprintf(stderr, "HIP error %s at %s:%d\n", hipGetErrorString(_e), __FILE__, __LINE__); \
    return; } } while (0)

namespace {

#define LAS __attribute__((address_space(3)))
typedef short bf16x8 __attribute__((ext_vector_type(8)));
typedef float f32x4 __attribute__((ext_vector_type(4)));
typedef unsigned u32x4 __attribute__((ext_vector_type(4)));
typedef __attribute__((address_space(1))) unsigned gu32;

constexpr int M_ = 4096, K_ = 4096, N_ = 16384, P_ = 8, KS_ = K_ / P_;
#define DW_ 832
#define CPD_ 4
constexpr int DW = DW_;
constexpr int ND = (DW + 255) / 256;
constexpr int LW = DW - (ND - 1) * 256;
constexpr int COM0 = P_ * DW;
constexpr int NCOM = (N_ - COM0) / 256;
constexpr int NWP = ND + NCOM;
static_assert(DW % 64 == 0 && DW >= 512 && DW <= 1024, "columns");
constexpr int NXP = M_ / 256;
constexpr int CPD = CPD_;
constexpr int NCOPY = 8 * CPD;
constexpr int NWG = 256;
constexpr int GPX = NWG / 8 - CPD;
constexpr int NROUND = (4 * NWP + 2 * GPX - 1) / (2 * GPX);
static_assert(NROUND <= 3 && 4 * ND <= 2 * GPX && CPD <= 4, "schedule");
constexpr int CW = CPD * 8;
constexpr int first_wp(int r) { return (r * GPX + 1) / 2 < NWP ? (r * GPX + 1) / 2 : NWP; }
constexpr int F1 = first_wp(1), F2 = first_wp(2);
constexpr int NS0 = NXP + F1, NS1 = F2 - F1, NS2 = NWP - F2;
constexpr int SB1 = 8 * NS0, SB2 = SB1 + 8 * NS1, SB3 = SB2 + 8 * NS2;
constexpr int NBI = 4 * SB3;
constexpr int NBO = M_ / 4;
constexpr int STEPS_IN = (NBI + CW - 1) / CW, STEPS_ALL = (NBI + NBO + CW - 1) / CW;
constexpr int STEP_OUT0 = NBI / CW;
constexpr unsigned PSH = 256;
static_assert(STEPS_ALL < (int)PSH && NS1 > 0 && NS2 > 0, "steps");
__host__ __device__ constexpr int steps_for(int st) {
    return (4 * (st < 8 ? (st + 1) * NS0 : st < 16 ? SB1 + (st - 7) * NS1 : SB2 + (st - 15) * NS2) + CW - 1) / CW;
}

constexpr size_t OFF_PROG = 0;
constexpr size_t OFF_TMO = 4096;
constexpr size_t OFF_DONE = 4096 + 256;
constexpr size_t OFF_CALL = 8192;
constexpr size_t OFF_CLAIM = 16384;
constexpr size_t OFF_DBG = 65536;
constexpr size_t OFF_IMG = 1u << 20;
constexpr size_t REGION = (size_t)SB3 * 32768;
constexpr size_t IMG_BYTES = 8 * REGION;
constexpr size_t WS_NEED = OFF_IMG + 2 * IMG_BYTES;
__host__ __device__ constexpr unsigned batch_off(unsigned n) { return (n >> 5) * 262144u + (n & 31u) * 2048u; }
constexpr unsigned long long DEADLINE_TICKS = 100000000ull;

constexpr int HTB = 16384;
constexpr int LDS_BYTES = 8 * HTB + 256;
constexpr int AUX_SC1 = 16, AUX_SYS = 17;
constexpr int RSRC3 = 0x00020000;

struct Params {
    const char* x; const char* w;
    char* out[P_]; char* ws[P_];
    int rank;
    unsigned dmap;
};

__device__ __forceinline__ unsigned ld_sys(const char* p) { return __hip_atomic_load((gu32*)p, __ATOMIC_RELAXED, __HIP_MEMORY_SCOPE_SYSTEM); }
__device__ __forceinline__ unsigned ld_agent(const char* p) { return __hip_atomic_load((gu32*)p, __ATOMIC_RELAXED, __HIP_MEMORY_SCOPE_AGENT); }
__device__ __forceinline__ void st_sys(char* p, unsigned v) { __hip_atomic_store((gu32*)p, v, __ATOMIC_RELAXED, __HIP_MEMORY_SCOPE_SYSTEM); }
__device__ __forceinline__ void st_agent(char* p, unsigned v) { __hip_atomic_store((gu32*)p, v, __ATOMIC_RELAXED, __HIP_MEMORY_SCOPE_AGENT); }
__device__ __forceinline__ unsigned uni(unsigned v) { return (unsigned)__builtin_amdgcn_readfirstlane((int)v); }
__device__ __forceinline__ char* uni_ptr(const char* q) {
    const unsigned long long a = (unsigned long long)q;
    return (char*)(((unsigned long long)uni((unsigned)(a >> 32)) << 32) | uni((unsigned)a));
}

#define STAMP(ws, idx) do { } while (0)

__device__ __forceinline__ unsigned wave_wait_ge(const char* lane_word, unsigned need, char* tmo, unsigned call) {
    const unsigned long long t0 = __builtin_amdgcn_s_memrealtime();
    unsigned v;
    for (;;) {
        const unsigned tw = ld_agent(tmo);
        v = ld_sys(lane_word);
        if (__all(v >= need)) break;
        if (uni(tw) == call || __builtin_amdgcn_s_memrealtime() - t0 > DEADLINE_TICKS) {
            if ((threadIdx.x & 63) == 0) st_agent(tmo, call);
            return 0u;
        }
        __builtin_amdgcn_s_sleep(8);
    }
#pragma unroll
    for (int o = 32; o >= 1; o >>= 1) { const unsigned u = (unsigned)__shfl_xor((int)v, o); v = u < v ? u : v; }
    return uni(v);
}

__device__ __forceinline__ unsigned cvt_pk_bf16(float lo, float hi) { unsigned r; asm("v_cvt_pk_bf16_f32 %0, %1, %2" : "=v"(r) : "v"(lo), "v"(hi)); return r; }
__device__ __forceinline__ float gelu(float y) {
    constexpr float k1 = -2.0f * 0.7978845608028654f * 1.4426950408889634f, k2 = k1 * 0.044715f;
    const float t = y * __builtin_fmaf(y * y, k2, k1);
    return y * __builtin_amdgcn_rcpf(1.0f + __builtin_amdgcn_exp2f(t));
}

struct Batch {
    const char* sp; char* dp;
    unsigned sn, dn;
    unsigned s1, d1;
    unsigned a0, a1, c0, c1;
    bool tr;
};
__device__ __forceinline__ int stage_of(int slot) {
    if (slot < SB1) return slot / NS0;
    if (slot < SB2) return 8 + (slot - SB1) / NS1;
    if (slot < SB3) return 16 + (slot - SB2) / NS2;
    return 24;
}

__device__ __forceinline__ unsigned copier(const Params& p, LAS unsigned char* lds, const char* cw) {
    const int j = blockIdx.x >> 3;
    const int rank = p.rank;
    const int wid = (int)uni(threadIdx.x >> 6), lane = threadIdx.x & 63;
    char* const myws = p.ws[rank];
    char* const myout = uni_ptr(p.out[rank]);
    constexpr unsigned OOB = 0x80000000u;
    const unsigned vL = (unsigned)lane * 16u;
    const unsigned vL1 = vL + 1024u < DW * 2u ? vL + 1024u : OOB;
    const unsigned vX = (unsigned)(lane >> 3) * (KS_ * 2u) + (unsigned)((lane & 7) ^ (lane >> 4)) * 16u;
    const unsigned vW = (unsigned)(lane & 15) * 16u + (unsigned)(lane >> 4) * (8u * N_ * 2u);
    LAS unsigned* bc = (LAS unsigned*)lds;
    int dst = 0, total = NBI; unsigned img = 0; char* dws = myws; char* dout = myout;

    const char* const px = uni_ptr(p.x);
    const char* const pw = uni_ptr(p.w);
    auto decode = [&](int n) {
        Batch b;
        b.tr = false;
        if (n >= NBI) {
            const unsigned off = n < total ? (unsigned)(n - NBI) * (N_ * 2u) + (unsigned)rank * (DW * 2u) : (OOB >> 1);
            b.sp = myout; b.dp = dout; b.sn = b.dn = M_ * N_ * 2; b.s1 = b.d1 = (M_ / 4) * (N_ * 2u);
            b.a0 = b.c0 = vL + off; b.a1 = b.c1 = vL1 + off;
            return b;
        }
        const int slot = n >> 2, sub = n & 3, st = stage_of(slot);
        const int g = st & 7, r = st >> 3;
        const int pan = slot - (r == 0 ? g * NS0 : r == 1 ? SB1 + g * NS1 : SB2 + g * NS2);
        b.dp = dws; b.dn = (unsigned)WS_NEED; b.d1 = 65536u;
        b.c0 = vL + (img + batch_off((unsigned)n)); b.c1 = b.c0 + 1024u;
        if (r == 0 && pan < NXP) {
            const unsigned so = (unsigned)(pan * 256 + sub * 64) * (KS_ * 2u) + (unsigned)g * 128u;
            b.sp = px; b.sn = M_ * KS_ * 2; b.s1 = 16u * KS_ * 2u;
            b.a0 = vX + so; b.a1 = (vX ^ 64u) + (so + 8u * KS_ * 2u);
        } else {
            const int lp = r == 0 ? pan - NXP : (r == 1 ? F1 + pan : F2 + pan);
            const int col = (lp < ND ? dst * DW : COM0 - ND * 256) + lp * 256 + (sub >> 1) * 128;
            const unsigned so = (unsigned)(g * 64 + (sub & 1) * 32) * (N_ * 2u) + (unsigned)col * 2u;
            b.sp = pw; b.sn = KS_ * N_ * 2; b.s1 = N_ * 4u; b.tr = true;
            b.a0 = vW + so; b.a1 = b.a0 + N_ * 2u;
            if (lp == ND - 1 && (sub >> 1) * 128 + (lane & 15) * 8 >= LW) {
                b.a0 |= OOB; b.a1 |= OOB; b.c0 |= OOB; b.c1 |= OOB;
            }
        }
        return b;
    };
    auto load2 = [&](u32x4& v0, u32x4& v1, const Batch& b, int h) {
        const __amdgpu_buffer_rsrc_t rs = __builtin_amdgcn_make_buffer_rsrc((void*)b.sp, 0, (int)uni(b.sn), RSRC3);
        v0 = __builtin_amdgcn_raw_buffer_load_b128(rs, b.a0 + (unsigned)h * b.s1, 0, AUX_SC1);
        v1 = __builtin_amdgcn_raw_buffer_load_b128(rs, b.a1 + (unsigned)h * b.s1, 0, AUX_SC1);
    };
    auto store2 = [&](const u32x4& v0, const u32x4& v1, const Batch& b, int h) {
        const __amdgpu_buffer_rsrc_t rd = __builtin_amdgcn_make_buffer_rsrc((void*)b.dp, 0, (int)uni(b.dn), RSRC3);
        __builtin_amdgcn_raw_buffer_store_b128(v0, rd, b.c0 + (unsigned)h * b.d1, 0, AUX_SYS);
        __builtin_amdgcn_raw_buffer_store_b128(v1, rd, b.c1 + (unsigned)h * b.d1, 0, AUX_SYS);
    };
    auto arrange = [&](u32x4 (&t)[8], const u32x4 (&v)[8], const Batch& b) {
        if (b.tr) {
#pragma unroll
            for (int c = 0; c < 8; ++c)
#pragma unroll
                for (int e = 0; e < 4; ++e)
                    t[c][e] = __builtin_amdgcn_perm(v[2 * e + 1][c >> 1], v[2 * e][c >> 1], (c & 1) ? 0x07060302u : 0x05040100u);
        } else {
#pragma unroll
            for (int c = 0; c < 8; ++c) t[c] = v[c];
        }
#pragma unroll
        for (int c = 0; c < 8; ++c) asm volatile("" : "+v"(t[c]));
    };
#define WIN 6
    constexpr int VM0 = 0x0070, VMW = 0x0070 + WIN;
    static_assert(WIN == 6 || WIN == 10, "window");
#define PACE(imm) do { asm volatile("" ::: "memory"); __builtin_amdgcn_s_waitcnt(imm); __builtin_amdgcn_s_barrier(); \
        asm volatile("" ::: "memory"); __builtin_amdgcn_sched_barrier(0); } while (0)

    u32x4 R[8], T[8], Pa[4], Pb[4];
    const int n0 = j * 8 + wid;
    static_assert(2 * CW <= 4 * NXP, "the first two steps are x batches: their sources do not depend on the destination");
    Batch b0 = decode(n0), b1 = decode(n0 + CW);
    __builtin_amdgcn_s_waitcnt(VM0);
    const unsigned callv = ld_agent(cw);
    load2(Pa[0], Pa[1], b0, 0); load2(Pa[2], Pa[3], b0, 1); load2(R[4], R[5], b0, 2); load2(R[6], R[7], b0, 3);
    load2(Pb[0], Pb[1], b1, 0); load2(Pb[2], Pb[3], b1, 1);
    const unsigned call = uni(callv) + 1u;
    if (threadIdx.x == 0) {
        const unsigned xcc = __builtin_amdgcn_s_getreg((20 << 0) | (0 << 6) | ((4 - 1) << 11)) & 7u;
        unsigned d = (p.dmap >> (xcc * 4u)) & 7u;
        for (int i = 0; i < 7; ++i) {
            if (__hip_atomic_fetch_max((gu32*)(myws + OFF_CLAIM) + (j * 8 + d) * 32, call, __ATOMIC_RELAXED, __HIP_MEMORY_SCOPE_AGENT) < call) break;
            d = (d + 1u) & 7u;
        }
        bc[0] = d;
        bc[1] = 0u;
    }
    PACE(0x0074);
    dst = (int)(uni(bc[0]) & 7u);
    const bool self = dst == rank;
    dws = uni_ptr(p.ws[dst]); dout = uni_ptr(p.out[dst]);
    img = (unsigned)OFF_IMG + (call & 1u) * (unsigned)IMG_BYTES + (unsigned)rank * (unsigned)REGION;
    total = self ? NBI : NBI + NBO;
    b0.dp = b1.dp = dws; b0.c0 += img; b0.c1 += img; b1.c0 += img; b1.c1 += img;
    const int steps = self ? STEPS_IN : STEPS_ALL;
    char* const flag = dws + OFF_PROG + (unsigned)(lane & 7) * 256u + (unsigned)(rank * 4 + j) * 4u;
    const unsigned tiles_need = call * (unsigned)(NXP * ND * 8);
    auto publish = [&](int done) {
        if (wid == 0 && lane < 8) st_sys(flag, call * PSH + (unsigned)done);
    };
    int k = 0, st_prev = 0;
#define COPY_STEP(PC) { \
        R[0] = PC[0]; R[1] = PC[1]; R[2] = PC[2]; R[3] = PC[3]; \
        arrange(T, R, b0); \
        if (wid == 0) { STAMP(myws, 2 + (k * CW >> 5)); if (k == STEP_OUT0) STAMP(myws, 121); } \
        const bool chk = k + 2 == STEP_OUT0 && !self; \
        unsigned done = 0; \
        if (chk) done = ld_agent(myws + OFF_DONE); \
        load2(R[4], R[5], b1, 2); store2(T[0], T[1], b0, 0); \
        const Batch b2 = decode(n0 + (k + 2) * CW); \
        const int st = stage_of(k * CW >> 2); \
        PACE(VMW); \
        load2(R[6], R[7], b1, 3); store2(T[2], T[3], b0, 1); \
        PACE(VMW); \
        if (WIN == 6 && st != st_prev) publish(k); \
        if (chk && uni(done) < tiles_need) { \
            if (wave_wait_ge(myws + OFF_DONE, tiles_need, myws + OFF_TMO, call) == 0u && lane == 0) bc[1] = 1u; \
        } \
        load2(PC[0], PC[1], b2, 0); store2(T[4], T[5], b0, 2); \
        PACE(VMW); \
        if (WIN == 10 && st != st_prev) publish(k); \
        st_prev = st; \
        load2(PC[2], PC[3], b2, 1); store2(T[6], T[7], b0, 3); \
        if (++k == steps) break; \
        PACE(VMW); \
        b0 = b1; b1 = b2; }
    for (;;) {
        COPY_STEP(Pa)
        COPY_STEP(Pb)
    }
#undef COPY_STEP
    PACE(VM0);
    if (wid == 0) STAMP(myws, 123);
    if (uni(bc[1]) == 0u) publish(steps);
#undef PACE

    if (self && j == 0 && wid == 0) {
        int src = (lane & 31) >> 2;
        if (src == rank) src = (rank + 1) & 7;
        const char* word = myws + OFF_PROG + (unsigned)(src * 4 + (lane & 3) % CPD) * 4u;
        const unsigned got = wave_wait_ge(word, call * PSH + (unsigned)STEPS_ALL, myws + OFF_TMO, call);
        if (got == 0u && lane == 0) *(unsigned short*)(myout + (size_t)rank * DW * 2) = 0x7fc0u;
        STAMP(myws, 120);
    }
    return call;
}

struct Unit { int rb, lp, slot, ns, st; };
__device__ __forceinline__ bool unit_of(int r, int q, int i, Unit& u) {
    const int t = (2 * r + (q & 1)) * GPX + i;
    if (t >= 4 * NWP) return false;
    u.rb = (q >> 1) * 4 + (t & 3);
    u.lp = t >> 2;
    if (u.lp < F1) { u.slot = NXP + u.lp; u.ns = NS0; u.st = 0; }
    else if (u.lp < F2) { u.slot = SB1 + u.lp - F1; u.ns = NS1; u.st = 8; }
    else { u.slot = SB2 + u.lp - F2; u.ns = NS2; u.st = 16; }
    return true;
}

__device__ __forceinline__ void gemm(const Params& p, LAS unsigned char* lds, unsigned call) {
    const int tid = threadIdx.x, wid = (int)uni(tid >> 6), lane = tid & 63, wr = wid >> 2, wc = wid & 3;
    const int q = blockIdx.x & 7, gi = (blockIdx.x >> 3) - CPD;
    const int rank = p.rank;
    char* const myws = p.ws[rank];
    const char* const img = myws + OFF_IMG + (size_t)(call & 1u) * IMG_BYTES;
    const unsigned voff = (unsigned)lane * 16u + (unsigned)(wid & 1) * 2048u + (unsigned)((wid >> 1) & 1) * 1024u + (unsigned)(wid >> 2) * 65536u;
    const unsigned ldsw = (unsigned)wid * 1024u;
    const int aoff = wr * 8192 + lane * 16;
    const int boff0 = (wc >> 1) * 8192 + (wc & 1) * 1024 + ((lane >> 3) & 1) * 2048 + (lane & 7) * 128 + (((lane >> 4) ^ ((lane >> 1) & 7)) * 16);
    const int boff1 = boff0 ^ 64;
    LAS unsigned* bc = (LAS unsigned*)(lds + 8 * HTB);
    unsigned known = 0; bool bad = false;
    constexpr size_t kstep = REGION, hstep = 131072;
    constexpr int nt = K_ / 64;

#define SA(b, h) (((b) * 2 + (h)) * HTB)
#define SB(b, h) ((4 + (b) * 2 + (h)) * HTB)
#define STAGE(bufoff, gbase) do { _Pragma("unroll") for (int _i = 0; _i < 2; ++_i) \
        __builtin_amdgcn_global_load_lds((const unsigned*)((gbase) + voff + _i * 4096), (LAS unsigned*)(lds + (bufoff) + ldsw + _i * 8192), 16, 0, AUX_SYS); } while (0)
#define LDA(dst, b, h) do { _Pragma("unroll") for (int m = 0; m < 4; ++m) _Pragma("unroll") for (int k = 0; k < 2; ++k) dst[m][k] = *(const LAS bf16x8*)(lds + SA(b, h) + aoff + m * 2048 + k * 1024); } while (0)
#define LDB(dst, b, h) do { _Pragma("unroll") for (int n = 0; n < 2; ++n) _Pragma("unroll") for (int k = 0; k < 2; ++k) dst[n][k] = *(const LAS bf16x8*)(lds + SB(b, h) + (k ? boff1 : boff0) + n * 4096); } while (0)
#define MMA(ai, bj, At, Bt) do { __builtin_amdgcn_s_setprio(1); _Pragma("unroll") for (int m = 0; m < 4; ++m) _Pragma("unroll") for (int n = 0; n < 2; ++n) _Pragma("unroll") for (int k = 0; k < 2; ++k) \
        acc[ai][bj][m][n] = __builtin_amdgcn_mfma_f32_16x16x32_bf16(Bt[n][k], At[m][k], acc[ai][bj][m][n], 0, 0, 0); __builtin_amdgcn_s_setprio(0); } while (0)
#define WAIT_V(n) asm volatile("s_waitcnt vmcnt(" #n ")" ::: "memory")
#define WAIT_L(n) asm volatile("s_waitcnt lgkmcnt(" #n ")" ::: "memory")
#define BAR __builtin_amdgcn_s_barrier()
#define SCHED __builtin_amdgcn_sched_barrier(0)
#define READY(st) do { const unsigned _need = call * PSH + (unsigned)steps_for(st); if (_need > known) { \
        if (wid == 0) { \
            unsigned _got = wave_wait_ge(myws + OFF_PROG + (unsigned)q * 256u + (unsigned)((lane & 28) + (lane & 3) % CPD) * 4u, _need, myws + OFF_TMO, call); \
            if (lane == 0) bc[0] = _got; \
            asm volatile("s_waitcnt vmcnt(0) lgkmcnt(0)" ::: "memory"); \
            STAMP(myws, 2 + (st)); \
        } \
        asm volatile("" ::: "memory"); BAR; asm volatile("" ::: "memory"); \
        known = uni(bc[0]); if (known == 0u) { bad = true; known = 0xffffffffu; } \
        asm volatile("s_waitcnt lgkmcnt(0)" ::: "memory"); } } while (0)

    Unit u, nu; int ui = 0;
    if (!unit_of(0, q, gi, u)) return;
    f32x4 acc[2][2][4][2];
#pragma unroll
    for (int a = 0; a < 2; ++a)
#pragma unroll
        for (int b = 0; b < 2; ++b)
#pragma unroll
            for (int m = 0; m < 4; ++m)
#pragma unroll
                for (int n = 0; n < 2; ++n) acc[a][b][m][n] = (f32x4){0.f, 0.f, 0.f, 0.f};
    bf16x8 At[4][2], B0[2][2], B1[2][2];
    const char* cA = img + batch_off(4u * (unsigned)u.slot);
    const char* cB = img + batch_off(4u * (unsigned)u.rb);
    READY(u.st);
    STAGE(SB(0, 0), cB); STAGE(SA(0, 0), cA); STAGE(SB(0, 1), cB + hstep); STAGE(SA(0, 1), cA + hstep);
    if (wr == 1) BAR;
    WAIT_V(4); BAR;
    STAGE(SB(1, 0), cB + kstep); STAGE(SA(1, 0), cA + kstep); STAGE(SB(1, 1), cB + hstep + kstep);
    WAIT_V(6); BAR;
    for (;;) {
        const bool has_next = unit_of(ui + 1, q, gi, nu);
        if (!has_next) nu = u;
        for (int t = 0; t < nt; t += 2) {
            const bool last = (t == nt - 2);
            const char* a1 = cA + kstep;
            const char* a2 = cA + 2 * kstep; const char* b2 = cB + 2 * kstep;
            if ((t & 7) == 6) {
                const int g = (t + 2) >> 3;
                a2 = img + batch_off(4u * (unsigned)(last ? nu.slot : u.slot + g * u.ns));
                b2 = img + batch_off(4u * (unsigned)(last ? nu.rb : u.rb + g * NS0));
                READY(last ? nu.st : u.st + g);
            }
            const char* a3 = a2 + kstep; const char* b3 = b2 + kstep;
            cA = a2; cB = b2;
            LDB(B0, 0, 0); SCHED; LDA(At, 0, 0); STAGE(SA(1, 1), a1 + hstep);
            WAIT_L(8); BAR; WAIT_L(0); MMA(0, 0, At, B0); BAR; SCHED;
            LDB(B1, 0, 1); STAGE(SB(0, 0), b2);
            BAR; WAIT_L(0); MMA(0, 1, At, B1); BAR;
            LDA(At, 0, 1); STAGE(SA(0, 0), a2);
            BAR; WAIT_L(0); MMA(1, 0, At, B0); BAR; SCHED;
            STAGE(SB(0, 1), b2 + hstep);
            WAIT_V(6); BAR; MMA(1, 1, At, B1); BAR;
            LDB(B0, 1, 0); SCHED; LDA(At, 1, 0); STAGE(SA(0, 1), a2 + hstep);
            WAIT_L(8); BAR; WAIT_L(0); MMA(0, 0, At, B0); BAR; SCHED;
            LDB(B1, 1, 1); STAGE(SB(1, 0), b3);
            BAR; WAIT_L(0); MMA(0, 1, At, B1); BAR;
            LDA(At, 1, 1); STAGE(SA(1, 0), a3);
            BAR; WAIT_L(0); MMA(1, 0, At, B0); BAR; SCHED;
            STAGE(SB(1, 1), b3 + hstep);
            WAIT_V(6); BAR; MMA(1, 1, At, B1); BAR;
        }
        if (wid == 0) STAMP(myws, 32 + 2 * ui);
        {
            int tz = threadIdx.x; asm volatile("" : "+v"(tz));
            const int fr = tz & 15, fq = (tz >> 4) & 3;
            const bool dist = u.lp < ND;
            const int col = (dist ? rank * DW : COM0 - ND * 256) + u.lp * 256 + wr * 128;
            const __amdgpu_buffer_rsrc_t rO = __builtin_amdgcn_make_buffer_rsrc((void*)p.out[rank], 0, M_ * N_ * 2, RSRC3);
            const unsigned so = (unsigned)(u.rb * 256 + wc * 64) * (N_ * 2u) + (unsigned)col * 2u;
            unsigned vo = (unsigned)(4 * fq) * (N_ * 2u) + (unsigned)fr * 16u + so;
            if (u.lp == ND - 1 && wr * 128 + fr * 8 >= LW) vo |= 0x80000000u;
#pragma unroll
            for (int bj = 0; bj < 2; ++bj)
#pragma unroll
                for (int n = 0; n < 2; ++n)
#pragma unroll
                    for (int r = 0; r < 4; ++r) {
                        u32x4 o;
                        o[0] = cvt_pk_bf16(gelu(acc[0][bj][0][n][r]), gelu(acc[0][bj][1][n][r]));
                        o[1] = cvt_pk_bf16(gelu(acc[0][bj][2][n][r]), gelu(acc[0][bj][3][n][r]));
                        o[2] = cvt_pk_bf16(gelu(acc[1][bj][0][n][r]), gelu(acc[1][bj][1][n][r]));
                        o[3] = cvt_pk_bf16(gelu(acc[1][bj][2][n][r]), gelu(acc[1][bj][3][n][r]));
                        if (bad) o = (u32x4){0x7fc07fc0u, 0x7fc07fc0u, 0x7fc07fc0u, 0x7fc07fc0u};
                        __builtin_amdgcn_raw_buffer_store_b128(o, rO, vo + (unsigned)(bj * 32 + n * 16 + r) * (N_ * 2u), 0, AUX_SC1);
                    }
            if (dist) {
                WAIT_V(0);
                if (lane == 0) __hip_atomic_fetch_add((gu32*)(myws + OFF_DONE), 1u, __ATOMIC_RELAXED, __HIP_MEMORY_SCOPE_AGENT);
            }
        }
        if (wid == 0) STAMP(myws, 33 + 2 * ui);
        if (!has_next) break;
#pragma unroll
        for (int a = 0; a < 2; ++a)
#pragma unroll
            for (int b = 0; b < 2; ++b)
#pragma unroll
                for (int m = 0; m < 4; ++m)
#pragma unroll
                    for (int n = 0; n < 2; ++n) acc[a][b][m][n] = (f32x4){0.f, 0.f, 0.f, 0.f};
        u = nu; ++ui;
    }
    WAIT_V(0);
    if (wr == 0) BAR;
#undef SA
#undef SB
#undef STAGE
#undef LDA
#undef LDB
#undef MMA
#undef READY
}

__global__ __launch_bounds__(512, 2) void fused_gemm_ar(const Params p) {
    __shared__ __attribute__((aligned(16))) unsigned char shm[LDS_BYTES];
    LAS unsigned char* lds = (LAS unsigned char*)shm;
    char* const myws = p.ws[p.rank];
    char* const cw = myws + OFF_CALL + blockIdx.x * 4u;
    if (threadIdx.x == 0) STAMP(myws, 0);
    unsigned call;
    if (blockIdx.x < NCOPY) {
        call = copier(p, lds, cw);
    } else {
        call = uni(ld_agent(cw)) + 1u;
        gemm(p, lds, call);
    }
    asm volatile("s_waitcnt vmcnt(0) lgkmcnt(0)" ::: "memory");
    __builtin_amdgcn_s_barrier();
    if (threadIdx.x == 0) { st_agent(cw, call); STAMP(myws, 1); }
}

}

extern "C" void kernel_launch(int rank, int nranks, ncclComm_t comm, hipStream_t stream,
                              void* const* d_in, const int* in_sizes, int n_in,
                              void* d_out, int out_size, const int* in_dtypes, int out_dtype,
                              void* const* d_out_all, void* const* d_ws_all, size_t ws_bytes) {
    (void)comm; (void)in_dtypes; (void)out_dtype; (void)d_out;
    if (nranks != P_ || (unsigned)rank >= (unsigned)P_ || n_in != 2 || in_sizes[0] != M_ * KS_ || in_sizes[1] != KS_ * N_ ||
        out_size != M_ * N_ || ws_bytes < WS_NEED) {
        fprintf(stderr, "kernel_launch: unexpected problem shape\n");
        return;
    }
    Params p;
    p.x = (const char*)d_in[0]; p.w = (const char*)d_in[1]; p.rank = rank;
    static const unsigned dmap[P_] = {0x71546302u, 0x26307415u, 0x14572360u, 0x14570632u, 0x60237451u, 0x06235147u, 0x57410362u, 0x36201574u};
    p.dmap = dmap[rank];
    for (int i = 0; i < P_; ++i) { p.out[i] = (char*)d_out_all[i]; p.ws[i] = (char*)d_ws_all[i]; }
    hipLaunchKernelGGL(fused_gemm_ar, dim3(NWG), dim3(512), 0, stream, p);
    HIP_CHECK(hipGetLastError());
}
```

```cpp
#include <hip/hip_runtime.h>
#include <rccl/rccl.h>
#include <cstdio>
#include <cstdint>

#define HIP_CHECK(expr) do { hipError_t _e = (expr); if (_e != hipSuccess) { \
    fprintf(stderr, "HIP error %s at %s:%d\n", hipGetErrorString(_e), __FILE__, __LINE__); \
    return; } } while (0)

namespace {

#define LAS __attribute__((address_space(3)))
typedef short bf16x8 __attribute__((ext_vector_type(8)));
typedef float f32x4 __attribute__((ext_vector_type(4)));
typedef unsigned u32x4 __attribute__((ext_vector_type(4)));
typedef __attribute__((address_space(1))) unsigned gu32;

constexpr int M_ = 4096, K_ = 4096, N_ = 16384, P_ = 8, KS_ = K_ / P_;
#define DW_ 832
#define CPD_ 4
constexpr int DW = DW_;
constexpr int ND = (DW + 255) / 256;
constexpr int LW = DW - (ND - 1) * 256;
constexpr int COM0 = P_ * DW;
constexpr int NCOM = (N_ - COM0) / 256;
constexpr int NWP = ND + NCOM;
static_assert(DW % 64 == 0 && DW >= 512 && DW <= 1024, "columns");
constexpr int NXP = M_ / 256;
constexpr int CPD = CPD_;
constexpr int NCOPY = 8 * CPD;
constexpr int NWG = 256;
constexpr int GPX = NWG / 8 - CPD;
constexpr int NROUND = (4 * NWP + 2 * GPX - 1) / (2 * GPX);
static_assert(NROUND <= 3 && 4 * ND <= 2 * GPX && CPD <= 4, "schedule");
constexpr int CW = CPD * 8;
constexpr int first_wp(int r) { return (r * GPX + 1) / 2 < NWP ? (r * GPX + 1) / 2 : NWP; }
constexpr int F1 = first_wp(1), F2 = first_wp(2);
constexpr int NS0 = NXP + F1, NS1 = F2 - F1, NS2 = NWP - F2;
constexpr int SB1 = 8 * NS0, SB2 = SB1 + 8 * NS1, SB3 = SB2 + 8 * NS2;
constexpr int NBI = 4 * SB3;
constexpr int NBO = M_ / 4;
constexpr int STEPS_IN = (NBI + CW - 1) / CW, STEPS_ALL = (NBI + NBO + CW - 1) / CW;
constexpr int STEP_OUT0 = NBI / CW;
constexpr unsigned PSH = 256;
static_assert(STEPS_ALL < (int)PSH && NS1 > 0 && NS2 > 0, "steps");
__host__ __device__ constexpr int steps_for(int st) {
    return (4 * (st < 8 ? (st + 1) * NS0 : st < 16 ? SB1 + (st - 7) * NS1 : SB2 + (st - 15) * NS2) + CW - 1) / CW;
}

constexpr size_t OFF_PROG = 0;
constexpr size_t OFF_TMO = 4096;
constexpr size_t OFF_DONE = 4096 + 256;
constexpr size_t OFF_CALL = 8192;
constexpr size_t OFF_CLAIM = 16384;
constexpr size_t OFF_DBG = 65536;
constexpr size_t OFF_IMG = 1u << 20;
constexpr size_t REGION = (size_t)SB3 * 32768;
constexpr size_t IMG_BYTES = 8 * REGION;
constexpr size_t WS_NEED = OFF_IMG + 2 * IMG_BYTES;
__host__ __device__ constexpr unsigned batch_off(unsigned n) { return (n >> 5) * 262144u + (n & 31u) * 2048u; }
constexpr unsigned long long DEADLINE_TICKS = 100000000ull;

constexpr int HTB = 16384;
constexpr int LDS_BYTES = 8 * HTB + 256;
constexpr int AUX_SC1 = 16, AUX_SYS = 17;
constexpr int RSRC3 = 0x00020000;

struct Params {
    const char* x; const char* w;
    char* myout; char* myws;
    int rank;
    unsigned dmap;
    struct { char* ws; char* out; } peer[P_];
};

__device__ __forceinline__ unsigned ld_sys(const char* p) { return __hip_atomic_load((gu32*)p, __ATOMIC_RELAXED, __HIP_MEMORY_SCOPE_SYSTEM); }
__device__ __forceinline__ unsigned ld_agent(const char* p) { return __hip_atomic_load((gu32*)p, __ATOMIC_RELAXED, __HIP_MEMORY_SCOPE_AGENT); }
__device__ __forceinline__ void st_sys(char* p, unsigned v) { __hip_atomic_store((gu32*)p, v, __ATOMIC_RELAXED, __HIP_MEMORY_SCOPE_SYSTEM); }
__device__ __forceinline__ void st_agent(char* p, unsigned v) { __hip_atomic_store((gu32*)p, v, __ATOMIC_RELAXED, __HIP_MEMORY_SCOPE_AGENT); }
__device__ __forceinline__ unsigned uni(unsigned v) { return (unsigned)__builtin_amdgcn_readfirstlane((int)v); }
__device__ __forceinline__ char* uni_ptr(const char* q) {
    const unsigned long long a = (unsigned long long)q;
    return (char*)(((unsigned long long)uni((unsigned)(a >> 32)) << 32) | uni((unsigned)a));
}

#define STAMP(ws, idx) do { } while (0)

__device__ __forceinline__ unsigned wave_wait_ge(const char* lane_word, unsigned need, char* tmo, unsigned call) {
    const unsigned long long t0 = __builtin_amdgcn_s_memrealtime();
    unsigned v;
    for (;;) {
        const unsigned tw = ld_agent(tmo);
        v = ld_sys(lane_word);
        if (__all(v >= need)) break;
        if (uni(tw) == call || __builtin_amdgcn_s_memrealtime() - t0 > DEADLINE_TICKS) {
            if ((threadIdx.x & 63) == 0) st_agent(tmo, call);
            return 0u;
        }
        __builtin_amdgcn_s_sleep(8);
    }
#pragma unroll
    for (int o = 32; o >= 1; o >>= 1) { const unsigned u = (unsigned)__shfl_xor((int)v, o); v = u < v ? u : v; }
    return uni(v);
}

__device__ __forceinline__ unsigned cvt_pk_bf16(float lo, float hi) { unsigned r; asm("v_cvt_pk_bf16_f32 %0, %1, %2" : "=v"(r) : "v"(lo), "v"(hi)); return r; }
__device__ __forceinline__ float gelu(float y) {
    constexpr float k1 = -2.0f * 0.7978845608028654f * 1.4426950408889634f, k2 = k1 * 0.044715f;
    const float t = y * __builtin_fmaf(y * y, k2, k1);
    return y * __builtin_amdgcn_rcpf(1.0f + __builtin_amdgcn_exp2f(t));
}

struct Batch {
    const char* sp; char* dp;
    unsigned sn, dn;
    unsigned s1, d1;
    unsigned a0, a1, c0, c1;
    bool tr;
};
__device__ __forceinline__ int stage_of(int slot) {
    if (slot < SB1) return slot / NS0;
    if (slot < SB2) return 8 + (slot - SB1) / NS1;
    if (slot < SB3) return 16 + (slot - SB2) / NS2;
    return 24;
}

__device__ __forceinline__ void copier(const Params& p, LAS unsigned char* lds, char* cw) {
    const int j = blockIdx.x >> 3;
    const int rank = p.rank;
    const int wid = (int)uni(threadIdx.x >> 6), lane = threadIdx.x & 63;
    char* const myws = p.myws;
    char* const myout = uni_ptr(p.myout);
    constexpr unsigned OOB = 0x80000000u;
    const unsigned vL = (unsigned)lane * 16u;
    const unsigned vL1 = vL + 1024u < DW * 2u ? vL + 1024u : OOB;
    const unsigned vX = (unsigned)(lane >> 3) * (KS_ * 2u) + (unsigned)((lane & 7) ^ (lane >> 4)) * 16u;
    const unsigned vW = (unsigned)(lane & 15) * 16u + (unsigned)(lane >> 4) * (8u * N_ * 2u);
    LAS unsigned* bc = (LAS unsigned*)lds;
    int dst = 0, total = NBI; unsigned img = 0; char* dws = myws; char* dout = myout;

    const char* const px = uni_ptr(p.x);
    const char* const pw = uni_ptr(p.w);
    auto decode = [&](int n) {
        Batch b;
        b.tr = false;
        if (n >= NBI) {
            const unsigned off = n < total ? (unsigned)(n - NBI) * (N_ * 2u) + (unsigned)rank * (DW * 2u) : (OOB >> 1);
            b.sp = myout; b.dp = dout; b.sn = b.dn = M_ * N_ * 2; b.s1 = b.d1 = (M_ / 4) * (N_ * 2u);
            b.a0 = b.c0 = vL + off; b.a1 = b.c1 = vL1 + off;
            return b;
        }
        const int slot = n >> 2, sub = n & 3, st = stage_of(slot);
        const int g = st & 7, r = st >> 3;
        const int pan = slot - (r == 0 ? g * NS0 : r == 1 ? SB1 + g * NS1 : SB2 + g * NS2);
        b.dp = dws; b.dn = (unsigned)WS_NEED; b.d1 = 65536u;
        b.c0 = vL + (img + batch_off((unsigned)n)); b.c1 = b.c0 + 1024u;
        if (r == 0 && pan < NXP) {
            const unsigned so = (unsigned)(pan * 256 + sub * 64) * (KS_ * 2u) + (unsigned)g * 128u;
            b.sp = px; b.sn = M_ * KS_ * 2; b.s1 = 16u * KS_ * 2u;
            b.a0 = vX + so; b.a1 = (vX ^ 64u) + (so + 8u * KS_ * 2u);
        } else {
            const int lp = r == 0 ? pan - NXP : (r == 1 ? F1 + pan : F2 + pan);
            const int col = (lp < ND ? dst * DW : COM0 - ND * 256) + lp * 256 + (sub >> 1) * 128;
            const unsigned so = (unsigned)(g * 64 + (sub & 1) * 32) * (N_ * 2u) + (unsigned)col * 2u;
            b.sp = pw; b.sn = KS_ * N_ * 2; b.s1 = N_ * 4u; b.tr = true;
            b.a0 = vW + so; b.a1 = b.a0 + N_ * 2u;
            if (lp == ND - 1 && (sub >> 1) * 128 + (lane & 15) * 8 >= LW) {
                b.a0 |= OOB; b.a1 |= OOB; b.c0 |= OOB; b.c1 |= OOB;
            }
        }
        return b;
    };
    auto load2 = [&](u32x4& v0, u32x4& v1, const Batch& b, int h) {
        const __amdgpu_buffer_rsrc_t rs = __builtin_amdgcn_make_buffer_rsrc((void*)b.sp, 0, (int)uni(b.sn), RSRC3);
        v0 = __builtin_amdgcn_raw_buffer_load_b128(rs, b.a0 + (unsigned)h * b.s1, 0, AUX_SC1);
        v1 = __builtin_amdgcn_raw_buffer_load_b128(rs, b.a1 + (unsigned)h * b.s1, 0, AUX_SC1);
    };
    auto store2 = [&](const u32x4& v0, const u32x4& v1, const Batch& b, int h) {
        const __amdgpu_buffer_rsrc_t rd = __builtin_amdgcn_make_buffer_rsrc((void*)b.dp, 0, (int)uni(b.dn), RSRC3);
        __builtin_amdgcn_raw_buffer_store_b128(v0, rd, b.c0 + (unsigned)h * b.d1, 0, AUX_SYS);
        __builtin_amdgcn_raw_buffer_store_b128(v1, rd, b.c1 + (unsigned)h * b.d1, 0, AUX_SYS);
    };
    auto arrange = [&](u32x4 (&t)[8], const u32x4 (&v)[8], const Batch& b) {
        if (b.tr) {
#pragma unroll
            for (int c = 0; c < 8; ++c)
#pragma unroll
                for (int e = 0; e < 4; ++e)
                    t[c][e] = __builtin_amdgcn_perm(v[2 * e + 1][c >> 1], v[2 * e][c >> 1], (c & 1) ? 0x07060302u : 0x05040100u);
        } else {
#pragma unroll
            for (int c = 0; c < 8; ++c) t[c] = v[c];
        }
#pragma unroll
        for (int c = 0; c < 8; ++c) asm volatile("" : "+v"(t[c]));
    };
#define WIN 6
    constexpr int VM0 = 0x0070, VMW = 0x0070 + WIN;
    static_assert(WIN == 6 || WIN == 10, "window");
#define PACE(imm) do { asm volatile("" ::: "memory"); __builtin_amdgcn_s_waitcnt(imm); __builtin_amdgcn_s_barrier(); \
        asm volatile("" ::: "memory"); __builtin_amdgcn_sched_barrier(0); } while (0)

    u32x4 R[8], T[8], Pa[4], Pb[4];
    const int n0 = j * 8 + wid;
    static_assert(2 * CW <= 4 * NXP, "the first two steps are x batches: their sources do not depend on the destination");
    auto first = [&](int n) {
        Batch b;
        b.tr = false; b.sp = px; b.sn = M_ * KS_ * 2; b.s1 = 16u * KS_ * 2u; b.dp = dws; b.dn = (unsigned)WS_NEED; b.d1 = 65536u;
        b.a0 = vX + (unsigned)n * (64u * KS_ * 2u); b.a1 = (vX ^ 64u) + ((unsigned)n * (64u * KS_ * 2u) + 8u * KS_ * 2u);
        b.c0 = vL + batch_off((unsigned)n); b.c1 = b.c0 + 1024u;
        return b;
    };
    Batch b0 = first(n0), b1 = first(n0 + CW);
    const unsigned pref = (p.dmap >> ((__builtin_amdgcn_s_getreg((20 << 0) | (0 << 6) | ((4 - 1) << 11)) & 7u) * 4u)) & 7u;
    unsigned vz; asm volatile("v_mov_b32 %0, 0" : "=v"(vz));
    gu32* const slots = (gu32*)(myws + OFF_CLAIM) + j * 8 * 32 + vz;
    __builtin_amdgcn_s_waitcnt(VM0);
    unsigned old = 0, callv = 0;
    if (threadIdx.x == 0) {
        old = __hip_atomic_fetch_add(slots + pref * 32, 1u, __ATOMIC_RELAXED, __HIP_MEMORY_SCOPE_AGENT);
        callv = ld_agent(cw);
    }
    load2(T[0], T[1], b0, 0); load2(T[2], T[3], b0, 1);
    dws = p.peer[pref].ws; dout = p.peer[pref].out;
    asm volatile("" : "+s"(dws), "+s"(dout), "+v"(old), "+v"(callv));
    if (threadIdx.x == 0) {
        unsigned d = pref;
        for (int i = 0; i < 7 && old != callv; ++i) {
            __hip_atomic_fetch_sub(slots + d * 32, 1u, __ATOMIC_RELAXED, __HIP_MEMORY_SCOPE_AGENT);
            d = (d + 1u) & 7u;
            old = __hip_atomic_fetch_add(slots + d * 32, 1u, __ATOMIC_RELAXED, __HIP_MEMORY_SCOPE_AGENT);
        }
        bc[0] = d;
        bc[1] = 0u;
        bc[2] = callv + 1u;
        st_agent(cw, callv + 1u);
    }
    PACE(0xc07f);
    dst = (int)(uni(bc[0]) & 7u);
    const unsigned call = uni(bc[2]);
    if (dst != (int)pref) { dws = uni_ptr(p.peer[dst].ws); dout = uni_ptr(p.peer[dst].out); }
    const bool self = dst == rank;
    img = (unsigned)OFF_IMG + (call & 1u) * (unsigned)IMG_BYTES + (unsigned)rank * (unsigned)REGION;
    total = self ? NBI : NBI + NBO;
    b0.dp = b1.dp = dws; b0.c0 += img; b0.c1 += img; b1.c0 += img; b1.c1 += img;
    const int steps = self ? STEPS_IN : STEPS_ALL;
    char* const flag = dws + OFF_PROG + (unsigned)(lane & 7) * 256u + (unsigned)(rank * 4 + j) * 4u;
    const unsigned tiles_need = call * (unsigned)(NXP * ND * 8);
    auto publish = [&](int done) {
        if (wid == 0 && lane < 8) st_sys(flag, call * PSH + (unsigned)done);
    };
#define FENCE __builtin_amdgcn_sched_barrier(0)
    if (wid == 0) STAMP(myws, 2);
    store2(T[0], T[1], b0, 0); FENCE; load2(T[4], T[5], b0, 2); load2(T[6], T[7], b0, 3); FENCE;
    store2(T[2], T[3], b0, 1); FENCE; load2(R[0], R[1], b1, 0); load2(R[2], R[3], b1, 1); FENCE;
    store2(T[4], T[5], b0, 2); FENCE; load2(R[4], R[5], b1, 2); load2(R[6], R[7], b1, 3); FENCE;
    store2(T[6], T[7], b0, 3); FENCE;
#undef FENCE
    b0 = b1; b1 = decode(n0 + 2 * CW);
    load2(Pa[0], Pa[1], b1, 0); load2(Pa[2], Pa[3], b1, 1);
    PACE(VMW);
    arrange(T, R, b0);
    int k = 1, st_prev = 0;
#define COPY_STEP(PC, PN) { \
        if (wid == 0) { STAMP(myws, 2 + (k * CW >> 5)); if (k == STEP_OUT0) STAMP(myws, 121); } \
        const bool chk = k + 2 == STEP_OUT0 && !self; \
        unsigned done = 0; \
        if (chk) done = ld_agent(myws + OFF_DONE); \
        load2(R[4], R[5], b1, 2); store2(T[0], T[1], b0, 0); \
        const Batch b2 = decode(n0 + (k + 2) * CW); \
        const int st = stage_of(k * CW >> 2); \
        PACE(VMW); \
        load2(R[6], R[7], b1, 3); store2(T[2], T[3], b0, 1); \
        PACE(VMW); \
        if (WIN == 6 && st != st_prev) publish(k); \
        if (chk && uni(done) < tiles_need) { \
            if (wave_wait_ge(myws + OFF_DONE, tiles_need, myws + OFF_TMO, call) == 0u && lane == 0) bc[1] = 1u; \
        } \
        load2(PC[0], PC[1], b2, 0); store2(T[4], T[5], b0, 2); \
        PACE(VMW); \
        if (WIN == 10 && st != st_prev) publish(k); \
        st_prev = st; \
        load2(PC[2], PC[3], b2, 1); store2(T[6], T[7], b0, 3); \
        if (++k == steps) break; \
        PACE(VMW); \
        b0 = b1; b1 = b2; \
        R[0] = PN[0]; R[1] = PN[1]; R[2] = PN[2]; R[3] = PN[3]; \
        arrange(T, R, b0); }
    for (;;) {
        COPY_STEP(Pb, Pa)
        COPY_STEP(Pa, Pb)
    }
#undef COPY_STEP
    PACE(VM0);
    if (wid == 0) STAMP(myws, 123);
    if (uni(bc[1]) == 0u) publish(steps);
#undef PACE

    if (self && j == 0 && wid == 0) {
        int src = (lane & 31) >> 2;
        if (src == rank) src = (rank + 1) & 7;
        const char* word = myws + OFF_PROG + (unsigned)(src * 4 + (lane & 3) % CPD) * 4u;
        const unsigned need = call * PSH + (unsigned)STEPS_ALL;
        const unsigned long long t0 = __builtin_amdgcn_s_memrealtime();
        unsigned v0 = ld_sys(word), v1, v2;
        __builtin_amdgcn_s_sleep(4);
        v1 = ld_sys(word);
        for (;;) {
            __builtin_amdgcn_s_sleep(4);
            v2 = ld_sys(word);
            if (__all(v0 >= need)) break;
            __builtin_amdgcn_s_sleep(4);
            v0 = ld_sys(word);
            if (__all(v1 >= need)) break;
            __builtin_amdgcn_s_sleep(4);
            v1 = ld_sys(word);
            if (__all(v2 >= need)) break;
            if (__builtin_amdgcn_s_memrealtime() - t0 > DEADLINE_TICKS) {
                if (lane == 0) { st_agent(myws + OFF_TMO, call); *(unsigned short*)(myout + (size_t)rank * DW * 2) = 0x7fc0u; }
                break;
            }
        }
        STAMP(myws, 120);
    }
    if (wid == 0) STAMP(myws, 1);
}

struct Unit { int rb, lp, slot, ns, st; };
__device__ __forceinline__ bool unit_of(int r, int q, int i, Unit& u) {
    const int t = (2 * r + (q & 1)) * GPX + i;
    if (t >= 4 * NWP) return false;
    u.rb = (q >> 1) * 4 + (t & 3);
    u.lp = t >> 2;
    if (u.lp < F1) { u.slot = NXP + u.lp; u.ns = NS0; u.st = 0; }
    else if (u.lp < F2) { u.slot = SB1 + u.lp - F1; u.ns = NS1; u.st = 8; }
    else { u.slot = SB2 + u.lp - F2; u.ns = NS2; u.st = 16; }
    return true;
}

__device__ __forceinline__ void gemm(const Params& p, LAS unsigned char* lds, unsigned call) {
    const int tid = threadIdx.x, wid = (int)uni(tid >> 6), lane = tid & 63, wr = wid >> 2, wc = wid & 3;
    const int q = blockIdx.x & 7, gi = (blockIdx.x >> 3) - CPD;
    const int rank = p.rank;
    char* const myws = p.myws;
    const char* const img = myws + OFF_IMG + (size_t)(call & 1u) * IMG_BYTES;
    const unsigned voff = (unsigned)lane * 16u + (unsigned)(wid & 1) * 2048u + (unsigned)((wid >> 1) & 1) * 1024u + (unsigned)(wid >> 2) * 65536u;
    const unsigned ldsw = (unsigned)wid * 1024u;
    const int aoff = wr * 8192 + lane * 16;
    const int boff0 = (wc >> 1) * 8192 + (wc & 1) * 1024 + ((lane >> 3) & 1) * 2048 + (lane & 7) * 128 + (((lane >> 4) ^ ((lane >> 1) & 7)) * 16);
    const int boff1 = boff0 ^ 64;
    LAS unsigned* bc = (LAS unsigned*)(lds + 8 * HTB);
    unsigned known = 0; bool bad = false;
    constexpr size_t kstep = REGION, hstep = 131072;
    constexpr int nt = K_ / 64;

#define SA(b, h) (((b) * 2 + (h)) * HTB)
#define SB(b, h) ((4 + (b) * 2 + (h)) * HTB)
#define STAGE(bufoff, gbase) do { _Pragma("unroll") for (int _i = 0; _i < 2; ++_i) \
        __builtin_amdgcn_global_load_lds((const unsigned*)((gbase) + voff + _i * 4096), (LAS unsigned*)(lds + (bufoff) + ldsw + _i * 8192), 16, 0, AUX_SYS); } while (0)
#define LDA(dst, b, h) do { _Pragma("unroll") for (int m = 0; m < 4; ++m) _Pragma("unroll") for (int k = 0; k < 2; ++k) dst[m][k] = *(const LAS bf16x8*)(lds + SA(b, h) + aoff + m * 2048 + k * 1024); } while (0)
#define LDB(dst, b, h) do { _Pragma("unroll") for (int n = 0; n < 2; ++n) _Pragma("unroll") for (int k = 0; k < 2; ++k) dst[n][k] = *(const LAS bf16x8*)(lds + SB(b, h) + (k ? boff1 : boff0) + n * 4096); } while (0)
#define MMA(ai, bj, At, Bt) do { __builtin_amdgcn_s_setprio(1); _Pragma("unroll") for (int m = 0; m < 4; ++m) _Pragma("unroll") for (int n = 0; n < 2; ++n) _Pragma("unroll") for (int k = 0; k < 2; ++k) \
        acc[ai][bj][m][n] = __builtin_amdgcn_mfma_f32_16x16x32_bf16(Bt[n][k], At[m][k], acc[ai][bj][m][n], 0, 0, 0); __builtin_amdgcn_s_setprio(0); } while (0)
#define WAIT_V(n) asm volatile("s_waitcnt vmcnt(" #n ")" ::: "memory")
#define WAIT_L(n) asm volatile("s_waitcnt lgkmcnt(" #n ")" ::: "memory")
#define BAR __builtin_amdgcn_s_barrier()
#define SCHED __builtin_amdgcn_sched_barrier(0)
#define READY(st) do { const unsigned _need = call * PSH + (unsigned)steps_for(st); if (_need > known) { \
        if (wid == 0) { \
            unsigned _got = wave_wait_ge(myws + OFF_PROG + (unsigned)q * 256u + (unsigned)((lane & 28) + (lane & 3) % CPD) * 4u, _need, myws + OFF_TMO, call); \
            if (lane == 0) bc[0] = _got; \
            asm volatile("s_waitcnt vmcnt(0) lgkmcnt(0)" ::: "memory"); \
            STAMP(myws, 2 + (st)); \
        } \
        asm volatile("" ::: "memory"); BAR; asm volatile("" ::: "memory"); \
        known = uni(bc[0]); if (known == 0u) { bad = true; known = 0xffffffffu; } \
        asm volatile("s_waitcnt lgkmcnt(0)" ::: "memory"); } } while (0)

    Unit u, nu; int ui = 0;
    if (!unit_of(0, q, gi, u)) return;
    f32x4 acc[2][2][4][2];
#pragma unroll
    for (int a = 0; a < 2; ++a)
#pragma unroll
        for (int b = 0; b < 2; ++b)
#pragma unroll
            for (int m = 0; m < 4; ++m)
#pragma unroll
                for (int n = 0; n < 2; ++n) acc[a][b][m][n] = (f32x4){0.f, 0.f, 0.f, 0.f};
    bf16x8 At[4][2], B0[2][2], B1[2][2];
    const char* cA = img + batch_off(4u * (unsigned)u.slot);
    const char* cB = img + batch_off(4u * (unsigned)u.rb);
    READY(u.st);
    STAGE(SB(0, 0), cB); STAGE(SA(0, 0), cA); STAGE(SB(0, 1), cB + hstep); STAGE(SA(0, 1), cA + hstep);
    if (wr == 1) BAR;
    WAIT_V(4); BAR;
    STAGE(SB(1, 0), cB + kstep); STAGE(SA(1, 0), cA + kstep); STAGE(SB(1, 1), cB + hstep + kstep);
    WAIT_V(6); BAR;
    for (;;) {
        const bool has_next = unit_of(ui + 1, q, gi, nu);
        if (!has_next) nu = u;
        for (int t = 0; t < nt; t += 2) {
            const bool last = (t == nt - 2);
            const char* a1 = cA + kstep;
            const char* a2 = cA + 2 * kstep; const char* b2 = cB + 2 * kstep;
            if ((t & 7) == 6) {
                const int g = (t + 2) >> 3;
                a2 = img + batch_off(4u * (unsigned)(last ? nu.slot : u.slot + g * u.ns));
                b2 = img + batch_off(4u * (unsigned)(last ? nu.rb : u.rb + g * NS0));
                READY(last ? nu.st : u.st + g);
            }
            const char* a3 = a2 + kstep; const char* b3 = b2 + kstep;
            cA = a2; cB = b2;
            LDB(B0, 0, 0); SCHED; LDA(At, 0, 0); STAGE(SA(1, 1), a1 + hstep);
            WAIT_L(8); BAR; WAIT_L(0); MMA(0, 0, At, B0); BAR; SCHED;
            LDB(B1, 0, 1); STAGE(SB(0, 0), b2);
            BAR; WAIT_L(0); MMA(0, 1, At, B1); BAR;
            LDA(At, 0, 1); STAGE(SA(0, 0), a2);
            BAR; WAIT_L(0); MMA(1, 0, At, B0); BAR; SCHED;
            STAGE(SB(0, 1), b2 + hstep);
            WAIT_V(6); BAR; MMA(1, 1, At, B1); BAR;
            LDB(B0, 1, 0); SCHED; LDA(At, 1, 0); STAGE(SA(0, 1), a2 + hstep);
            WAIT_L(8); BAR; WAIT_L(0); MMA(0, 0, At, B0); BAR; SCHED;
            LDB(B1, 1, 1); STAGE(SB(1, 0), b3);
            BAR; WAIT_L(0); MMA(0, 1, At, B1); BAR;
            LDA(At, 1, 1); STAGE(SA(1, 0), a3);
            BAR; WAIT_L(0); MMA(1, 0, At, B0); BAR; SCHED;
            STAGE(SB(1, 1), b3 + hstep);
            WAIT_V(6); BAR; MMA(1, 1, At, B1); BAR;
        }
        if (wid == 0) STAMP(myws, 32 + 2 * ui);
        {
            int tz = threadIdx.x; asm volatile("" : "+v"(tz));
            const int fr = tz & 15, fq = (tz >> 4) & 3;
            const bool dist = u.lp < ND;
            const int col = (dist ? rank * DW : COM0 - ND * 256) + u.lp * 256 + wr * 128;
            const __amdgpu_buffer_rsrc_t rO = __builtin_amdgcn_make_buffer_rsrc((void*)p.myout, 0, M_ * N_ * 2, RSRC3);
            const unsigned so = (unsigned)(u.rb * 256 + wc * 64) * (N_ * 2u) + (unsigned)col * 2u;
            unsigned vo = (unsigned)(4 * fq) * (N_ * 2u) + (unsigned)fr * 16u + so;
            if (u.lp == ND - 1 && wr * 128 + fr * 8 >= LW) vo |= 0x80000000u;
#pragma unroll
            for (int bj = 0; bj < 2; ++bj)
#pragma unroll
                for (int n = 0; n < 2; ++n)
#pragma unroll
                    for (int r = 0; r < 4; ++r) {
                        u32x4 o;
                        o[0] = cvt_pk_bf16(gelu(acc[0][bj][0][n][r]), gelu(acc[0][bj][1][n][r]));
                        o[1] = cvt_pk_bf16(gelu(acc[0][bj][2][n][r]), gelu(acc[0][bj][3][n][r]));
                        o[2] = cvt_pk_bf16(gelu(acc[1][bj][0][n][r]), gelu(acc[1][bj][1][n][r]));
                        o[3] = cvt_pk_bf16(gelu(acc[1][bj][2][n][r]), gelu(acc[1][bj][3][n][r]));
                        if (bad) o = (u32x4){0x7fc07fc0u, 0x7fc07fc0u, 0x7fc07fc0u, 0x7fc07fc0u};
                        __builtin_amdgcn_raw_buffer_store_b128(o, rO, vo + (unsigned)(bj * 32 + n * 16 + r) * (N_ * 2u), 0, AUX_SC1);
                    }
            if (dist) {
                WAIT_V(0);
                if (lane == 0) __hip_atomic_fetch_add((gu32*)(myws + OFF_DONE), 1u, __ATOMIC_RELAXED, __HIP_MEMORY_SCOPE_AGENT);
            }
        }
        if (wid == 0) STAMP(myws, 33 + 2 * ui);
        if (!has_next) break;
#pragma unroll
        for (int a = 0; a < 2; ++a)
#pragma unroll
            for (int b = 0; b < 2; ++b)
#pragma unroll
                for (int m = 0; m < 4; ++m)
#pragma unroll
                    for (int n = 0; n < 2; ++n) acc[a][b][m][n] = (f32x4){0.f, 0.f, 0.f, 0.f};
        u = nu; ++ui;
    }
    WAIT_V(0);
    if (wr == 0) BAR;
#undef SA
#undef SB
#undef STAGE
#undef LDA
#undef LDB
#undef MMA
#undef READY
}

__global__ __launch_bounds__(512, 2) void fused_gemm_ar(const Params p) {
    __shared__ __attribute__((aligned(16))) unsigned char shm[LDS_BYTES];
    LAS unsigned char* lds = (LAS unsigned char*)shm;
    char* const cw = p.myws + OFF_CALL + blockIdx.x * 4u;
#define ENTRY_STAMP do { } while (0)
    if (__builtin_expect(blockIdx.x < NCOPY, 1)) {
        copier(p, lds, cw);
        ENTRY_STAMP;
        return;
    }
    const unsigned call = uni(ld_agent(cw)) + 1u;
    gemm(p, lds, call);
    asm volatile("s_waitcnt vmcnt(0) lgkmcnt(0)" ::: "memory");
    __builtin_amdgcn_s_barrier();
    if (threadIdx.x == 0) { st_agent(cw, call); STAMP(p.myws, 1); }
    ENTRY_STAMP;
}

}

extern "C" void kernel_launch(int rank, int nranks, ncclComm_t comm, hipStream_t stream,
                              void* const* d_in, const int* in_sizes, int n_in,
                              void* d_out, int out_size, const int* in_dtypes, int out_dtype,
                              void* const* d_out_all, void* const* d_ws_all, size_t ws_bytes) {
    (void)comm; (void)in_dtypes; (void)out_dtype; (void)d_out;
    if (nranks != P_ || (unsigned)rank >= (unsigned)P_ || n_in != 2 || in_sizes[0] != M_ * KS_ || in_sizes[1] != KS_ * N_ ||
        out_size != M_ * N_ || ws_bytes < WS_NEED) {
        fprintf(stderr, "kernel_launch: unexpected problem shape\n");
        return;
    }
    Params p;
    p.x = (const char*)d_in[0]; p.w = (const char*)d_in[1]; p.rank = rank;
    static const unsigned dmap[P_] = {0x71546302u, 0x26307415u, 0x14572360u, 0x14570632u, 0x60237451u, 0x06235147u, 0x57410362u, 0x36201574u};
    p.dmap = dmap[rank];
    for (int i = 0; i < P_; ++i) { p.peer[i].out = (char*)d_out_all[i]; p.peer[i].ws = (char*)d_ws_all[i]; }
    p.myout = p.peer[rank].out; p.myws = p.peer[rank].ws;
    hipLaunchKernelGGL(fused_gemm_ar, dim3(NWG), dim3(512), 0, stream, p);
    HIP_CHECK(hipGetLastError());
}
```

```cpp
#include <hip/hip_runtime.h>
#include <rccl/rccl.h>
#include <cstdio>
#include <cstdint>
#include <type_traits>

#define HIP_CHECK(expr) do { hipError_t _e = (expr); if (_e != hipSuccess) { \
    fprintf(stderr, "HIP error %s at %s:%d\n", hipGetErrorString(_e), __FILE__, __LINE__); \
    return; } } while (0)

namespace {

#define LAS __attribute__((address_space(3)))
typedef short bf16x8 __attribute__((ext_vector_type(8)));
typedef float f32x4 __attribute__((ext_vector_type(4)));
typedef unsigned u32x4 __attribute__((ext_vector_type(4)));
typedef float f32x2 __attribute__((ext_vector_type(2)));
typedef __attribute__((address_space(1))) unsigned gu32;

constexpr int M_ = 4096, K_ = 4096, N_ = 16384, P_ = 8, KS_ = K_ / P_;
constexpr int NXP = M_ / 256;
constexpr int CPD = 3;
constexpr int NCOPY = 8 * CPD;
constexpr int NWG = 256;
constexpr int GPX = NWG / 8 - CPD;
constexpr int NG = 8 * GPX;
constexpr int WIDE = 320, NW0 = GPX / 2;
constexpr int ND = 2, DW = ND * WIDE;
constexpr int COM0 = P_ * DW, COM1 = COM0 + (NW0 - ND) * WIDE;
static_assert(COM1 + (2 * NW0 + 1) * 256 == N_ && GPX == 2 * NW0 + 1, "columns");
constexpr int XW = NXP * 32 + 32, SK0 = XW + NW0 * 40, SK1 = NW0 * 32;
constexpr int XB1 = 8 * SK0, XE = XB1 + 16 * SK1;
constexpr unsigned PSH = 2048;
constexpr int NPUSH = 12;
constexpr int NFLAG = 64;
constexpr int EB0 = SK0 / 8 + 7 * ND * 5;
constexpr int NJOB = XE / 8 + 8 * 7 * ND * 5;
static_assert(NPUSH % 4 == 0 && NPUSH <= GPX && NFLAG - NPUSH <= 2 * GPX && EB0 <= NFLAG + NG && 3 * EB0 <= NFLAG + 3 * NG && NJOB - NFLAG <= 16 * NG && NJOB - NFLAG >= 8 * NG, "jobs");
constexpr int NCW = 4, NI = 3, BLK = NCW * NI, DEPTH = 3;
constexpr int TAIL = 240, HEAD = 512 - TAIL, LC = 5;
constexpr unsigned RP = DW * 2, QS = M_ * RP;
constexpr int TBL = TAIL * RP / (BLK * 1024), HK = HEAD * RP / 1024, NPOOL = 7 * TBL;
constexpr int X0 = 8 * NPUSH, XS = BLK * CPD;
constexpr int NBI = (XE - X0 + BLK - 1) / BLK, NBA = NBI + NPOOL, NB = NBA + (8 * HK + BLK - 1) / BLK;
constexpr int STEPS = (NB + CPD - 1) / CPD;
static_assert(XE / 8 + 7 < PSH - 1 && NPOOL + NCOPY * (LC + 1) < PSH && TAIL * RP == TBL * BLK * 1024 && HEAD * RP == HK * 1024, "blocks");

constexpr size_t OFF_POOL = 256;
constexpr size_t OFF_OUTRDY = 512;
constexpr size_t OFF_STAGED = 1024;
constexpr size_t OFF_BATCH = 1u << 19;
constexpr unsigned BST = 128;
constexpr size_t OFF_RELAY = 5120;
constexpr size_t OFF_TMO = 4096;
constexpr size_t OFF_DONE = 4096 + 256;
constexpr size_t OFF_CALL = 8192;
constexpr size_t OFF_CLAIM = 16384;
constexpr size_t OFF_CNT = 24576;
constexpr size_t OFF_PROG = 32768;
constexpr size_t OFF_HAVE = 36864;
constexpr size_t OFF_IMG = 1u << 20;
constexpr size_t REGION = (size_t)XE * 1024;
constexpr size_t DSZ = (size_t)8 * ND * 40960;
constexpr size_t IMG_BYTES = 8 * REGION + 8 * DSZ;
constexpr size_t OFF_OUTC = OFF_IMG + 2 * IMG_BYTES;
constexpr size_t WS_NEED = OFF_OUTC + (size_t)P_ * QS;
constexpr unsigned DEADLINE_TICKS = 100000000u;
constexpr unsigned OOB = 0x80000000u;

constexpr int HTB = 16384;
constexpr int LDS_BYTES = 9 * HTB + 256;
constexpr int AUX_SYS = 17, AUX_NT = 18, AUX_WB = 2;
constexpr int RSRC3 = 0x00020000;

struct Params {
    const char* x; const char* w;
    char* myout; char* myws;
    int rank;
    unsigned dmap;
    char* ws[P_];
    char* pushws[P_];
};

__device__ __forceinline__ unsigned ld_sys(const char* p) { return __hip_atomic_load((gu32*)p, __ATOMIC_RELAXED, __HIP_MEMORY_SCOPE_SYSTEM); }
__device__ __forceinline__ unsigned ld_agent(const char* p) { return __hip_atomic_load((gu32*)p, __ATOMIC_RELAXED, __HIP_MEMORY_SCOPE_AGENT); }
__device__ __forceinline__ void st_sys(char* p, unsigned v) { __hip_atomic_store((gu32*)p, v, __ATOMIC_RELAXED, __HIP_MEMORY_SCOPE_SYSTEM); }
__device__ __forceinline__ void st_agent(char* p, unsigned v) { __hip_atomic_store((gu32*)p, v, __ATOMIC_RELAXED, __HIP_MEMORY_SCOPE_AGENT); }
__device__ __forceinline__ unsigned uni(unsigned v) { return (unsigned)__builtin_amdgcn_readfirstlane((int)v); }
__device__ __forceinline__ char* uni_ptr(const char* q) {
    const unsigned long long a = (unsigned long long)q;
    return (char*)(((unsigned long long)uni((unsigned)(a >> 32)) << 32) | uni((unsigned)a));
}

__device__ __forceinline__ unsigned wave_wait_ge(const char* lane_word, unsigned need, char* tmo, unsigned call, int nap = 8) {
    const unsigned t0 = (unsigned)__builtin_amdgcn_s_memrealtime();
    for (;;) {
        const unsigned v = ld_sys(lane_word);
        if (__all(v >= need)) return v;
        const unsigned t = (unsigned)__builtin_amdgcn_s_memrealtime() - t0;
        if (t > DEADLINE_TICKS / 1000u && (t > DEADLINE_TICKS || uni(ld_agent(tmo)) == call)) {
            if ((threadIdx.x & 63) == 0) st_agent(tmo, call);
            return 0u;
        }
        if (nap == 8) __builtin_amdgcn_s_sleep(8); else if (nap) __builtin_amdgcn_s_sleep(64);
    }
}

__device__ __forceinline__ bool poll_release(const char* lane_word, unsigned need) {
    unsigned a, b, c, ok;
    const unsigned t0 = (unsigned)__builtin_amdgcn_s_memrealtime();
#define POLL(ld, chk) "s_sleep 3\n\tglobal_load_dword " ld ", %[p], off sc0 sc1\n\ts_waitcnt vmcnt(2)\n\tv_cmp_gt_u32 vcc, %[n], " chk "\n\ts_cbranch_vccz 2f\n\t"
    asm volatile("s_waitcnt vmcnt(0)\n\ts_mov_b32 %[ok], 1\n\t"
                 "global_load_dword %[a], %[p], off sc0 sc1\n\ts_sleep 3\n\tglobal_load_dword %[b], %[p], off sc0 sc1\n"
                 "1:\n\t" POLL("%[c]", "%[a]") POLL("%[a]", "%[b]") POLL("%[b]", "%[c]")
                 "s_memrealtime vcc\n\ts_waitcnt lgkmcnt(0)\n\ts_sub_u32 %[ok], vcc_lo, %[t0]\n\ts_cmp_lt_u32 %[ok], %[d]\n\t"
                 "s_mov_b32 %[ok], 1\n\ts_cbranch_scc1 1b\n\ts_mov_b32 %[ok], 0\n"
                 "2:\n\ts_barrier\n\ts_waitcnt vmcnt(0)"
                 : [a] "=&v"(a), [b] "=&v"(b), [c] "=&v"(c), [ok] "=&s"(ok)
                 : [p] "v"(lane_word), [n] "s"(need), [t0] "s"(t0), [d] "s"((unsigned)DEADLINE_TICKS) : "vcc", "scc", "memory");
#undef POLL
    return ok != 0u;
}

__device__ __forceinline__ unsigned cvt_pk_bf16(float lo, float hi) { unsigned r; asm("v_cvt_pk_bf16_f32 %0, %1, %2" : "=v"(r) : "v"(lo), "v"(hi)); return r; }
constexpr float GK1 = -2.0f * 0.7978845608028654f * 1.4426950408889634f, GK2 = GK1 * 0.044715f;
__device__ __forceinline__ f32x2 gelu2(f32x2 y, float k1) {
    const f32x2 t = y * __builtin_elementwise_fma(y * y, (f32x2){GK2, GK2}, (f32x2){k1, k1});
    const f32x2 d = (f32x2){__builtin_amdgcn_exp2f(t[0]), __builtin_amdgcn_exp2f(t[1])} + 1.0f;
    return y * (f32x2){__builtin_amdgcn_rcpf(d[0]), __builtin_amdgcn_rcpf(d[1])};
}

__device__ __forceinline__ unsigned row_mix(unsigned rb) { return (rb & 3u) * 16u; }

constexpr int wait_vm(int n) { return (n & 15) | ((n >> 4) << 14) | 0x0f70; }
#define CFENCE do { asm volatile("" ::: "memory"); __builtin_amdgcn_sched_barrier(0); } while (0)
#define CBAR do { CFENCE; __builtin_amdgcn_s_barrier(); CFENCE; } while (0)
__device__ __forceinline__ int stage_of(int x) {
    return x < XB1 ? x / SK0 : 8 + (x - XB1) / SK1;
}
__device__ __forceinline__ unsigned y_of(unsigned o, unsigned q) { return o / RP * (N_ * 2u - RP) + o + q * RP; }
__device__ __forceinline__ unsigned tail_of(int r, int v) { return (unsigned)(((r + 1 + v / TBL) & 7) * 512 + HEAD) * RP + (unsigned)(v % TBL) * (BLK * 1024u); }

__device__ __forceinline__ void move_rows(const Params& p, unsigned call, int h) {
    const int lane = threadIdx.x & 63, rank = p.rank;
    const __amdgpu_buffer_rsrc_t rs = __builtin_amdgcn_make_buffer_rsrc(p.myws + OFF_OUTC, 0, P_ * QS, RSRC3);
    const __amdgpu_buffer_rsrc_t rd = __builtin_amdgcn_make_buffer_rsrc(p.myout, 0, M_ * N_ * 2, RSRC3);
    unsigned known = 0;
#pragma unroll 1
    for (int v = h; v < 7 * NPOOL; v += (CPD + 8) * 8 - 1) {
        const int t = v / NPOOL, e = v - t * NPOOL;
        const unsigned q = (rank + 1 + e / TBL) & 7, o = tail_of(rank + t, e % TBL) + (unsigned)lane * 16u, need = call * PSH + XE / 8 + t + 1;
        if (known < need) known = uni(wave_wait_ge(p.myws + OFF_HAVE + (blockIdx.x & 7) * 256u, need, p.myws + OFF_TMO, call, 64));
        u32x4 d[BLK];
#pragma unroll
        for (int i = 0; i < BLK; ++i) d[i] = __builtin_amdgcn_raw_buffer_load_b128(rs, q * QS + o + i * 1024u, 0, AUX_SYS);
        if (!known) d[0][0] = 0x7fc07fc0u;
#pragma unroll
        for (int i = 0; i < BLK; ++i) __builtin_amdgcn_raw_buffer_store_b128(d[i], rd, y_of(o + i * 1024u, q), 0, AUX_NT);
        if (!known) break;
    }
}

__device__ __forceinline__ int puller(const Params& p, LAS unsigned char* lds, unsigned& call) {
    const int j = blockIdx.x >> 3;
    const int rank = p.rank;
    const int wid = (int)uni(threadIdx.x >> 6), lane = threadIdx.x & 63;
    char* const myws = p.myws;
    char* const myout = p.myout;
    LAS unsigned* bc = (LAS unsigned*)lds;
    if (threadIdx.x == 0) {
        unsigned d = (p.dmap >> ((__builtin_amdgcn_s_getreg((20 << 0) | (0 << 6) | ((4 - 1) << 11)) & 7u) * 4u)) & 7u;
        gu32* const slots = (gu32*)(myws + OFF_CLAIM) + j * 8 * 32;
        char* const cw = myws + OFF_CALL + blockIdx.x * 4u;
        const unsigned callv = ld_agent(cw);
        unsigned old = __hip_atomic_fetch_add(slots + d * 32, 1u, __ATOMIC_RELAXED, __HIP_MEMORY_SCOPE_AGENT);
        for (int i = 0; i < 7 && old != callv; ++i) {
            __hip_atomic_fetch_sub(slots + d * 32, 1u, __ATOMIC_RELAXED, __HIP_MEMORY_SCOPE_AGENT);
            d = (d + 1u) & 7u;
            old = __hip_atomic_fetch_add(slots + d * 32, 1u, __ATOMIC_RELAXED, __HIP_MEMORY_SCOPE_AGENT);
        }
        bc[0] = d;
        bc[1] = callv + 1u;
        st_agent(cw, callv + 1u);
    }
    __builtin_amdgcn_s_waitcnt(0xc07f);
    CBAR;
    const int src = (int)(uni(bc[0]) & 7u);
    call = uni(bc[1]);
    if (src == rank) {
        for (unsigned v = call * PSH; j + wid == 0 && v < call * PSH + XE / 8 + 7;) {
            v = wave_wait_ge(myws + OFF_PROG + (unsigned)(((rank + 1 + (lane >> 2) % 7) & 7) * 64 + (lane & 3) % CPD) * 4u, v + 1u, myws + OFF_TMO, call, 0);
#define XMIN(o) v = min(v, (unsigned)__builtin_amdgcn_ds_swizzle((int)v, ((o) << 10) | 31))
            XMIN(16); XMIN(8); XMIN(4); XMIN(2); XMIN(1);
#undef XMIN
            v = uni(v);
            if (v == 0u) break;
            if (lane < 8) st_sys(myws + OFF_HAVE + (unsigned)lane * 256u, v);
        }
        return j * 8 + wid - 1;
    }
    if (wid > NCW) return -1;
    char* const sws = uni_ptr(p.ws[src]);

    if (wid == NCW) {
        char* const tmo = myws + OFF_TMO;
        bool bad = false;
        int k = 0, st_prev = 0;
        auto run_to = [&](int kend) {
            for (; k < kend; ++k) {
                CBAR;
                const int x = min(X0 + (k + 1 - DEPTH) * XS, XE), t = min(max((k + 1 - DEPTH) * CPD - NBI, 0) / TBL, 7), st = stage_of(x) + t;
                if ((st != st_prev || (st >= 1 && st < 8 && x - st * SK0 >= XW + 40)) && !bad && lane == 0) st_sys(myws + OFF_PROG + (unsigned)(src * 64 + j) * 4u, call * PSH + (unsigned)((x >> 3) + t));
                st_prev = st;
            }
        };
        auto gate = [&](int kg, const char* word, unsigned need) {
            run_to(kg - 3);
            const unsigned v = ld_sys(word);
            run_to(kg);
            if (!__all(v >= need) && uni(wave_wait_ge(word, need, tmo, call)) == 0u) bad = true;
        };
        constexpr int KG1 = (8 * NFLAG - X0) / XS - DEPTH, KG2 = (SK0 - X0) / XS - DEPTH, KG3 = (3 * SK0 - X0) / XS - DEPTH, KG4 = NBI / CPD - DEPTH;
        static_assert(0 <= KG1 && X0 >= (DEPTH - 1) * XS, "gates");
        const char* const batch = myws + OFF_BATCH + (unsigned)src * (64u * BST);
        const char* const staged = myws + OFF_STAGED + (unsigned)src * 128u + (unsigned)(lane & 7) * 4u;
        if (!poll_release(batch + (unsigned)(lane < NPUSH ? NFLAG - 1 : lane) * BST, call * PSH)) { bad = true; if (lane == 0) st_agent(tmo, call); }
        gate(KG1, staged, call * PSH);
        gate(KG2, lane < NPUSH ? batch + (unsigned)lane * BST : staged + 32, call * PSH);
        gate(KG3, staged + 64, call * PSH);
        if (lane == 0) __hip_atomic_fetch_max((gu32*)(myws + OFF_POOL), call * PSH, __ATOMIC_RELAXED, __HIP_MEMORY_SCOPE_AGENT);
        char* const pws = p.ws[lane & 7];
        gate(KG4, myws + OFF_OUTRDY + (unsigned)src * 4u, call);
        run_to((NBA - 1) / CPD + DEPTH + 1);
        if (!bad && lane < P_) st_sys(pws + OFF_RELAY + (unsigned)(rank * 32 + src * 4 + j) * 4u, call);
        constexpr int KP = STEPS - DEPTH - 1;
        gate(KP - LC, myws + OFF_RELAY + (unsigned)(src * 32 + ((src + 1 + (lane >> 2) % 7) & 7) * 4 + (lane & 3) % CPD) * 4u, call);
        unsigned c[LC];
        const __amdgpu_buffer_rsrc_t pool = __builtin_amdgcn_make_buffer_rsrc(myws + OFF_POOL, 0, 4, RSRC3);
        bool live = true;
        auto claim = [&](int i) { c[i] = (unsigned)__builtin_amdgcn_raw_ptr_buffer_atomic_add_i32(1, pool, lane * 4, live ? 0 : 4, 0); };
#pragma unroll
        for (int i = 0; i < LC; ++i) { CBAR; claim(i); }
        for (unsigned t = 0; live;) {
#pragma unroll
            for (int i = 0; i < LC; ++i) {
                const unsigned v = uni(c[i]) - call * PSH;
                if (live) {
                    bc[2 + (t ^= 1u)] = v;
                    __builtin_amdgcn_s_waitcnt(0xc07f);
                    CBAR;
                    live = v < (unsigned)NPOOL;
                }
                claim(i);
            }
        }
        if (bad && lane == 0) { st_agent(tmo, call); *(unsigned short*)(myout + (size_t)src * DW * 2) = 0x7fc0u; }
        return -1;
    }

    const unsigned par = (unsigned)OFF_IMG + (call & 1u) * (unsigned)IMG_BYTES;
    const unsigned img = par + (unsigned)src * (unsigned)REGION;
    const unsigned dist = (unsigned)(8 - src) * (unsigned)REGION + (unsigned)rank * (unsigned)DSZ - XW * 1024u;
    const int rot = ((src - rank - 1) & 7) * 80;
    const unsigned lin = (unsigned)(wid * 1024 + lane * 16);
    auto rsrc = [](const char* base, unsigned bytes) { return (u32x4){(unsigned)(size_t)base, (unsigned)((size_t)base >> 32) & 0xffffu, bytes, (unsigned)RSRC3}; };
    const u32x4 rs = rsrc(sws, WS_NEED), rw = rsrc(myws, WS_NEED), ry = rsrc(myout, M_ * N_ * 2);
    u32x4 R[DEPTH][NI]; unsigned c[DEPTH][NI]; bool out[DEPTH];
    unsigned sa[NI], da[NI]; bool no = false;
    int b = j, idle = 0; unsigned tog = 0;
    auto next = [&](auto early) {
        if (early || b < NBI) {
#pragma unroll
            for (int i = 0; i < NI; ++i) {
                const int x = X0 + b * BLK + i * NCW, xi = x + wid, g = xi / SK0, o = xi - g * SK0 - XW;
                const int r = early || xi >= XB1 || o < 0 ? 0 : o < rot ? NW0 * 40 - rot : -rot;
                const unsigned d = early || xi < XE ? img + (unsigned)(x + r) * 1024u : OOB;
                sa[i] = lin + (!early && xi < XB1 && (unsigned)(o + r) < ND * 40u ? d + dist - (unsigned)g * ((SK0 - ND * 40) * 1024u) : d);
                da[i] = lin + d;
            }
        } else {
            unsigned q = (unsigned)src, o = 0; bool ok = true;
            no = b >= NBA;
            if (!no) o = tail_of(rank, b - NBI);
            else if (b >= STEPS * CPD) {
                const int v = idle ? NPOOL : (int)uni(bc[2 + (tog ^= 1u)]);
                ok = (unsigned)v < (unsigned)NPOOL;
                q = (unsigned)(rank + 1 + v / TBL) & 7u; o = tail_of(rank - 1, v % TBL);
                if (!ok) ++idle;
            }
#pragma unroll
            for (int i = 0; i < NI; ++i) {
                unsigned oi = o + i * NCW * 1024u;
                if (no && b < STEPS * CPD) {
                    const int v = (b - NBA) * BLK + i * NCW;
                    ok = v + wid < 8 * HK;
                    oi = (unsigned)(v + (v + wid) / HK * TBL * BLK) * 1024u;
                }
                sa[i] = lin + (ok ? (unsigned)OFF_OUTC + q * QS + oi : OOB);
                da[i] = no && ok ? y_of(lin + oi, q) : sa[i];
            }
        }
        b += CPD;
    };
    auto load = [&](int s) {
#pragma unroll
        for (int i = 0; i < NI; ++i) asm volatile("buffer_load_dwordx4 %0, %1, %2, 0 offen sc0 sc1" : "+v"(R[s][i]) : "v"(sa[i]), "s"(rs) : "memory");
        CFENCE;
#pragma unroll
        for (int i = 0; i < NI; ++i) c[s][i] = da[i];
        out[s] = no;
    };
    auto step = [&](int s, auto wait, auto stream, auto early) {
        u32x4 rd = out[s] ? ry : rw;
#pragma unroll
        for (int i = 0; i < NI; ++i) asm volatile("" : "+v"(sa[i]), "+v"(da[i]), "+v"(c[s][i]));
        if (stream) asm volatile("s_cmp_lg_u32 %1, 0\n\ts_waitcnt vmcnt(%2)\n\ts_cbranch_scc1 1f\n\ts_barrier\n1:" : "+s"(rd) : "s"(uni(idle)), "i"(decltype(wait)::value) : "memory", "scc");
        else asm volatile("s_waitcnt vmcnt(%1)\n\ts_barrier" : "+s"(rd) : "i"(decltype(wait)::value) : "memory");
        CFENCE;
#pragma unroll
        for (int i = 0; i < NI; ++i) {
            if (stream) asm volatile("buffer_store_dwordx4 %0, %1, %2, 0 offen sc1 nt" :: "v"(R[s][i]), "v"(c[s][i]), "s"(rd) : "memory");
            else asm volatile("buffer_store_dwordx4 %0, %1, %2, 0 offen sc0 sc1" :: "v"(R[s][i]), "v"(c[s][i]), "s"(rd) : "memory");
        }
        load(s);
        next(early);
    };
    const std::true_type yes; const std::false_type nay;
    static_assert(X0 + (2 * DEPTH + 1) * XS <= 32 * NXP && DEPTH == 3, "early steps");
    unsigned sb[DEPTH][NI];
#pragma unroll
    for (int s = 0; s < DEPTH; ++s) {
        next(yes);
#pragma unroll
        for (int i = 0; i < NI; ++i) { sb[s][i] = sa[i]; asm volatile("" : "+v"(sb[s][i]), "=v"(R[s][i])); }
    }
    __builtin_amdgcn_s_waitcnt(0x0070);
    CBAR;
#pragma unroll
    for (int s = 0; s < DEPTH; ++s) {
#pragma unroll
        for (int i = 0; i < NI; ++i) sa[i] = da[i] = sb[s][i];
        load(s);
    }
    next(yes);
    step(0, std::integral_constant<int, NI * (DEPTH - 1)>(), nay, yes);
    step(1, std::integral_constant<int, NI * DEPTH>(), nay, yes);
    step(2, std::integral_constant<int, NI * (DEPTH + 1)>(), nay, yes);
    const std::integral_constant<int, 2 * NI * (DEPTH - 1)> wait;
    constexpr int KOUT = (NBA + CPD * DEPTH - 1) / (CPD * DEPTH);
#pragma unroll 1
    for (int k = 1; k < KOUT; ++k) {
#pragma unroll
        for (int s = 0; s < DEPTH; ++s) step(s, wait, nay, nay);
    }
#pragma unroll 1
    while (idle <= DEPTH) {
#pragma unroll
        for (int s = 0; s < DEPTH; ++s) step(s, wait, yes, nay);
    }
    asm volatile("s_waitcnt vmcnt(0)" ::: "memory");
    return -1;
}

__device__ __forceinline__ unsigned stage_in(const Params& p) {
    unsigned long long t0 = __builtin_amdgcn_s_memrealtime();
    const int wid = (int)uni(threadIdx.x >> 6), lane = threadIdx.x & 63;
    const int q = blockIdx.x & 7, gi = (int)(blockIdx.x >> 3) - CPD;
    const char* px = p.x; const char* pw = p.w; char* myws = p.myws; char* to_ws = p.pushws[q]; int rank = p.rank;
    char* pws = p.ws[lane & 7];
    __builtin_amdgcn_sched_barrier(0);
    int ebp = gi < NPUSH && wid == 0 ? gi : -1, ebh = NPUSH + 2 * gi + wid < NFLAG && wid < 2 ? NPUSH + 2 * gi + wid : -1;
    unsigned xa[8];
#pragma unroll
    for (int i = 0; i < 8; ++i) { xa[i] = (((unsigned)(lane >> 3) * (KS_ * 2u) + (unsigned)((lane & 7) ^ (lane >> 4)) * 16u) ^ ((i & 1) * 64u)) + (unsigned)i * (8u * KS_ * 2u); asm volatile("" : "+v"(xa[i])); }
    asm volatile("" : "+s"(ebp), "+s"(ebh));
    __builtin_amdgcn_sched_barrier(0);
    asm volatile("" : "+s"(px), "+s"(pw), "+s"(myws), "+s"(to_ws), "+s"(rank), "+s"(t0));
    char* const cw = myws + OFF_CALL + blockIdx.x * 4u;
    unsigned callv = ld_agent(cw);
    gu32* const cnt = (gu32*)(myws + OFF_CNT) + q * 32;
    unsigned call = 0;
    auto arrive = [&](int part, unsigned n) {
        unsigned old = 0;
        if (lane == 0) old = __hip_atomic_fetch_add(cnt + part * 256, 1u, __ATOMIC_RELAXED, __HIP_MEMORY_SCOPE_AGENT);
        if (uni(old) + 1u == call * n && lane < P_) st_sys(pws + OFF_STAGED + (unsigned)(rank * 32 + part * 8 + q) * 4u, call * PSH + PSH - 1u);
    };
    auto wait_until = [&](int ticks) { while ((int)(unsigned)(__builtin_amdgcn_s_memrealtime() - t0) < ticks) __builtin_amdgcn_s_sleep(1); };
    auto load_x = [&](u32x4 (&w)[8], int row0, int g) {
        const __amdgpu_buffer_rsrc_t rs = __builtin_amdgcn_make_buffer_rsrc((void*)px, 0, M_ * KS_ * 2, RSRC3);
        const unsigned so = (unsigned)row0 * (KS_ * 2u) + (unsigned)g * 128u;
#pragma unroll
        for (int i = 0; i < 8; ++i) w[i] = __builtin_amdgcn_raw_buffer_load_b128(rs, xa[i] ^ (row_mix((unsigned)row0 >> 8) * (KS_ * 2u)), so, 0);
    };
    auto store = [&](const u32x4 (&w)[8], char* dws, unsigned c, int sub) {
        const __amdgpu_buffer_rsrc_t rd = __builtin_amdgcn_make_buffer_rsrc(dws, 0, (int)OFF_OUTC, RSRC3);
        c += (unsigned)OFF_IMG + (call & 1u) * (unsigned)IMG_BYTES + (unsigned)(sub >> 1) * 8192u + (unsigned)(sub & 1) * 1024u;
#pragma unroll
        for (int i = 0; i < 8; ++i) __builtin_amdgcn_raw_buffer_store_b128(w[i], rd, c + (unsigned)(sub < 4 ? (i >> 2) * 16384 + (i & 3) * 2048 : 16384 + i * 1024), 0, AUX_SYS);
    };
    {
        const bool push = to_ws != myws;
        const int eb = push ? ebp : ebh;
        if (eb >= 0) {
            u32x4 w[8];
            load_x(w, eb * 64, 0);
            const unsigned dof = (unsigned)lane * 16u + (unsigned)rank * (unsigned)REGION + (unsigned)(eb >> 2) * 32768u;
            call = uni(callv) + 1u;
            store(w, to_ws, dof, eb & 3);
            CFENCE; __builtin_amdgcn_s_waitcnt(wait_vm(0)); CFENCE;
            if (lane < P_ && pws != myws && (!push || pws == to_ws)) st_sys(pws + OFF_BATCH + (unsigned)(rank * 64 + eb) * BST, call * PSH);
            if (p.pushws[q ^ 1] == myws) store(w, myws, dof, eb & 3);
        }
    }
    auto job = [&](int row) {
        const int eb = NFLAG + row * NG + (int)blockIdx.x - NCOPY;
        if (eb >= NJOB) return;
        wait_until(row ? 200 + row * 300 + gi * 83 / 8 : 150 + gi * 5);
        int g, sub, col = -1, e; unsigned dof = (unsigned)rank * (unsigned)REGION;
        if (eb < 8 * EB0) {
            g = eb / EB0; e = eb - g * EB0; sub = e & 3;
            if (e < XW / 8) {
                dof += (unsigned)(g * SK0 + (e >> 2) * 32) * 1024u;
                if (e >= 4 * NXP) col = COM1;
            } else {
                const int lp = (e - XW / 8) / 5;
                sub = e - XW / 8 - lp * 5;
                col = (lp < ND ? rank * DW : COM0 - ND * WIDE) + lp * WIDE;
                dof += (unsigned)(g * SK0 + XW + lp * 40) * 1024u;
                if (lp >= NW0) {
                    const int own = (rank + 1 + (lp - NW0) / ND) & 7, l = (lp - NW0) % ND;
                    col = own * DW + l * WIDE;
                    dof = 8u * (unsigned)REGION + (unsigned)own * (unsigned)DSZ + (unsigned)(g * ND + l) * 40960u;
                }
            }
        } else {
            e = eb - 8 * EB0; sub = e & 3;
            const int st = (e >> 2) / NW0;
            g = st & 7; col = COM1 + (1 + (e >> 2) - (st - (st >> 3)) * NW0) * 256;
            dof += (unsigned)(XB1 + (e >> 2) * 32) * 1024u;
        }
        u32x4 v[8], w[8];
        const __amdgpu_buffer_rsrc_t rs = __builtin_amdgcn_make_buffer_rsrc((void*)pw, 0, KS_ * N_ * 2, RSRC3);
        const unsigned a = (unsigned)(g * 64 + (lane >> 4) * 8) * (N_ * 2u) + (unsigned)col * 2u;
        if (col < 0) {
            load_x(w, e * 64, g);
        } else {
            if (sub < 4) {
#pragma unroll
                for (int i = 0; i < 8; ++i) v[i] = __builtin_amdgcn_raw_buffer_load_b128(rs, a + (unsigned)((lane & 15) * 16 + (sub >> 1) * 256) + (unsigned)((sub & 1) * 32 + i) * (N_ * 2u), 0, 2);
            } else {
#pragma unroll
                for (int i = 0; i < 32; ++i) v[i & 7][i >> 3] = __builtin_amdgcn_raw_buffer_load_b32(rs, a + (unsigned)((lane & 15) * 4 + 512 + (i >> 4) * 64) + (unsigned)((i & 8) * 4 + (i & 7)) * (N_ * 2u), 0, 0);
            }
#pragma unroll
            for (int i = 0; i < 8; ++i)
#pragma unroll
                for (int e = 0; e < 4; ++e)
                    w[i][e] = __builtin_amdgcn_perm(v[2 * e + 1][i >> 1], v[2 * e][i >> 1], (i & 1) ? 0x07060302u : 0x05040100u);
        }
        if (row < 8) { asm volatile("" : "+v"(callv), "+v"(pws)); call = uni(callv) + 1u; }
        store(w, myws, (unsigned)lane * 16u + dof, sub);
        if (row < 3) {
            CFENCE; __builtin_amdgcn_s_waitcnt(wait_vm(0)); CFENCE;
            arrive(row ? 1 : 0, row ? 2 * GPX : GPX);
        }
    };
    job((wid + 6) & 7);
    job(8 + ((wid + 6) & 7));
    CFENCE; __builtin_amdgcn_s_waitcnt(wait_vm(0));
    CBAR;
    if (wid == 7) { arrive(2, GPX); if (lane == 0) st_agent(cw, call); }
    return call;
}
#undef CBAR
#undef CFENCE

struct Unit { int rb, col, x, sk, end; };
__device__ __forceinline__ Unit unit_of(int r, int q, int i) {
    const int lp = (q & 1) * (NW0 / 2) + (i >> 2), rb = (q >> 1) * 4 + (i & 3);
    if (i == 2 * NW0) return {q * 2 + r - 1, COM1, NXP * 32, SK0, SK0};
    if (r == 0) return {rb, (lp < ND ? 0 : COM0) + (lp - ND) * WIDE, (int)uni(XW + lp * 40), SK0, SK0};
    return {rb, COM1 + (1 + (r - 1) * NW0 + lp) * 256, (int)uni(XB1 + (r - 1) * 8 * SK1 + lp * 32), SK1, (int)uni(XB1 + (r * 8 - 7) * SK1)};
}

__device__ __forceinline__ void gemm(const Params& p, LAS unsigned char* lds, unsigned call) {
    const int tid = threadIdx.x, wid = (int)uni(tid >> 6), lane = tid & 63, wr = wid >> 2, wc = wid & 3;
    const int q = blockIdx.x & 7, gi = (blockIdx.x >> 3) - CPD;
    const int rank = p.rank;
    char* const myws = p.myws;
    const __amdgpu_buffer_rsrc_t img = __builtin_amdgcn_make_buffer_rsrc(myws + OFF_IMG + (size_t)(call & 1u) * IMG_BYTES, 0, 8 * REGION, RSRC3);
    const unsigned voff = (unsigned)tid * 16u;
    const unsigned ldsw = (unsigned)wid * 1024u;
    const int aoff = wr * 8192 + lane * 16, xoff = 8 * HTB + wr * 4096 + lane * 16;
    int boff0 = 4 * HTB + (wc >> 1) * 8192 + (wc & 1) * 1024 + ((lane >> 3) & 1) * 2048 + (lane & 7) * 128 + (((lane >> 4) ^ ((lane >> 1) & 7)) * 16);
    int boff1 = boff0 ^ 64;
    asm volatile("" : "+v"(boff0), "+v"(boff1));
    LAS unsigned* bc = (LAS unsigned*)(lds + 9 * HTB);
    unsigned known = 0, k1 = __float_as_uint(GK1);
    constexpr unsigned hstep = 16384;
    constexpr int nt = K_ / 64;

#define SA(b, h) (((b) * 2 + (h)) * HTB)
#define SB(b, h) ((4 + (b) * 2 + (h)) * HTB)
#define STAGE(bufoff, goff) do { _Pragma("unroll") for (int _i = 0; _i < 2; ++_i) \
        __builtin_amdgcn_raw_ptr_buffer_load_lds(img, (LAS unsigned*)(lds + (bufoff) + ldsw + _i * 8192), 16, voff, (goff) + _i * 8192, 0, AUX_SYS); } while (0)
#define STAGEX(b, goff) do { if (MM == 5) __builtin_amdgcn_raw_ptr_buffer_load_lds(img, (LAS unsigned*)(lds + 8 * HTB + (b) * 8192 + ldsw), 16, voff, goff, 0, AUX_SYS); } while (0)
#define LDA(dst, b, h) do { _Pragma("unroll") for (int m = 0; m < 4; ++m) _Pragma("unroll") for (int k = 0; k < 2; ++k) dst[m][k] = *(const LAS bf16x8*)(lds + SA(b, h) + aoff + m * 2048 + k * 1024); \
        if (MM == 5) { _Pragma("unroll") for (int k = 0; k < 2; ++k) dst[4][k] = *(const LAS bf16x8*)(lds + xoff + (b) * 8192 + k * 2048 + (h) * 1024); } } while (0)
#define LDB(dst, b, h) do { _Pragma("unroll") for (int n = 0; n < 2; ++n) _Pragma("unroll") for (int k = 0; k < 2; ++k) dst[n][k] = *(const LAS bf16x8*)(lds + SA(b, h) + (k ? boff1 : boff0) + n * 4096); } while (0)
#define MMA(ai, bj, At, Bt) do { __builtin_amdgcn_s_setprio(1); _Pragma("unroll") for (int m = 0; m < MM; ++m) _Pragma("unroll") for (int n = 0; n < 2; ++n) _Pragma("unroll") for (int k = 0; k < 2; ++k) \
        acc[ai][bj][m][n] = __builtin_amdgcn_mfma_f32_16x16x32_bf16(Bt[n][k], At[m][k], acc[ai][bj][m][n], 0, 0, 0); __builtin_amdgcn_s_setprio(0); } while (0)
#define WAIT_V(n) asm volatile("s_waitcnt vmcnt(" #n ")" ::: "memory")
#define WAIT_L(n) do { asm volatile("" ::: "memory"); __builtin_amdgcn_s_waitcnt(0xc07f | ((n) << 8)); asm volatile("" ::: "memory"); } while (0)
#define BAR __builtin_amdgcn_s_barrier()
#define SCHED __builtin_amdgcn_sched_barrier(0)
#define READY(need, first) do { if (__builtin_expect((need) > known, 0)) { \
        if (wid == 0) { \
            bc[0] = uni(wave_wait_ge(myws + ((first) ? OFF_STAGED + (unsigned)(rank * 32 + 16 + (lane & 7)) * 4u : OFF_HAVE + (unsigned)q * 256u), need, myws + OFF_TMO, call)); \
            asm volatile("s_waitcnt vmcnt(0) lgkmcnt(0)" ::: "memory"); \
        } \
        asm volatile("" ::: "memory"); BAR; asm volatile("" ::: "memory"); \
        known = uni(bc[0]); if (known == 0u) { k1 = 0x7fc00000u; known = 0xffffffffu; } \
        asm volatile("s_waitcnt lgkmcnt(0)" ::: "memory"); } } while (0)

    int ui = gi < 2 * NW0 ? 0 : 1;
    Unit u = unit_of(ui, q, gi);
    f32x4 acc[2][2][5][2];
    bf16x8 At[5][2], B0[2][2], B1[2][2];
    unsigned seq = rank, sh = 0, lim = 4, ga = u.x * 1024u, gb = u.rb * 32768u, da = u.sk * 1024u, gn = 0, dn = 0, wn = 0;
    auto locate = [&](unsigned& a, unsigned& b, unsigned& need) {
        unsigned o;
        asm("s_mul_i32 %0, %1, %2" : "=s"(o) : "s"((seq >> sh) & 7u), "s"((unsigned)REGION));
        a = o + ga; b = o + gb; need = gn;
        sh += 4u;
        const bool wrap = sh == lim;
        sh = wrap ? 0u : sh; ga += wrap ? da : 0u; gb += wrap ? SK0 * 1024u : 0u; gn += wrap ? wn : dn;
        asm volatile("" : "+s"(a), "+s"(b), "+s"(need), "+s"(sh), "+s"(ga), "+s"(gb), "+s"(gn));
    };
    unsigned a1, a2, b2, b3, n2, n3;
    READY(call * PSH, true);
    if (known != 0xffffffffu) known = call * PSH;
    locate(a2, b2, n2); locate(a1, b3, n3);
    { constexpr int MM = 5; STAGEX(0, a2 + (ui ? 0 : 2 * hstep)); }
    STAGE(SB(0, 0), b2); STAGE(SA(0, 0), a2); STAGE(SB(0, 1), b2 + hstep); STAGE(SA(0, 1), a2 + hstep);
    if (wr == 1) BAR;
    WAIT_V(4); BAR;
    STAGE(SB(1, 0), b3); STAGE(SA(1, 0), a1); STAGE(SB(1, 1), b3 + hstep);
    locate(a2, b2, n2);
    WAIT_V(6); BAR;
    auto tile = [&](auto mm) {
        constexpr int MM = decltype(mm)::value;
        const Unit nu = unit_of(ui < 2 ? ui + 1 : ui, q, gi);
#pragma unroll
        for (int a = 0; a < 2; ++a)
#pragma unroll
            for (int b = 0; b < 2; ++b)
#pragma unroll
                for (int m = 0; m < MM; ++m)
#pragma unroll
                    for (int n = 0; n < 2; ++n) acc[a][b][m][n] = (f32x4){0.f, 0.f, 0.f, 0.f};
        for (int t = 0; t < nt; t += 2) {
            LDB(B0, 0, 0); SCHED; LDA(At, 0, 0); STAGE(SA(1, 1), a1 + hstep); STAGEX(1, a1 + 2 * hstep);
            WAIT_L(2 * MM); BAR; WAIT_L(0); MMA(0, 0, At, B0); BAR; SCHED;
            READY(n2, false);
            LDB(B1, 0, 1); STAGE(SB(0, 0), b2);
            BAR; WAIT_L(0); MMA(0, 1, At, B1); BAR;
            LDA(At, 0, 1); STAGE(SA(0, 0), a2);
            BAR; WAIT_L(0); MMA(1, 0, At, B0); BAR; SCHED;
            STAGE(SB(0, 1), b2 + hstep); SCHED;
            locate(a1, b3, n3);
            WAIT_V(6); BAR; MMA(1, 1, At, B1); BAR;
            LDB(B0, 1, 0); SCHED; LDA(At, 1, 0); STAGE(SA(0, 1), a2 + hstep); STAGEX(0, a2 + (t == nt - 2 ? 0 : 2 * hstep));
            WAIT_L(2 * MM); BAR; WAIT_L(0); MMA(0, 0, At, B0); BAR; SCHED;
            READY(n3, false);
            LDB(B1, 1, 1); STAGE(SB(1, 0), b3);
            BAR; WAIT_L(0); MMA(0, 1, At, B1); BAR;
            LDA(At, 1, 1); STAGE(SA(1, 0), a1);
            BAR; WAIT_L(0); MMA(1, 0, At, B0); BAR; SCHED;
            STAGE(SB(1, 1), b3 + hstep); SCHED;
            if (__builtin_expect(t == 4, 0)) {
                const int lp = (u.x - XW) / 40;
                seq = 0;
                for (int i = 6; i >= 0; --i) seq = seq * 16u + ((rank + 1 + ((MM == 5 ? 7 - (lp >> 1) : 0) + i) % 7) & 7);
                seq = uni(seq); lim = 28; ga = u.x * 1024u; gb = u.rb * 32768u;
                gn = uni(call * PSH + (MM == 5 ? XW / 8 + 5 * (lp & 1) + 5 : u.end / 8)); dn = MM == 5 ? 10 : 0; wn = u.sk / 8 - 6 * dn;
            } else if (__builtin_expect(t == nt - 4, 0)) {
                seq = rank; lim = 4; ga = nu.x * 1024u; gb = nu.rb * 32768u; da = nu.sk * 1024u; gn = dn = wn = 0;
            }
            locate(a2, b2, n2);
            WAIT_V(6); BAR; MMA(1, 1, At, B1); BAR;
        }
        if (MM == 5 && wr == 0) {
            const unsigned t0 = (unsigned)__builtin_amdgcn_s_memrealtime();
            while ((unsigned)__builtin_amdgcn_s_memrealtime() - t0 < (unsigned)((gi & 3) * 50)) __builtin_amdgcn_s_sleep(4);
        }
        int tz = threadIdx.x; asm volatile("" : "+v"(tz));
        const int fr = tz & 15, row = u.rb * 256 + wc * 64 + ((tz >> 2) & 12);
        const bool dist = MM == 5 && u.col < 0;
        unsigned mix = row_mix(u.rb); asm volatile("" : "+s"(mix));
        const __amdgpu_buffer_rsrc_t rO = __builtin_amdgcn_make_buffer_rsrc(p.myout, 0, M_ * N_ * 2, RSRC3);
        const __amdgpu_buffer_rsrc_t rC = __builtin_amdgcn_make_buffer_rsrc(myws + OFF_OUTC + (unsigned)rank * QS, 0, QS, RSRC3);
        const unsigned vo = (unsigned)row * (N_ * 2u) + (unsigned)(u.col + (dist ? (rank + 1) * DW : 0)) * 2u, vc = (unsigned)row * RP + (unsigned)(u.col + DW) * 2u;
        const unsigned c8 = (unsigned)(wr * 256 + fr * 16), c2 = (unsigned)(512 + wr * 64 + fr * 4);
#pragma unroll
        for (int bj = 0; bj < 2; ++bj)
#pragma unroll
            for (int n = 0; n < 2; ++n)
#pragma unroll
                for (int h = 0; h < 2; ++h) {
                    f32x2 y[2][MM];
#pragma unroll
                    for (int a = 0; a < 2; ++a)
#pragma unroll
                        for (int m = 0; m < MM; ++m) y[a][m] = gelu2(h ? acc[a][bj][m][n].hi : acc[a][bj][m][n].lo, __uint_as_float(k1));
#pragma unroll
                    for (int r = 0; r < 2; ++r) {
                        const u32x4 o = {cvt_pk_bf16(y[0][0][r], y[0][1][r]), cvt_pk_bf16(y[0][2][r], y[0][3][r]), cvt_pk_bf16(y[1][0][r], y[1][1][r]), cvt_pk_bf16(y[1][2][r], y[1][3][r])};
                        const unsigned rr = (unsigned)(bj * 32 + n * 16 + h * 2 + r) ^ mix;
                        __builtin_amdgcn_raw_buffer_store_b128(o, rO, vo + c8 + rr * (N_ * 2u), 0, MM == 5 ? AUX_WB : AUX_NT);
                        if (MM == 5) {
                            const unsigned e = cvt_pk_bf16(y[0][MM - 1][r], y[1][MM - 1][r]);
                            __builtin_amdgcn_raw_buffer_store_b32(e, rO, vo + c2 + rr * (N_ * 2u), 0, AUX_WB);
                            if (dist) {
                                __builtin_amdgcn_raw_buffer_store_b128(o, rC, vc + c8 + rr * RP, 0, AUX_SYS);
                                __builtin_amdgcn_raw_buffer_store_b32(e, rC, vc + c2 + rr * RP, 0, AUX_SYS);
                            }
                        }
                    }
                }
        if (dist) {
            WAIT_V(0);
            unsigned old = 0;
            if (lane == 0) old = __hip_atomic_fetch_add((gu32*)(myws + OFF_DONE), 1u, __ATOMIC_RELAXED, __HIP_MEMORY_SCOPE_AGENT);
            if (uni(old) + 1u == call * (unsigned)(NXP * ND * 8) && lane < P_) st_sys(p.ws[lane] + OFF_OUTRDY + (unsigned)rank * 4u, call);
        }
        u = nu; ++ui;
    };
    if (ui == 0) tile(std::integral_constant<int, 5>());
#pragma unroll 1
    while (ui < 3) tile(std::integral_constant<int, 4>());
    if (wr == 0) BAR;
    WAIT_V(0);
#undef SA
#undef SB
#undef STAGE
#undef STAGEX
#undef LDA
#undef LDB
#undef MMA
#undef READY
}

__global__ __launch_bounds__(512, 2) void fused_gemm_ar(const Params p) {
    __shared__ __attribute__((aligned(16))) unsigned char shm[LDS_BYTES];
    LAS unsigned char* lds = (LAS unsigned char*)shm;
    unsigned call; int h;
    if (blockIdx.x < NCOPY) {
        h = puller(p, lds, call);
    } else {
        call = stage_in(p);
        gemm(p, lds, call);
        h = blockIdx.x < NWG - 8 ? -1 : (int)(blockIdx.x - (NWG - 8) + CPD) * 8 + (int)uni(threadIdx.x >> 6) - 1;
    }
    if (h >= 0) move_rows(p, call, h);
}

}

extern "C" void kernel_launch(int rank, int nranks, ncclComm_t comm, hipStream_t stream,
                              void* const* d_in, const int* in_sizes, int n_in,
                              void* d_out, int out_size, const int* in_dtypes, int out_dtype,
                              void* const* d_out_all, void* const* d_ws_all, size_t ws_bytes) {
    (void)comm; (void)in_dtypes; (void)out_dtype; (void)d_out_all;
    if (nranks != P_ || (unsigned)rank >= (unsigned)P_ || n_in != 2 || in_sizes[0] != M_ * KS_ || in_sizes[1] != KS_ * N_ ||
        out_size != M_ * N_ || ws_bytes < WS_NEED) {
        fprintf(stderr, "kernel_launch: unexpected problem shape\n");
        return;
    }
    Params p;
    p.x = (const char*)d_in[0]; p.w = (const char*)d_in[1]; p.rank = rank;
    static const unsigned dmap[P_] = {0x71546302u, 0x26307415u, 0x14572360u, 0x14570632u, 0x60237451u, 0x06235147u, 0x57410362u, 0x36201574u};
    p.dmap = dmap[rank];
    for (int i = 0; i < P_; ++i) p.ws[i] = (char*)d_ws_all[i];
    for (int i = 0; i < P_; ++i) p.pushws[i] = p.ws[(p.dmap >> (4 * i)) & 7u];
    p.myout = (char*)d_out; p.myws = p.ws[rank];
    hipLaunchKernelGGL(fused_gemm_ar, dim3(NWG), dim3(512), 0, stream, p);
    HIP_CHECK(hipGetLastError());
}
```
